```python
import numpy as np

kernel_name = "gpu_gemm_ar_m4096_k4096_n32768_f32_none"


ABSMAX_RTOL = 0.001

COMPUTE_DTYPE = "float32"
EPILOGUE = "none"
IN_QUANT = None
OUT_QUANT = None
IN_DTYPES = {"x": "float32", "w_mat": "float32"}
WORLD_SIZE = 8

def setup_inputs():

    x = np.random.randn(4096, 4096).astype(np.float32)
    w_mat = (np.random.randn(4096, 32768) / np.sqrt(4096)).astype(np.float32)
    return {"x": x, "w_mat": w_mat}

def _epilogue(y):
    return y

def reference(x, w_mat):
    y = (x @ w_mat).astype(np.float32)
    return _epilogue(y)
```

```cpp
#include <hip/hip_runtime.h>
#include <rccl/rccl.h>
#include <cstdio>

#define HIP_CHECK(expr) do { hipError_t _e = (expr); if (_e != hipSuccess) { \
    fprintf(stderr, "HIP error %s at %s:%d\n", hipGetErrorString(_e), __FILE__, __LINE__); \
    return; } } while (0)

namespace {

#define LAS __attribute__((address_space(3)))
#define GAS __attribute__((address_space(1)))
typedef _Float16 f16x8 __attribute__((ext_vector_type(8)));
typedef _Float16 f16x2 __attribute__((ext_vector_type(2)));
typedef float f32x4 __attribute__((ext_vector_type(4)));
typedef unsigned u32x4 __attribute__((ext_vector_type(4)));
typedef unsigned long long u64;
typedef GAS unsigned gu32;
typedef GAS u64 gu64;

constexpr int M = 4096, N = 32768, KS = 512, NR = 8;
constexpr int NPA = 16, NPB = 128, NPAN = NPA + NPB;
constexpr unsigned BLK = 32768, PANEL = 64 * BLK, HTB = 16384;
constexpr int NGRP = 7, NSTR = 2 * NGRP, ROUNDS = 8, NSH = NSTR * ROUNDS;
constexpr int NCOMM = 16;
constexpr int LDS_BYTES = 8 * HTB;
constexpr unsigned OFF_CALL = 0;
constexpr unsigned OFF_TMO = 16384;
constexpr unsigned OFF_OWN = 20480;
constexpr unsigned OFF_DONE = 32768;
constexpr unsigned OFF_FLAG = 65536;
constexpr unsigned OFF_STAGE = 1u << 20;
constexpr unsigned STAGE_PAR = NPAN * PANEL;
constexpr size_t WS_NEED = (size_t)OFF_STAGE + 2ull * STAGE_PAR;
constexpr unsigned OUT_BYTES = (unsigned)((size_t)M * N * 4);
constexpr u64 DEADLINE = 100000000ull;
constexpr int ZSENT = 0x40000000;
constexpr int J0 = 8 * (NPA + NSTR);
#define OWN_AFTER 2
constexpr int J1 = J0 + OWN_AFTER * 8 * NSTR;
constexpr int J2 = J1 + 8 * 16;
constexpr int J3 = J2 + (ROUNDS - 1 - OWN_AFTER) * 8 * NSTR;
constexpr int NFWD = 32 * 8;
#define GLDS_AUX 0
#define C_AUX 2
#define ST_AUX 17

struct Params {
    const float* x; const float* w;
    char* ws[NR];
    char* out[NR];
    int rank;
};

#define RLX_SYS __ATOMIC_RELAXED, __HIP_MEMORY_SCOPE_SYSTEM
#define RLX_AGENT __ATOMIC_RELAXED, __HIP_MEMORY_SCOPE_AGENT
#define WAIT_V(n) asm volatile("s_waitcnt vmcnt(" #n ")" ::: "memory")
#define WAIT_L(n) asm volatile("s_waitcnt lgkmcnt(" #n ")" ::: "memory")
#define BAR __builtin_amdgcn_s_barrier()
#define SCHED __builtin_amdgcn_sched_barrier(0)
#define FENCE do { asm volatile("" ::: "memory"); SCHED; } while (0)

__device__ __forceinline__ unsigned pk16(float a, float b, int s) {
    f16x2 v; v.x = (_Float16)__builtin_ldexpf(a, s); v.y = (_Float16)__builtin_ldexpf(b, s);
    return __builtin_bit_cast(unsigned, v);
}
__device__ __forceinline__ bool give_up(gu32* tmo, unsigned call, u64 t0) {
    const bool bad = (unsigned)__builtin_amdgcn_readfirstlane(__hip_atomic_load(tmo, RLX_AGENT)) == call ||
                     __builtin_amdgcn_s_memrealtime() - t0 > DEADLINE;
    if (bad) __hip_atomic_store(tmo, call, RLX_AGENT);
    return bad;
}

struct Job { int panel, u, dq; };
__device__ __forceinline__ Job decode_job(int n, int rank) {
    Job j; j.dq = -1;
    if (n < J0) { j.u = n / (NPA + NSTR); j.panel = n % (NPA + NSTR); }
    else if (n < J1) { const int m = n - J0, c = m / (8 * NSTR), r = m % (8 * NSTR); j.u = r / NSTR; j.panel = NPA + (c + 1) * NSTR + r % NSTR; }
    else if (n < J2) { const int m = n - J1; j.u = m >> 4; j.dq = (m + 1) & 7; j.panel = NPA + NSH + 2 * ((rank + j.dq) & 7) + ((m >> 3) & 1); }
    else { const int m = n - J2, c = m / (8 * NSTR), r = m % (8 * NSTR); j.u = r / NSTR; j.panel = NPA + (c + 1 + OWN_AFTER) * NSTR + r % NSTR; }
    return j;
}

__device__ __forceinline__ void comm_role(const Params& p, LAS unsigned char* lds, int cw, unsigned call) {
    const int tid = threadIdx.x, wid = __builtin_amdgcn_readfirstlane(tid >> 6), lane = tid & 63, fr = lane & 15, fq = lane >> 4;
    const unsigned par = call & 1u;
    gu32* tmo = (gu32*)(p.ws[0] + OFF_TMO);
    LAS unsigned* red = (LAS unsigned*)lds;
    char* wsl = p.ws[0];
#pragma unroll
    for (int q = 1; q < NR; ++q) wsl = (lane & 7) == q ? p.ws[q] : wsl;
    gu64* flagl = (gu64*)(wsl + OFF_FLAG) + par * (NPAN * 64) + ((8 - (lane & 7)) & 7);
    const unsigned srcA = (unsigned)(((wid >> 2) * 128 + (wid & 3) * 32 + fr) * KS + fq * 8);
    const unsigned srcB = (unsigned)(((wid >> 2) * 32 + fq * 8) * N + (wid & 3) * 64 + fr * 4);
    const unsigned dstA = (unsigned)(wid * 4096 + lane * 16), dstB = (unsigned)((wid & 3) * 4096 + (wid >> 2) * 1024 + lane * 16);
    f32x4 v[8];
    auto issue_loads = [&](const Job& j) {
        if (j.panel < NPA) {
            const float* s = p.x + (size_t)j.panel * 256 * KS + j.u * 64 + srcA;
#pragma unroll
            for (int i = 0; i < 4; ++i) { const float* t = s + (i >> 1) * 16 * KS + (i & 1) * 32; v[2 * i] = *(const f32x4*)t; v[2 * i + 1] = *(const f32x4*)(t + 4); }
        } else {
            const float* s = p.w + (size_t)j.u * 64 * N + (j.panel - NPA) * 256 + srcB;
#pragma unroll
            for (int r = 0; r < 8; ++r) v[r] = __builtin_nontemporal_load((const f32x4*)(s + (size_t)r * N));
        }
    };
    int n = cw, it = 0;
    Job cur = decode_job(n, p.rank), prev = cur;
    int sprev = 0; bool have_prev = false;
    issue_loads(cur);
    for (;;) {
        const bool active = n < J3;
        WAIT_V(0);
        if (active) {
            unsigned mx = 0;
#pragma unroll
            for (int r = 0; r < 8; ++r)
#pragma unroll
                for (int c = 0; c < 4; ++c) { const unsigned b = __builtin_bit_cast(unsigned, v[r][c]) & 0x7fffffffu; mx = max(mx, b < 0x7f800000u ? b : 0u); }
#pragma unroll
            for (int o = 32; o >= 1; o >>= 1) mx = max(mx, (unsigned)__shfl_xor((int)mx, o));
            if (lane == 0) red[(it & 1) * 8 + wid] = mx;
        }
        WAIT_L(0); BAR; FENCE;
        if (have_prev && wid == 0 && lane < 8 && (prev.dq < 0 || prev.dq == lane))
            __hip_atomic_store(flagl + prev.panel * 64 + prev.u * 8, ((u64)call << 32) | (unsigned)sprev, RLX_SYS);
        if (!active) break;
        unsigned mx = red[(it & 1) * 8 + (lane & 7)];
#pragma unroll
        for (int o = 4; o >= 1; o >>= 1) mx = max(mx, (unsigned)__shfl_xor((int)mx, o));
        mx = __builtin_amdgcn_readfirstlane(mx);
        const int s = ((14 - ((int)(mx >> 23) - 127)) >> 4) * 16;
        u32x4 o[4];
        const bool isA = cur.panel < NPA;
        if (isA) {
#pragma unroll
            for (int i = 0; i < 4; ++i) { const f32x4 a = v[2 * i], c = v[2 * i + 1]; o[i].x = pk16(a[0], a[1], s); o[i].y = pk16(a[2], a[3], s); o[i].z = pk16(c[0], c[1], s); o[i].w = pk16(c[2], c[3], s); }
        } else {
#pragma unroll
            for (int i = 0; i < 4; ++i) { o[i].x = pk16(v[0][i], v[1][i], s); o[i].y = pk16(v[2][i], v[3][i], s); o[i].z = pk16(v[4][i], v[5][i], s); o[i].w = pk16(v[6][i], v[7][i], s); }
        }
        prev = cur; sprev = mx ? s : ZSENT; have_prev = true;
        n += NCOMM; ++it;
        if (n < J3) { cur = decode_job(n, p.rank); issue_loads(cur); }
        FENCE;
        const unsigned sbase = OFF_STAGE + par * STAGE_PAR + (unsigned)prev.panel * PANEL + (unsigned)prev.u * 8u * BLK;
        unsigned d[4];
#pragma unroll
        for (int i = 0; i < 4; ++i) d[i] = isA ? dstA + i * 1024 : dstB + (i >> 1) * HTB + (i & 1) * 2048;
#pragma unroll
        for (int qq = 1; qq <= NR; ++qq) {
            const int q = qq & 7;
            if (prev.dq >= 0 && prev.dq != q) continue;
            const __amdgpu_buffer_rsrc_t rs = __builtin_amdgcn_make_buffer_rsrc(p.ws[q], 0, (int)WS_NEED, 0x00020000);
#pragma unroll
            for (int i = 0; i < 4; ++i) __builtin_amdgcn_raw_buffer_store_b128(o[i], rs, (int)d[i], (int)(sbase + ((8 - q) & 7) * BLK), ST_AUX);
        }
    }
    bool bad = false;
    for (int f = cw; f < NFWD; f += NCOMM) {
        const int tile = f >> 3, slab = f & 7;
        const int ow = tile & 15, x = ow & 7;
        const int pm = (x & 3) * 4 + (tile >> 4) * 2 + (ow >> 3), pn = NSH + 2 * p.rank + (x >> 2);
        gu32* cnt = (gu32*)(p.ws[0] + OFF_OWN + tile * 128);
        if (!bad) {
            const u64 t0 = __builtin_amdgcn_s_memrealtime();
            while ((unsigned)__builtin_amdgcn_readfirstlane(__hip_atomic_load(cnt, RLX_AGENT)) < 8u * call) {
                if (give_up(tmo, call, t0)) { bad = true; break; }
                __builtin_amdgcn_s_sleep(8);
            }
        }
        const unsigned off = (unsigned)((pm * 256 + slab * 32 + wid * 4) * (N * 4) + pn * 1024);
        const __amdgpu_buffer_rsrc_t rl = __builtin_amdgcn_make_buffer_rsrc(p.out[0], 0, (int)OUT_BYTES, 0x00020000);
        u32x4 o[4];
#pragma unroll
        for (int i = 0; i < 4; ++i) o[i] = __builtin_amdgcn_raw_buffer_load_b128(rl, lane * 16 + i * (N * 4), (int)off, 16);
        if (bad) {
#pragma unroll
            for (int i = 0; i < 4; ++i) o[i] = (u32x4){0x7fc00000u, 0x7fc00000u, 0x7fc00000u, 0x7fc00000u};
        }
#pragma unroll
        for (int q = 1; q < NR; ++q) {
            const __amdgpu_buffer_rsrc_t rs = __builtin_amdgcn_make_buffer_rsrc(p.out[q], 0, (int)OUT_BYTES, 0x00020000);
#pragma unroll
            for (int i = 0; i < 4; ++i) __builtin_amdgcn_raw_buffer_store_b128(o[i], rs, lane * 16 + i * (N * 4), (int)off, ST_AUX);
        }
    }
    WAIT_V(0); BAR; FENCE;
    if (wid == 0) {
        if (lane >= 1 && lane < 8) __hip_atomic_store((gu32*)(wsl + OFF_DONE + cw * 128) + ((8 - lane) & 7), call, RLX_SYS);
        const u64 t0 = __builtin_amdgcn_s_memrealtime();
        gu32* dn = (gu32*)(p.ws[0] + OFF_DONE + cw * 128) + (lane & 7);
        for (;;) {
            const unsigned val = __hip_atomic_load(dn, RLX_SYS);
            if (__all((lane & 7) == 0 || val >= call)) break;
            if (give_up(tmo, call, t0)) { bad = true; break; }
            __builtin_amdgcn_s_sleep(4);
        }
        __builtin_amdgcn_fence(__ATOMIC_ACQUIRE, "");
        if (bad && lane == 0) ((float*)p.out[0])[(size_t)(NSH + 2 * p.rank) * 256 + cw] = __builtin_nanf("");
    }
}

__device__ __forceinline__ void gemm_role(const Params& p, LAS unsigned char* lds, int pm, int pn, int pmstep, int pnstep, int ntiles, int own_id, unsigned call) {
    const int tid = threadIdx.x, wid = __builtin_amdgcn_readfirstlane(tid >> 6), lane = tid & 63, wr = wid >> 2, wc = wid & 3;
    const unsigned par = call & 1u;
    const unsigned voff = (unsigned)wid * 2048u + (unsigned)lane * 16u;
    const unsigned ldsw = (unsigned)wid * 2048u;
    const int aoff = wr * 8192 + lane * 16, boff = wc * 4096 + lane * 16;
    gu32* tmo = (gu32*)(p.ws[0] + OFF_TMO);
    const __amdgpu_buffer_rsrc_t rsS = __builtin_amdgcn_make_buffer_rsrc(p.ws[0] + OFF_STAGE + (size_t)par * STAGE_PAR, 0, (int)STAGE_PAR, 0x00020000);
    const __amdgpu_buffer_rsrc_t rsC = __builtin_amdgcn_make_buffer_rsrc(p.out[0], 0, (int)OUT_BYTES, 0x00020000);
    gu64* flags = (gu64*)(p.ws[0] + OFF_FLAG) + par * (NPAN * 64);
    bool bad = false;
#define SA_(b, h) (((b) * 2 + (h)) * HTB)
#define SB_(b, h) ((4 + (b) * 2 + (h)) * HTB)
#define STAGE(bufoff, soff) do { \
    __builtin_amdgcn_raw_ptr_buffer_load_lds(rsS, (LAS void*)(lds + (bufoff) + ldsw), 16, (int)voff, (int)(soff), 0, GLDS_AUX); \
    __builtin_amdgcn_raw_ptr_buffer_load_lds(rsS, (LAS void*)(lds + (bufoff) + ldsw), 16, (int)voff, (int)(soff), 1024, GLDS_AUX); } while (0)
#define LDA(dst, b, h) do { _Pragma("unroll") for (int m = 0; m < 4; ++m) _Pragma("unroll") for (int k = 0; k < 2; ++k) dst[m][k] = *(const LAS f16x8*)(lds + SA_(b, h) + aoff + m * 2048 + k * 1024); } while (0)
#define LDB(dst, b, h) do { _Pragma("unroll") for (int n = 0; n < 2; ++n) _Pragma("unroll") for (int k = 0; k < 2; ++k) dst[n][k] = *(const LAS f16x8*)(lds + SB_(b, h) + boff + n * 2048 + k * 1024); } while (0)
#define ZERO4 ((f32x4){0.f, 0.f, 0.f, 0.f})
#define MF(Z, ai, bj, m, n, k, Bt) acc[ai][bj][m][n] = __builtin_amdgcn_mfma_f32_16x16x32_f16(At[m][k], Bt[n][k], ((Z) && (k) == 0) ? ZERO4 : acc[ai][bj][m][n], 0, 0, 0)
#define MMA2(Z, ai) do { __builtin_amdgcn_s_setprio(1); \
    _Pragma("unroll") for (int m = 0; m < 4; ++m) _Pragma("unroll") for (int n = 0; n < 2; ++n) _Pragma("unroll") for (int k = 0; k < 2; ++k) MF(Z, ai, 0, m, n, k, B0); \
    _Pragma("unroll") for (int m = 0; m < 4; ++m) _Pragma("unroll") for (int n = 0; n < 2; ++n) _Pragma("unroll") for (int k = 0; k < 2; ++k) MF(Z, ai, 1, m, n, k, B1); \
    __builtin_amdgcn_s_setprio(0); } while (0)
#define STORE_ROWS(cbase, ex, ai, m) do { _Pragma("unroll") for (int j = 0; j < 4; ++j) { \
      f32x4 o = {__builtin_ldexpf(acc[ai][0][m][0][j], ex), __builtin_ldexpf(acc[ai][0][m][1][j], ex), __builtin_ldexpf(acc[ai][1][m][0][j], ex), __builtin_ldexpf(acc[ai][1][m][1][j], ex)}; \
      __builtin_amdgcn_raw_buffer_store_b128(__builtin_bit_cast(u32x4, o), rsC, (int)coff, (int)((cbase) + (unsigned)((ai) * 128 + (m) * 16 + j) * (N * 4)), C_AUX); } } while (0)
#define POLL(a, b, pos2, maskv, expv) do { const u64 _t0 = __builtin_amdgcn_s_memrealtime(); \
      for (;;) { const u64 _fa = __hip_atomic_load(flags + (a) * 64 + lane, RLX_SYS), _fb = __hip_atomic_load(flags + (NPA + (b)) * 64 + lane, RLX_SYS); \
        maskv = __ballot((unsigned)(_fa >> 32) == call && (unsigned)(_fb >> 32) == call); \
        expv = ((int)_fa == ZSENT || (int)_fb == ZSENT) ? ZSENT : (int)_fa + (int)_fb; \
        if (((maskv >> (2 * (pos2))) & 3ull) == 3ull) break; \
        if (bad || give_up(tmo, call, _t0)) { bad = true; maskv = ~0ull; expv = ZSENT; break; } \
        __builtin_amdgcn_s_sleep(4); } } while (0)
#define RESCALE(pos) do { const int _e = __builtin_amdgcn_readlane(ecur, (pos)); \
      if (__builtin_expect(_e != esc && _e != ZSENT, 0)) { const int _d = _e - esc; esc = _e; asm volatile("s_nop 15"); \
        _Pragma("unroll") for (int a = 0; a < 2; ++a) _Pragma("unroll") for (int b = 0; b < 2; ++b) _Pragma("unroll") for (int m = 0; m < 4; ++m) _Pragma("unroll") for (int n = 0; n < 2; ++n) \
          _Pragma("unroll") for (int j = 0; j < 4; ++j) { float _f = acc[a][b][m][n][j]; asm volatile("v_ldexp_f32 %0, %0, %1" : "+v"(_f) : "s"(_d)); acc[a][b][m][n][j] = _f; } \
        asm volatile("s_nop 3"); } } while (0)
#define BODY(FIRST, LAST) do { \
      const unsigned a1 = cA + (unsigned)(it * 2 + 1) * BLK; \
      const unsigned a2 = (LAST) ? nA : cA + (unsigned)(it * 2 + 2) * BLK, b2 = (LAST) ? nB : cB + (unsigned)(it * 2 + 2) * BLK; \
      const unsigned a3 = a2 + BLK, b3 = b2 + BLK; \
      if (LAST) { if (has_next) POLL(pm + pmstep, pn + pnstep, 0, rmask, enext); } \
      else if (((rmask >> (2 * it + 2)) & 3ull) != 3ull) POLL(pm, pn, it + 1, rmask, ecur); \
      if (FIRST) { const int e = __builtin_amdgcn_readlane(ecur, 0); esc = e == ZSENT ? 0 : e; } else RESCALE(2 * it); \
      LDB(B0, 0, 0); LDB(B1, 0, 1); SCHED; LDA(At, 0, 0); STAGE(SA_(1, 1), a1 + HTB); \
      if ((FIRST) && ui > 0) { FENCE; STORE_ROWS(cPrev, eprev, 0, 3); STORE_ROWS(cPrev, eprev, 1, 0); STORE_ROWS(cPrev, eprev, 1, 1); WAIT_V(32); } else WAIT_V(8); \
      WAIT_L(0); BAR; MMA2(FIRST, 0); BAR; SCHED; \
      LDA(At, 0, 1); STAGE(SB_(0, 0), b2); STAGE(SB_(0, 1), b2 + HTB); STAGE(SA_(0, 0), a2); \
      if ((FIRST) && ui > 0) { FENCE; STORE_ROWS(cPrev, eprev, 1, 2); STORE_ROWS(cPrev, eprev, 1, 3); WAIT_V(40); } else WAIT_V(8); \
      WAIT_L(0); BAR; MMA2(FIRST, 1); BAR; SCHED; \
      RESCALE(2 * it + 1); \
      LDB(B0, 1, 0); LDB(B1, 1, 1); SCHED; LDA(At, 1, 0); STAGE(SA_(0, 1), a2 + HTB); \
      if ((FIRST) && ui > 0) WAIT_V(28); else WAIT_V(8); \
      WAIT_L(0); BAR; MMA2(0, 0); BAR; SCHED; \
      LDA(At, 1, 1); STAGE(SB_(1, 0), b3); STAGE(SB_(1, 1), b3 + HTB); STAGE(SA_(1, 0), a3); \
      if (LAST) { FENCE; STORE_ROWS(cC, -esc, 0, 0); STORE_ROWS(cC, -esc, 0, 1); STORE_ROWS(cC, -esc, 0, 2); WAIT_V(20); } else if ((FIRST) && ui > 0) WAIT_V(16); else WAIT_V(8); \
      WAIT_L(0); BAR; MMA2(0, 1); BAR; SCHED; } while (0)
    f32x4 acc[2][2][4][2];
#pragma unroll
    for (int a = 0; a < 2; ++a)
#pragma unroll
        for (int b = 0; b < 2; ++b)
#pragma unroll
            for (int m = 0; m < 4; ++m)
#pragma unroll
                for (int n = 0; n < 2; ++n) acc[a][b][m][n] = ZERO4;
    f16x8 At[4][2], B0[2][2], B1[2][2];
    unsigned cA = (unsigned)pm * PANEL, cB = (unsigned)(NPA + pn) * PANEL;
    const unsigned coff = (unsigned)((wr * 64 + (lane >> 4) * 4) * (N * 4) + (wc * 64 + (lane & 15) * 4) * 4);
    unsigned cC = (unsigned)pm * 256u * (N * 4) + (unsigned)pn * 1024u, cPrev = cC;
    u64 rmask; int ecur, enext = 0, esc = 0, eprev = 0;
    POLL(pm, pn, 0, rmask, ecur);
    STAGE(SB_(0, 0), cB); STAGE(SB_(0, 1), cB + HTB); STAGE(SA_(0, 0), cA); STAGE(SA_(0, 1), cA + HTB);
    if (wr == 1) BAR;
    WAIT_V(2); BAR;
    STAGE(SB_(1, 0), cB + BLK); STAGE(SA_(1, 0), cA + BLK); STAGE(SB_(1, 1), cB + BLK + HTB);
    WAIT_V(6); BAR;
#pragma unroll 1
    for (int ui = 0; ui < ntiles; ++ui) {
        const bool has_next = ui + 1 < ntiles;
        const unsigned nA = has_next ? cA + (unsigned)pmstep * PANEL : cA, nB = has_next ? cB + (unsigned)pnstep * PANEL : cB;
        int it = 0;
        BODY(1, 0);
#pragma unroll 1
        for (it = 1; it < 31; ++it) {
            if (own_id >= 0 && it == 2 && ui > 0) {
                __builtin_amdgcn_fence(__ATOMIC_RELEASE, "agent"); WAIT_V(0);
                if (lane == 0) __hip_atomic_fetch_add((gu32*)(p.ws[0] + OFF_OWN + ((ui - 1) * 16 + own_id) * 128), 1u, RLX_AGENT);
            }
            BODY(0, 0);
        }
        BODY(0, 1);
        cPrev = cC; eprev = -esc; ecur = enext;
        cC += (unsigned)pmstep * 256u * (N * 4) + (unsigned)pnstep * 1024u; cA = nA; cB = nB; pm += pmstep; pn += pnstep;
    }
    STORE_ROWS(cPrev, eprev, 0, 3); STORE_ROWS(cPrev, eprev, 1, 0); STORE_ROWS(cPrev, eprev, 1, 1); STORE_ROWS(cPrev, eprev, 1, 2); STORE_ROWS(cPrev, eprev, 1, 3);
    WAIT_V(0);
    if (bad) { *(float*)(p.out[0] + cPrev + coff) = __builtin_nanf(""); WAIT_V(0); }
    if (own_id >= 0) {
        __builtin_amdgcn_fence(__ATOMIC_RELEASE, "agent"); WAIT_V(0);
        if (lane == 0) __hip_atomic_fetch_add((gu32*)(p.ws[0] + OFF_OWN + ((ntiles - 1) * 16 + own_id) * 128), 1u, RLX_AGENT);
    }
    if (wr == 0) BAR;
}

__global__ __launch_bounds__(512, 2) void fused_kernel(Params p) {
    extern __shared__ __attribute__((aligned(16))) unsigned char shm[];
    LAS unsigned char* lds = (LAS unsigned char*)shm;
    gu32* cc = (gu32*)(p.ws[0] + OFF_CALL + blockIdx.x * 64);
    const unsigned call = (unsigned)__builtin_amdgcn_readfirstlane(__hip_atomic_load(cc, RLX_AGENT)) + 1u;
    BAR;
    if (threadIdx.x == 0) __hip_atomic_store(cc, call, RLX_AGENT);
    const int x = blockIdx.x & 7, o = blockIdx.x >> 3;
    if (o < 2) { comm_role(p, lds, blockIdx.x, call); return; }
    int pm, pn, pmstep, pnstep, nt, own;
    if (o < 4) { pm = (x & 3) * 4 + (o - 2); pn = NSH + 2 * p.rank + (x >> 2); pmstep = 2; pnstep = 0; nt = 2; own = (o - 2) * 8 + x; }
    else { const int g = (o - 4) >> 2; pm = (x & 3) * 4 + (o & 3); pn = (x >> 2) * NGRP + g; pmstep = 0; pnstep = NSTR; nt = ROUNDS; own = -1; }
    gemm_role(p, lds, pm, pn, pmstep, pnstep, nt, own, call);
}

}

extern "C" void kernel_launch(int rank, int nranks, ncclComm_t comm, hipStream_t stream,
                              void* const* d_in, const int* in_sizes, int n_in,
                              void* d_out, int out_size, const int* in_dtypes, int out_dtype,
                              void* const* d_out_all, void* const* d_ws_all, size_t ws_bytes) {
    if (nranks != NR || (unsigned)rank >= (unsigned)NR || ws_bytes < WS_NEED || n_in < 2 ||
        in_sizes[0] != M * KS || in_sizes[1] != KS * N || out_size != M * N) {
        fprintf(stderr, "kernel_launch: unexpected problem shape\n");
        return;
    }
    Params p;
    p.x = (const float*)d_in[0]; p.w = (const float*)d_in[1]; p.rank = rank;
    for (int q = 0; q < NR; ++q) { p.ws[q] = (char*)d_ws_all[(rank + q) % NR]; p.out[q] = (char*)d_out_all[(rank + q) % NR]; }
    p.out[0] = (char*)d_out;
    HIP_CHECK(hipFuncSetAttribute((const void*)fused_kernel, hipFuncAttributeMaxDynamicSharedMemorySize, LDS_BYTES));
    hipLaunchKernelGGL(fused_kernel, dim3(256), dim3(512), LDS_BYTES, stream, p);
    HIP_CHECK(hipGetLastError());
}
```

```cpp
#include <hip/hip_runtime.h>
#include <rccl/rccl.h>
#include <cstdio>

#define HIP_CHECK(expr) do { hipError_t _e = (expr); if (_e != hipSuccess) { \
    fprintf(stderr, "HIP error %s at %s:%d\n", hipGetErrorString(_e), __FILE__, __LINE__); \
    return; } } while (0)

namespace {

#define LAS __attribute__((address_space(3)))
#define GAS __attribute__((address_space(1)))
typedef _Float16 f16x8 __attribute__((ext_vector_type(8)));
typedef _Float16 f16x2 __attribute__((ext_vector_type(2)));
typedef float f32x4 __attribute__((ext_vector_type(4)));
typedef unsigned u32x4 __attribute__((ext_vector_type(4)));
typedef unsigned long long u64;
typedef GAS unsigned gu32;
typedef GAS u64 gu64;

constexpr int M = 4096, N = 32768, KS = 512, NR = 8;
constexpr int NPA = 16, NPB = 128, NPAN = NPA + NPB;
#define PPAD 0
constexpr unsigned BLK = 32768, PANEL = 64 * BLK + PPAD, HTB = 16384;
constexpr int NGRP = 7, NSTR = 2 * NGRP, ROUNDS = 8, NSH = NSTR * ROUNDS;
#define NOWN 8
#define NCW (32 - NOWN)
#define WIN 2
#define STAG 0
constexpr int OWN_TILES = 32 / NOWN;
constexpr int LDS_BYTES = 8 * HTB + 256;
constexpr unsigned OFF_CALL = 0;
constexpr unsigned OFF_TMO = 16384;
constexpr unsigned OFF_OWN = 20480;
constexpr unsigned OFF_TICKET = 28672;
constexpr unsigned OFF_DONE = 32768;
constexpr unsigned OFF_FLAG = 65536;
constexpr unsigned OFF_STAGE = 1u << 20;
constexpr unsigned STAGE_PAR = NPAN * PANEL;
constexpr size_t WS_NEED = (size_t)OFF_STAGE + 2ull * STAGE_PAR;
constexpr unsigned OUT_BYTES = (unsigned)((size_t)M * N * 4);
constexpr u64 DEADLINE = 100000000ull;
constexpr int ZSENT = 0x40000000;
#define MIXR 2
constexpr int J0 = 8 * (NPA + NSTR);
constexpr int J1 = J0 + (MIXR - 1) * 8 * NSTR;
constexpr int J2 = J1 + 8 * (NSTR + 16);
constexpr int J3 = J2 + (ROUNDS - 1 - MIXR) * 8 * NSTR;
constexpr int JTOT = J3 + 32 * 8;
#define LAYOUT 0
#define GLDS_AUX 17
#define C_AUX 2
#define ST_AUX 17

struct Params {
    const float* x; const float* w;
    char* ws[NR];
    char* out[NR];
    int rank;
};

#define STAMP(idx) do {} while (0)
#define STAMPV(idx, v) do {} while (0)
#define RLX_SYS __ATOMIC_RELAXED, __HIP_MEMORY_SCOPE_SYSTEM
#define RLX_AGENT __ATOMIC_RELAXED, __HIP_MEMORY_SCOPE_AGENT
#define WAIT_V(n) asm volatile("s_waitcnt vmcnt(" #n ")" ::: "memory")
#define WAIT_L(n) asm volatile("s_waitcnt lgkmcnt(" #n ")" ::: "memory")
#define BAR __builtin_amdgcn_s_barrier()
#define SCHED __builtin_amdgcn_sched_barrier(0)
#define FENCE do { asm volatile("" ::: "memory"); SCHED; } while (0)

__device__ __forceinline__ unsigned pk16(float a, float b, int s) {
    f16x2 v; v.x = (_Float16)__builtin_ldexpf(a, s); v.y = (_Float16)__builtin_ldexpf(b, s);
    return __builtin_bit_cast(unsigned, v);
}
__device__ __forceinline__ __amdgpu_buffer_rsrc_t make_rsrc(const char* base, unsigned bytes) {
    const u64 a = (u64)base;
    const u64 u = ((u64)(unsigned)__builtin_amdgcn_readfirstlane((int)(a >> 32)) << 32) | (unsigned)__builtin_amdgcn_readfirstlane((int)a);
    return __builtin_amdgcn_make_buffer_rsrc((void*)u, 0, __builtin_amdgcn_readfirstlane((int)bytes), 0x00020000);
}
__device__ __forceinline__ bool give_up(gu32* tmo, unsigned call, u64 t0) {
    const bool bad = (unsigned)__builtin_amdgcn_readfirstlane(__hip_atomic_load(tmo, RLX_AGENT)) == call ||
                     __builtin_amdgcn_s_memrealtime() - t0 > DEADLINE;
    if (bad) __hip_atomic_store(tmo, call, RLX_AGENT);
    return bad;
}

__device__ __forceinline__ void draw_async(unsigned& t, gu32* ctr) {
    asm volatile("global_atomic_add %0, %1, %2, off sc0" : "=v"(t) : "v"(ctr), "v"(1u) : "memory");
}

struct Job { int panel, u, dq; };
__device__ __forceinline__ Job decode_job(int n, int rank) {
    Job j; j.dq = -1;
    if (n < J0) { j.u = n / (NPA + NSTR); j.panel = n % (NPA + NSTR); }
    else if (n < J1) { const int m = n - J0, c = m / (8 * NSTR), r = m % (8 * NSTR); j.u = r / NSTR; j.panel = NPA + (c + 1) * NSTR + r % NSTR; }
    else if (n < J2) { const int m = n - J1, r = m % (NSTR + 16); j.u = m / (NSTR + 16);
        if ((r & 1) && r < 2 * NSTR) j.panel = NPA + MIXR * NSTR + (r >> 1);
        else { const int k = r < 2 * NSTR ? r >> 1 : r - NSTR; j.dq = (k + 1) & 7; j.panel = NPA + NSH + 2 * ((rank + j.dq) & 7) + (k >> 3); } }
    else if (n < J3) { const int m = n - J2, c = m / (8 * NSTR), r = m % (8 * NSTR); j.u = r / NSTR; j.panel = NPA + (c + MIXR + 1) * NSTR + r % NSTR; }
    else { const int m = n - J3; j.u = m & 7; j.panel = NPAN + (m >> 3); }
    return j;
}
__device__ __forceinline__ void own_tile(int t, int rank, int& pm, int& pn) {
    const int ow = t % NOWN, r = t / NOWN, x = ow & 7, idx = r * (NOWN / 8) + (ow >> 3);
    pm = (x & 3) * 4 + idx; pn = NSH + 2 * rank + (x >> 2);
}

constexpr unsigned DRAWS = JTOT - 2 * NCW + NCW;
#define STR2(x) #x
#define STR(x) STR2(x)
#define CWAIT_V(n) do { __builtin_amdgcn_s_waitcnt(((n) & 15) | (7 << 4) | (15 << 8) | (((n) >> 4) << 14)); asm volatile("" ::: "memory"); } while (0)
#define WAIT_WIN asm volatile("s_waitcnt vmcnt(" STR(WIN) ")" ::: "memory")

__device__ __forceinline__ unsigned wave_max_u32(unsigned mx) {
#pragma unroll
    for (int k = 32; k >= 1; k >>= 1) mx = max(mx, (unsigned)__shfl_xor((int)mx, k));
    return __builtin_amdgcn_readfirstlane(mx);
}
__device__ __forceinline__ int scale_exp(unsigned mx) { return ((14 - ((int)(mx >> 23) - 127)) >> 4) * 16; }
__device__ __forceinline__ unsigned lds_load(LAS unsigned* w) { return __builtin_amdgcn_readfirstlane(__hip_atomic_load(w, __ATOMIC_RELAXED, __HIP_MEMORY_SCOPE_WORKGROUP)); }
__device__ __forceinline__ void lds_store(LAS unsigned* w, unsigned v) { __hip_atomic_store(w, v, __ATOMIC_RELAXED, __HIP_MEMORY_SCOPE_WORKGROUP); }

__device__ __forceinline__ void comm_role(const Params& p, LAS unsigned char* lds, int cw, unsigned call) {
    const int tid = threadIdx.x, wid = __builtin_amdgcn_readfirstlane(tid >> 6), lane = tid & 63, fr = lane & 15, fq = lane >> 4;
    const unsigned par = call & 1u;
    gu32* tmo = (gu32*)(p.ws[0] + OFF_TMO);
    LAS unsigned* ctl = (LAS unsigned*)(lds + 4 * BLK);
    if (tid < 8) ctl[tid] = 0;
    WAIT_L(0); BAR; FENCE;
    gu64* flagq = (gu64*)(p.ws[wid] + OFF_FLAG) + par * (NPAN * 64) + ((8 - wid) & 7);
    const __amdgpu_buffer_rsrc_t rsD = make_rsrc(p.ws[wid] + OFF_STAGE + (size_t)par * STAGE_PAR + ((8 - wid) & 7) * BLK, STAGE_PAR - 7 * BLK);
    const __amdgpu_buffer_rsrc_t rsO = make_rsrc(p.out[wid], OUT_BYTES);
    int pidx = -1, psc = 0;
#define PUBLISH_PENDING do { if (pidx >= 0 && lane == 0) __hip_atomic_store(flagq + pidx, ((u64)call << 32) | (unsigned)psc, RLX_SYS); pidx = -1; } while (0)
    bool bad = false;
    if (wid > 0) {
        for (unsigned n = 0;; ++n) {
            if (lds_load(ctl) <= n) {
                WAIT_V(0); PUBLISH_PENDING;
                while (lds_load(ctl) <= n) __builtin_amdgcn_s_sleep(2);
            }
            FENCE;
            LAS unsigned* jw = ctl + 8 + 4 * (n & 3);
            const unsigned soff = lds_load(jw), fidx = lds_load(jw + 1), sc = lds_load(jw + 2), dq = lds_load(jw + 3);
            if (dq == 10u) break;
            if (dq >= 8u || dq == (unsigned)wid) {
                const __amdgpu_buffer_rsrc_t rs = dq == 9u ? rsO : rsD;
                const unsigned kst = dq == 9u ? N * 4 : 1024;
                const LAS unsigned char* img = lds + (n & 3) * BLK + lane * 16;
#pragma unroll
                for (int g = 0; g < 4; ++g) {
                    u32x4 o[8];
#pragma unroll
                    for (int k = 0; k < 8; ++k) o[k] = *(const LAS u32x4*)(img + (g * 8 + k) * 1024);
                    if (g == 3) { WAIT_L(0); if (lane == 0) lds_store(ctl + wid, n + 1); }
#pragma unroll
                    for (int k = 0; k < 8; ++k) { __builtin_amdgcn_raw_buffer_store_b128(o[k], rs, lane * 16, (int)(soff + (unsigned)(g * 8 + k) * kst), ST_AUX); WAIT_WIN; }
                    if (g == 0) PUBLISH_PENDING;
                }
                if (dq != 9u) { pidx = (int)fidx; psc = (int)sc; }
            } else if (lane == 0) lds_store(ctl + wid, n + 1);
        }
        WAIT_V(0); PUBLISH_PENDING;
        if (lane == 0) __hip_atomic_store((gu32*)(p.ws[wid] + OFF_DONE + cw * 128) + ((8 - wid) & 7), call, RLX_SYS);
        return;
    }
    const unsigned tbase = (call - 1u) * DRAWS - 2u * NCW;
    gu32* tick = (gu32*)(p.ws[0] + OFF_TICKET);
    const __amdgpu_buffer_rsrc_t rsX = make_rsrc((const char*)p.x, M * KS * 4), rsW = make_rsrc((const char*)p.w, KS * N * 4);
    const unsigned lA = (unsigned)(fr * KS + fq * 8) * 4u, lB = (unsigned)(fq * 8 * N + fr * 4) * 4u;
    unsigned n = 0;
    auto slot_free = [&]() {
        while (n >= 4u) { const unsigned c = __hip_atomic_load(ctl + 1 + (lane % 7), __ATOMIC_RELAXED, __HIP_MEMORY_SCOPE_WORKGROUP);
            if (__all(c + 3u >= n)) break; __builtin_amdgcn_s_sleep(2); }
        FENCE;
    };
    auto post = [&](unsigned soff, unsigned fidx, unsigned sc, unsigned dq) {
        if (lane == 0) { LAS unsigned* jw = ctl + 8 + 4 * (n & 3); jw[0] = soff; jw[1] = fidx; jw[2] = sc; jw[3] = dq; }
        WAIT_L(0);
        ++n; if (lane == 0) lds_store(ctl, n);
    };
#define LOAD_PIECE(dst, j, t) do { const bool _a = (j).panel < NPA; \
        const unsigned _b = _a ? (unsigned)((j).panel * 256 * KS + (j).u * 64) * 4u + ((t) & 3) * (32 * KS * 4) + ((t) >> 2) * (128 * KS * 4) \
                               : (unsigned)((j).u * 64 * N + ((j).panel - NPA) * 256) * 4u + ((t) & 3) * 256 + ((t) >> 2) * (32 * N * 4); \
        const unsigned _s0 = _a ? 16 : N * 4, _s1 = _a ? 128 : 2 * N * 4, _s2 = _a ? 16 * KS * 4 : 4 * N * 4; \
        _Pragma("unroll") for (int r = 0; r < 8; ++r) dst[r] = __builtin_bit_cast(f32x4, __builtin_amdgcn_raw_buffer_load_b128(_a ? rsX : rsW, (int)(_a ? lA : lB), \
            (int)(_b + (r & 1) * _s0 + ((r >> 1) & 1) * _s1 + (r >> 2) * _s2), 0)); } while (0)
#define PIECE_MAX(mx, v) do { _Pragma("unroll") for (int r = 0; r < 8; ++r) { const u32x4 _u = __builtin_bit_cast(u32x4, v[r]) & 0x7fffffffu; \
        mx = max(max(mx, _u.x), _u.y); mx = max(max(mx, _u.z), _u.w); } asm volatile("" : "+v"(mx)); } while (0)
#define PUT_FRAG(o, d, loff) do { *(LAS u32x4*)(img + (d)) = o; __builtin_amdgcn_raw_buffer_store_b128(o, rsD, lane * 16, (int)((loff) + (d)), ST_AUX); } while (0)
#define CONV_PIECE(v, isA, t, s, loff) do { \
        if (isA) { _Pragma("unroll") for (int i = 0; i < 4; ++i) { const f32x4 a = v[2 * i], c = v[2 * i + 1]; u32x4 o; \
            o.x = pk16(a.x, a.y, s); o.y = pk16(a.z, a.w, s); o.z = pk16(c.x, c.y, s); o.w = pk16(c.z, c.w, s); PUT_FRAG(o, (t) * 4096 + i * 1024, loff); } } \
        else { _Pragma("unroll") for (int i = 0; i < 4; ++i) { u32x4 o; \
            o.x = pk16(v[0][i], v[1][i], s); o.y = pk16(v[2][i], v[3][i], s); o.z = pk16(v[4][i], v[5][i], s); o.w = pk16(v[6][i], v[7][i], s); \
            PUT_FRAG(o, ((t) & 3) * 4096 + ((t) >> 2) * 1024 + (i >> 1) * HTB + (i & 1) * 2048, loff); } } \
        FENCE; } while (0)
    unsigned t1 = cw, t2 = NCW + cw, drawn = 0;
    f32x4 v[4][8];
    Job cur = decode_job((int)t1, p.rank), nxt = cur;
    LOAD_PIECE(v[0], cur, 0); LOAD_PIECE(v[1], cur, 1); LOAD_PIECE(v[2], cur, 2); LOAD_PIECE(v[3], cur, 3);
#pragma unroll 1
    while (t1 < (unsigned)J3) {
        if (lane == 0) draw_async(drawn, tick);
        nxt = t2 < (unsigned)J3 ? decode_job((int)t2, p.rank) : cur;
        const bool isA = cur.panel < NPA;
        const unsigned soff = (unsigned)cur.panel * PANEL + (unsigned)cur.u * 8u * BLK;
        const unsigned loff = cur.dq <= 0 ? soff : 0xf0000000u;
        slot_free();
        LAS unsigned char* img = lds + (n & 3) * BLK + lane * 16;
        unsigned mx = 0; PIECE_MAX(mx, v[0]);
        int s = scale_exp(wave_max_u32(mx));
#pragma unroll
        for (int t = 0; t < 8; ++t) {
            if (t) PIECE_MAX(mx, v[t & 3]);
            CONV_PIECE(v[t & 3], isA, t, s, loff);
            if (t < 4) LOAD_PIECE(v[t & 3], cur, t + 4); else LOAD_PIECE(v[t & 3], nxt, t - 4);
            FENCE;
            if (t == 4) PUBLISH_PENDING;
        }
        mx = wave_max_u32(mx);
        if (__builtin_expect(mx >= 0x7f800000u || scale_exp(mx) != s, 0)) {
            f32x4 w[8]; mx = 0;
#pragma unroll 1
            for (int t = 0; t < 8; ++t) { LOAD_PIECE(w, cur, t);
#pragma unroll
                for (int r = 0; r < 8; ++r) { const u32x4 u = __builtin_bit_cast(u32x4, w[r]) & 0x7fffffffu;
#pragma unroll
                    for (int c = 0; c < 4; ++c) { const unsigned b = u[c]; mx = max(mx, b < 0x7f800000u ? b : 0u); } } }
            mx = wave_max_u32(mx); s = scale_exp(mx);
            WAIT_V(0);
#pragma unroll 1
            for (int t = 0; t < 8; ++t) { LOAD_PIECE(w, cur, t); CONV_PIECE(w, isA, t, s, loff); }
        }
        const int sc = mx ? s : ZSENT, fidx = cur.panel * 64 + cur.u * 8;
        post(soff, (unsigned)fidx, (unsigned)sc, cur.dq < 0 ? 8u : (unsigned)cur.dq);
        STAMP(n);
        if (cur.dq <= 0) { pidx = fidx; psc = sc; }
        asm volatile("" : "+v"(drawn));
        t1 = t2; t2 = __builtin_amdgcn_readfirstlane(drawn) - tbase; cur = nxt;
    }
    WAIT_V(0); PUBLISH_PENDING;
    STAMP(99);
    unsigned ready = 0;
    while (t1 < (unsigned)JTOT) {
        const bool pending = t2 < (unsigned)JTOT;
        if (pending && lane == 0) draw_async(drawn, tick);
        const int tile = (int)(t1 - J3) >> 3;
        if (!((ready >> tile) & 1u)) {
            gu32* cnt = (gu32*)(p.ws[0] + OFF_OWN + (lane & 31) * 128);
            const u64 t0 = __builtin_amdgcn_s_memrealtime();
            for (;;) {
                ready = (unsigned)__ballot(__hip_atomic_load(cnt, RLX_AGENT) >= 8u * call);
                if ((ready >> tile) & 1u) break;
                if (give_up(tmo, call, t0)) { bad = true; ready = ~0u; break; }
                __builtin_amdgcn_s_sleep(8);
            }
        }
        int pm, pn; own_tile(tile, p.rank, pm, pn);
        const unsigned off = (unsigned)((pm * 256 + (int)((t1 - J3) & 7) * 32) * (N * 4) + pn * 1024);
        slot_free();
        if (!bad) {
#pragma unroll
            for (int k = 0; k < 32; ++k) __builtin_amdgcn_raw_ptr_buffer_load_lds(rsO, (LAS void*)(lds + (n & 3) * BLK + k * 1024), 16, lane * 16, (int)(off + (unsigned)k * (N * 4)), 0, 16);
        } else {
            for (int k = 0; k < 32; ++k) *(LAS u32x4*)(lds + (n & 3) * BLK + k * 1024 + lane * 16) = (u32x4){0x7fc00000u, 0x7fc00000u, 0x7fc00000u, 0x7fc00000u};
        }
        WAIT_V(0); asm volatile("" : "+v"(drawn));
        post(off, 0u, 0u, 9u);
        STAMP(n);
        t1 = t2; t2 = pending ? __builtin_amdgcn_readfirstlane(drawn) - tbase : ~0u;
    }
    slot_free(); post(0u, 0u, 0u, 10u);
    STAMP(100);
    {
        const u64 t0 = __builtin_amdgcn_s_memrealtime();
        gu32* dn = (gu32*)(p.ws[0] + OFF_DONE + cw * 128) + (lane & 7);
        for (;;) {
            const unsigned val = __hip_atomic_load(dn, RLX_SYS);
            if (__all((lane & 7) == 0 || val >= call)) break;
            if (give_up(tmo, call, t0)) { bad = true; break; }
            __builtin_amdgcn_s_sleep(4);
        }
        __builtin_amdgcn_fence(__ATOMIC_ACQUIRE, "");
        STAMP(101);
        if (bad && lane == 0) ((float*)p.out[0])[(size_t)(NSH + 2 * p.rank) * 256 + cw] = __builtin_nanf("");
    }
}

__device__ __forceinline__ void gemm_role(const Params& p, LAS unsigned char* lds, int pm, int pn, int pmstep, int pnstep, int ntiles, int own_id, int pp0, unsigned call) {
    const int tid = threadIdx.x, wid = __builtin_amdgcn_readfirstlane(tid >> 6), lane = tid & 63, wr = wid >> 2, wc = wid & 3;
    const unsigned par = call & 1u;
    const unsigned voff = (unsigned)wid * 2048u + (unsigned)lane * 16u;
    const unsigned ldsw = (unsigned)wid * 2048u;
    const int aoff = wr * 8192 + lane * 16, boff = wc * 4096 + lane * 16;
    gu32* tmo = (gu32*)(p.ws[0] + OFF_TMO);
    const __amdgpu_buffer_rsrc_t rsS = make_rsrc(p.ws[0] + OFF_STAGE + (size_t)par * STAGE_PAR, STAGE_PAR);
    const __amdgpu_buffer_rsrc_t rsC = make_rsrc(p.out[0], OUT_BYTES);
    gu64* flags = (gu64*)(p.ws[0] + OFF_FLAG) + par * (NPAN * 64);
    bool bad = false;
#define SA_(b, h) (((b) * 2 + (h)) * HTB)
#define SB_(b, h) ((4 + (b) * 2 + (h)) * HTB)
#define STAGE(bufoff, soff) do { \
    __builtin_amdgcn_raw_ptr_buffer_load_lds(rsS, (LAS void*)(lds + (bufoff) + ldsw), 16, (int)voff, (int)(soff), 0, GLDS_AUX); \
    __builtin_amdgcn_raw_ptr_buffer_load_lds(rsS, (LAS void*)(lds + (bufoff) + ldsw), 16, (int)voff, (int)(soff), 1024, GLDS_AUX); } while (0)
#define LDA(dst, b, h) do { _Pragma("unroll") for (int m = 0; m < 4; ++m) _Pragma("unroll") for (int k = 0; k < 2; ++k) dst[m][k] = *(const LAS f16x8*)(lds + SA_(b, h) + aoff + m * 2048 + k * 1024); } while (0)
#define LDB(dst, b, h) do { _Pragma("unroll") for (int n = 0; n < 2; ++n) _Pragma("unroll") for (int k = 0; k < 2; ++k) dst[n][k] = *(const LAS f16x8*)(lds + SB_(b, h) + boff + n * 2048 + k * 1024); } while (0)
#define POLL_ACC
#define ZERO4 ((f32x4){0.f, 0.f, 0.f, 0.f})
#define MF(Z, ai, bj, m, n, k, Bt) do { if ((Z) && (k) == 0) asm volatile("v_mfma_f32_16x16x32_f16 %0, %1, %2, 0" : "+v"(acc[ai][bj][m][n]) : "v"(At[m][k]), "v"(Bt[n][k])); \
      else asm volatile("v_mfma_f32_16x16x32_f16 %0, %1, %2, %0" : "+v"(acc[ai][bj][m][n]) : "v"(At[m][k]), "v"(Bt[n][k])); } while (0)
#define MORDER 1
#define MMA_SEQ(Z, ai) _Pragma("unroll") for (int m = 0; m < 4; ++m) { MF(Z, ai, 0, m, 0, 0, B0); MF(Z, ai, 0, m, 0, 1, B0); MF(Z, ai, 0, m, 1, 0, B0); MF(Z, ai, 0, m, 1, 1, B0); \
                                                                    MF(Z, ai, 1, m, 0, 0, B1); MF(Z, ai, 1, m, 0, 1, B1); MF(Z, ai, 1, m, 1, 0, B1); MF(Z, ai, 1, m, 1, 1, B1); }
#define MMA2(Z, ai) do { __builtin_amdgcn_s_setprio(1); MMA_SEQ(Z, ai) __builtin_amdgcn_s_setprio(0); } while (0)
#define STORE_ROWS(cbase, ex, ai, m) do { _Pragma("unroll") for (int j = 0; j < 4; ++j) { \
      f32x4 o = {__builtin_ldexpf(acc[ai][0][m][0][j], ex), __builtin_ldexpf(acc[ai][0][m][1][j], ex), __builtin_ldexpf(acc[ai][1][m][0][j], ex), __builtin_ldexpf(acc[ai][1][m][1][j], ex)}; \
      __builtin_amdgcn_raw_buffer_store_b128(__builtin_bit_cast(u32x4, o), rsC, (int)coff, (int)((cbase) + (unsigned)((ai) * 128 + (m) * 16 + j) * (N * 4)), C_AUX); } } while (0)
#define POLL_SLEEP 16
#define POLL(a, b, pos2, maskv, expv) do { const u64 _t0 = __builtin_amdgcn_s_memrealtime(); bool _ok = false; \
      for (int _n = 0;; ++_n) { \
        if (_n == 0 || (lane >> 4) == ((pos2) >> 3)) {     \
          const u64 _fa = __hip_atomic_load(flags + (a) * 64 + lane, RLX_SYS), _fb = __hip_atomic_load(flags + (NPA + (b)) * 64 + lane, RLX_SYS); \
          _ok = (unsigned)(_fa >> 32) == call && (unsigned)(_fb >> 32) == call; \
          expv = ((int)_fa == ZSENT || (int)_fb == ZSENT) ? ZSENT : (int)_fa + (int)_fb; } \
        maskv = __ballot(_ok); \
        if (((maskv >> (2 * (pos2))) & 3ull) == 3ull) break; \
        if (bad || ((_n & 63) == 63 && give_up(tmo, call, _t0))) { bad = true; maskv = ~0ull; expv = ZSENT; break; } \
        __builtin_amdgcn_s_sleep(POLL_SLEEP); } POLL_ACC; } while (0)
#define RESCALE(pos) do { const int _e = __builtin_amdgcn_readlane(ecur, (pos)); \
      if (__builtin_expect(_e != esc && _e != ZSENT, 0)) { const int _d = _e - esc; esc = _e; asm volatile("s_nop 15"); \
        _Pragma("unroll") for (int a = 0; a < 2; ++a) _Pragma("unroll") for (int b = 0; b < 2; ++b) _Pragma("unroll") for (int m = 0; m < 4; ++m) _Pragma("unroll") for (int n = 0; n < 2; ++n) \
          _Pragma("unroll") for (int j = 0; j < 4; ++j) { float _f = acc[a][b][m][n][j]; asm volatile("v_ldexp_f32 %0, %0, %1" : "+v"(_f) : "s"(_d)); acc[a][b][m][n][j] = _f; } \
        asm volatile("s_nop 3"); } } while (0)
#define BODY(FIRST, LAST) do { \
      const int pp = (it + pp0) & 31, ppn = (pp + 1) & 31; \
      const unsigned a1 = cA + (unsigned)(pp * 2 + 1) * BLK; \
      const unsigned a2 = ((LAST) ? nA : cA) + (unsigned)(ppn * 2) * BLK, b2 = ((LAST) ? nB : cB) + (unsigned)(ppn * 2) * BLK; \
      const unsigned a3 = a2 + BLK, b3 = b2 + BLK; \
      if (LAST) { if (has_next) POLL(pm + pmstep, pn + pnstep, ppn, rmask, enext); } \
      else if (((rmask >> (2 * ppn)) & 3ull) != 3ull) POLL(pm, pn, ppn, rmask, ecur); \
      if (FIRST) { const int e = __builtin_amdgcn_readlane(ecur, 2 * pp); esc = e == ZSENT ? 0 : e; } else RESCALE(2 * pp); \
      LDB(B0, 0, 0); LDB(B1, 0, 1); SCHED; LDA(At, 0, 0); STAGE(SA_(1, 1), a1 + HTB); \
      if ((FIRST) && ui > 0) { FENCE; STORE_ROWS(cPrev, eprev, 0, 3); STORE_ROWS(cPrev, eprev, 1, 0); STORE_ROWS(cPrev, eprev, 1, 1); WAIT_V(32); } else WAIT_V(8); \
      WAIT_L(0); BAR; MMA2(FIRST, 0); BAR; SCHED; \
      LDA(At, 0, 1); STAGE(SB_(0, 0), b2); STAGE(SB_(0, 1), b2 + HTB); STAGE(SA_(0, 0), a2); \
      if ((FIRST) && ui > 0) { FENCE; STORE_ROWS(cPrev, eprev, 1, 2); STORE_ROWS(cPrev, eprev, 1, 3); WAIT_V(40); } else WAIT_V(8); \
      WAIT_L(0); BAR; MMA2(FIRST, 1); BAR; SCHED; \
      RESCALE(2 * pp + 1); \
      LDB(B0, 1, 0); LDB(B1, 1, 1); SCHED; LDA(At, 1, 0); STAGE(SA_(0, 1), a2 + HTB); \
      if ((FIRST) && ui > 0) WAIT_V(28); else WAIT_V(8); \
      WAIT_L(0); BAR; MMA2(0, 0); BAR; SCHED; \
      LDA(At, 1, 1); STAGE(SB_(1, 0), b3); STAGE(SB_(1, 1), b3 + HTB); STAGE(SA_(1, 0), a3); \
      if (LAST) { FENCE; STORE_ROWS(cC, -esc, 0, 0); STORE_ROWS(cC, -esc, 0, 1); STORE_ROWS(cC, -esc, 0, 2); WAIT_V(20); } else if ((FIRST) && ui > 0) WAIT_V(16); else WAIT_V(8); \
      WAIT_L(0); BAR; MMA2(0, 1); BAR; SCHED; } while (0)
    f32x4 acc[2][2][4][2];
#pragma unroll
    for (int a = 0; a < 2; ++a)
#pragma unroll
        for (int b = 0; b < 2; ++b)
#pragma unroll
            for (int m = 0; m < 4; ++m)
#pragma unroll
                for (int n = 0; n < 2; ++n) acc[a][b][m][n] = ZERO4;
    f16x8 At[4][2], B0[2][2], B1[2][2];
    unsigned cA = (unsigned)pm * PANEL, cB = (unsigned)(NPA + pn) * PANEL;
    const unsigned coff = (unsigned)((wr * 64 + (lane >> 4) * 4) * (N * 4) + (wc * 64 + (lane & 15) * 4) * 4);
    unsigned cC = (unsigned)pm * 256u * (N * 4) + (unsigned)pn * 1024u, cPrev = cC;
    u64 rmask; int ecur, enext = 0, esc = 0, eprev = 0;
    POLL(pm, pn, pp0, rmask, ecur);
    { const unsigned a0 = cA + (unsigned)pp0 * 2u * BLK, b0 = cB + (unsigned)pp0 * 2u * BLK;
      STAGE(SB_(0, 0), b0); STAGE(SB_(0, 1), b0 + HTB); STAGE(SA_(0, 0), a0); STAGE(SA_(0, 1), a0 + HTB);
      if (wr == 1) BAR;
      WAIT_V(2); BAR;
      STAGE(SB_(1, 0), b0 + BLK); STAGE(SA_(1, 0), a0 + BLK); STAGE(SB_(1, 1), b0 + BLK + HTB);
      WAIT_V(6); BAR; }
    STAMP(sb + 1);
#pragma unroll 1
    for (int ui = 0; ui < ntiles; ++ui) {
        const bool has_next = ui + 1 < ntiles;
        const unsigned nA = has_next ? cA + (unsigned)pmstep * PANEL : cA, nB = has_next ? cB + (unsigned)pnstep * PANEL : cB;
        int it = 0;
        BODY(1, 0);
#pragma unroll 1
        for (it = 1; it < 31; ++it) {
            if (own_id >= 0 && it == 2 && ui > 0) {
                __builtin_amdgcn_fence(__ATOMIC_RELEASE, "agent"); WAIT_V(0);
                if (lane == 0) __hip_atomic_fetch_add((gu32*)(p.ws[0] + OFF_OWN + ((ui - 1) * NOWN + own_id) * 128), 1u, RLX_AGENT);
            }
            BODY(0, 0);
        }
        BODY(0, 1);
        STAMP(sb + 2 + ui); STAMPV(2 * sb + 20 + ui, pollt); STAMPV(2 * sb + 40 + ui, __builtin_amdgcn_s_memtime() - cyc0);
        cPrev = cC; eprev = -esc; ecur = enext;
        cC += (unsigned)pmstep * 256u * (N * 4) + (unsigned)pnstep * 1024u; cA = nA; cB = nB; pm += pmstep; pn += pnstep;
    }
    asm volatile("s_nop 15" ::: "memory"); SCHED;
    STORE_ROWS(cPrev, eprev, 0, 3); STORE_ROWS(cPrev, eprev, 1, 0); STORE_ROWS(cPrev, eprev, 1, 1); STORE_ROWS(cPrev, eprev, 1, 2); STORE_ROWS(cPrev, eprev, 1, 3);
    WAIT_V(0);
    STAMP(sb + 12);
    if (bad) { *(float*)(p.out[0] + cPrev + coff) = __builtin_nanf(""); WAIT_V(0); }
    if (own_id >= 0) {
        __builtin_amdgcn_fence(__ATOMIC_RELEASE, "agent"); WAIT_V(0);
        if (lane == 0) __hip_atomic_fetch_add((gu32*)(p.ws[0] + OFF_OWN + ((ntiles - 1) * NOWN + own_id) * 128), 1u, RLX_AGENT);
    }
    if (wr == 0) BAR;
    BAR;
}

__global__ __launch_bounds__(512, 2) void fused_kernel(Params p) {
    extern __shared__ __attribute__((aligned(16))) unsigned char shm[];
    LAS unsigned char* lds = (LAS unsigned char*)shm;
    gu32* cc = (gu32*)(p.ws[0] + OFF_CALL + blockIdx.x * 64);
    const unsigned call = (unsigned)__builtin_amdgcn_readfirstlane(__hip_atomic_load(cc, RLX_AGENT)) + 1u;
    WAIT_V(0); BAR;
    if (threadIdx.x == 0) __hip_atomic_store(cc, call, RLX_AGENT);
    STAMP(0);
    const int x = blockIdx.x & 7, o = blockIdx.x >> 3;
    int pm, pn, pmstep, pnstep, nt, pp0, own = -1;
    if ((int)blockIdx.x < 32 - NOWN) { if ((int)blockIdx.x < NCW) comm_role(p, lds, blockIdx.x, call); return; }
    own = o < 4 ? (int)blockIdx.x - (32 - NOWN) : -1;
    const int g = (o - 4) >> 2, rg = x & 3, strm = (x >> 2) * NGRP + g;
    if (own >= 0) { own_tile(own, p.rank, pm, pn); pmstep = NOWN / 8; pnstep = 0; nt = OWN_TILES; pp0 = 0; }
    else { pm = rg * 4 + (o & 3); pn = strm; pmstep = 0; pnstep = NSTR; nt = ROUNDS; pp0 = g * STAG; }
    gemm_role(p, lds, pm, pn, pmstep, pnstep, nt, own, pp0, call);
}

}

extern "C" void kernel_launch(int rank, int nranks, ncclComm_t comm, hipStream_t stream,
                              void* const* d_in, const int* in_sizes, int n_in,
                              void* d_out, int out_size, const int* in_dtypes, int out_dtype,
                              void* const* d_out_all, void* const* d_ws_all, size_t ws_bytes) {
    if (nranks != NR || (unsigned)rank >= (unsigned)NR || ws_bytes < WS_NEED || n_in < 2 ||
        in_sizes[0] != M * KS || in_sizes[1] != KS * N || out_size != M * N) {
        fprintf(stderr, "kernel_launch: unexpected problem shape\n");
        return;
    }
    Params p;
    p.x = (const float*)d_in[0]; p.w = (const float*)d_in[1]; p.rank = rank;
    for (int q = 0; q < NR; ++q) { p.ws[q] = (char*)d_ws_all[(rank + q) % NR]; p.out[q] = (char*)d_out_all[(rank + q) % NR]; }
    p.out[0] = (char*)d_out;
    HIP_CHECK(hipFuncSetAttribute((const void*)fused_kernel, hipFuncAttributeMaxDynamicSharedMemorySize, LDS_BYTES));
    hipLaunchKernelGGL(fused_kernel, dim3(256), dim3(512), LDS_BYTES, stream, p);
    HIP_CHECK(hipGetLastError());
}
```

```cpp
#include <hip/hip_runtime.h>
#include <rccl/rccl.h>
#include <cstdio>

#define HIP_CHECK(expr) do { hipError_t _e = (expr); if (_e != hipSuccess) { \
    fprintf(stderr, "HIP error %s at %s:%d\n", hipGetErrorString(_e), __FILE__, __LINE__); \
    return; } } while (0)

namespace {

#define LAS __attribute__((address_space(3)))
#define GAS __attribute__((address_space(1)))
typedef _Float16 f16x8 __attribute__((ext_vector_type(8)));
typedef _Float16 f16x2 __attribute__((ext_vector_type(2)));
typedef float f32x4 __attribute__((ext_vector_type(4)));
typedef unsigned u32x4 __attribute__((ext_vector_type(4)));
typedef unsigned long long u64;
typedef GAS unsigned gu32;
typedef GAS u64 gu64;

constexpr int M = 4096, N = 32768, KS = 512, NR = 8;
constexpr int NPA = 16, NPB = 128, NPAN = NPA + NPB;
#define PPAD 0
constexpr unsigned BLK = 32768, PANEL = 64 * BLK + PPAD, HTB = 16384;
constexpr int NGRP = 7, NSTR = 2 * NGRP, ROUNDS = 8, NSH = NSTR * ROUNDS;
#define NOWN 8
#define NCW (32 - NOWN)
#define WIN 2
#define STAG 0
constexpr int OWN_TILES = 32 / NOWN;
constexpr int LDS_BYTES = 8 * HTB + 256;
constexpr unsigned OFF_CALL = 0;
constexpr unsigned OFF_TMO = 16384;
constexpr unsigned OFF_OWN = 20480;
constexpr unsigned OFF_TICKET = 28672;
constexpr unsigned OFF_DONE = 32768;
constexpr unsigned OFF_FLAG = 65536;
constexpr unsigned OFF_STAGE = 1u << 20;
constexpr unsigned STAGE_PAR = NPAN * PANEL;
constexpr size_t WS_NEED = (size_t)OFF_STAGE + 2ull * STAGE_PAR;
constexpr unsigned OUT_BYTES = (unsigned)((size_t)M * N * 4);
constexpr u64 DEADLINE = 100000000ull;
constexpr int ZSENT = 0x40000000;
constexpr int J0 = 8 * (NPA + NSTR);
constexpr int J3 = J0 + (ROUNDS - 1) * 8 * NSTR;
constexpr int JTOT = J3 + 32 * 8;
#define OWN_GO (J0 + 8 * NCW)
#define GLDS_AUX 17
#define SRC_AUX 2
#define SLAB_AUX 19
#define OWN_AUX 16
#define C_AUX 2
#define ST_AUX 17

struct Params {
    const float* x; const float* w;
    char* ws[NR];
    char* out[NR];
    int rank;
};

#define STAMP(idx) do {} while (0)
#define STAMPV(idx, v) do {} while (0)
#define RLX_SYS __ATOMIC_RELAXED, __HIP_MEMORY_SCOPE_SYSTEM
#define RLX_AGENT __ATOMIC_RELAXED, __HIP_MEMORY_SCOPE_AGENT
#define WAIT_V(n) asm volatile("s_waitcnt vmcnt(" #n ")" ::: "memory")
#define WAIT_L(n) do { static_assert((n) == 0, ""); __builtin_amdgcn_s_waitcnt(0xC07F); asm volatile("" ::: "memory"); } while (0)
#define BAR __builtin_amdgcn_s_barrier()
#define SCHED __builtin_amdgcn_sched_barrier(0)
#define FENCE do { asm volatile("" ::: "memory"); SCHED; } while (0)

__device__ __forceinline__ unsigned pk16(float a, float b, int s) {
    f16x2 v; v.x = (_Float16)__builtin_ldexpf(a, s); v.y = (_Float16)__builtin_ldexpf(b, s);
    return __builtin_bit_cast(unsigned, v);
}
__device__ __forceinline__ __amdgpu_buffer_rsrc_t make_rsrc(const char* base, unsigned bytes) {
    const u64 a = (u64)base;
    const u64 u = ((u64)(unsigned)__builtin_amdgcn_readfirstlane((int)(a >> 32)) << 32) | (unsigned)__builtin_amdgcn_readfirstlane((int)a);
    return __builtin_amdgcn_make_buffer_rsrc((void*)u, 0, __builtin_amdgcn_readfirstlane((int)bytes), 0x00020000);
}
__device__ __forceinline__ bool give_up(gu32* tmo, unsigned call, u64 t0) {
    const bool bad = (unsigned)__builtin_amdgcn_readfirstlane(__hip_atomic_load(tmo, RLX_AGENT)) == call ||
                     __builtin_amdgcn_s_memrealtime() - t0 > DEADLINE;
    if (bad) __hip_atomic_store(tmo, call, RLX_AGENT);
    return bad;
}

__device__ __forceinline__ void draw_async(unsigned& t, gu32* ctr) {
    asm volatile("global_atomic_add %0, %1, %2, off sc0" : "=v"(t) : "v"(ctr), "v"(1u) : "memory");
}

struct Job { int panel, u, dq; };
__device__ __forceinline__ Job decode_job(int n, int own, int rank) {
    Job j; j.dq = -1;
    if (own >= 0) { j.u = n >> 1; j.dq = (own + n + 1) & 7; j.panel = NPA + NSH + 2 * ((rank + j.dq) & 7) + (n & 1); }
    else if (n < J0) { j.u = n / (NPA + NSTR); j.panel = n % (NPA + NSTR); }
    else { const int m = n - J0, c = m / (8 * NSTR), r = m % (8 * NSTR); j.u = r / NSTR; j.panel = NPA + (c + 1) * NSTR + r % NSTR; }
    return j;
}
__device__ __forceinline__ void own_tile(int t, int rank, int& pm, int& pn) {
    const int ow = t % NOWN, r = t / NOWN, x = ow & 7, idx = r * (NOWN / 8) + (ow >> 3);
    pm = (x & 3) * 4 + idx; pn = NSH + 2 * rank + (x >> 2);
}

#define NSTAT (J0 / NCW)
constexpr unsigned DRAWS = JTOT - NSTAT * NCW + NCW;
#define STR2(x) #x
#define STR(x) STR2(x)
#define WAIT_WIN asm volatile("s_waitcnt vmcnt(" STR(WIN) ")" ::: "memory")

__device__ __forceinline__ unsigned wave_max_u32(unsigned mx) {
#pragma unroll
    for (int k = 32; k >= 1; k >>= 1) mx = max(mx, (unsigned)__shfl_xor((int)mx, k));
    return __builtin_amdgcn_readfirstlane(mx);
}
__device__ __forceinline__ int scale_exp(unsigned mx) { return ((14 - ((int)(mx >> 23) - 127)) >> 4) * 16; }
__device__ __forceinline__ unsigned lds_load(LAS unsigned* w) { return __builtin_amdgcn_readfirstlane(__hip_atomic_load(w, __ATOMIC_RELAXED, __HIP_MEMORY_SCOPE_WORKGROUP)); }
__device__ __forceinline__ void lds_store(LAS unsigned* w, unsigned v) { __hip_atomic_store(w, v, __ATOMIC_RELAXED, __HIP_MEMORY_SCOPE_WORKGROUP); }

__device__ __forceinline__ void comm_role(const Params& p, LAS unsigned char* lds, int cw, int own, unsigned call) {
    const int tid = threadIdx.x, wid = __builtin_amdgcn_readfirstlane(tid >> 6), lane = tid & 63, fr = lane & 15, fq = lane >> 4;
    const unsigned par = call & 1u;
    gu32* tmo = (gu32*)(p.ws[0] + OFF_TMO);
    LAS unsigned* ctl = (LAS unsigned*)(lds + 4 * BLK);
    if (tid < 8) ctl[tid] = 0;
    WAIT_L(0); BAR; FENCE;
    gu64* flagq = (gu64*)(p.ws[wid] + OFF_FLAG) + par * (NPAN * 64) + ((8 - wid) & 7);
    const __amdgpu_buffer_rsrc_t rsD = make_rsrc(p.ws[wid] + OFF_STAGE + (size_t)par * STAGE_PAR + ((8 - wid) & 7) * BLK, STAGE_PAR - 7 * BLK);
    const __amdgpu_buffer_rsrc_t rsO = make_rsrc(p.out[wid], OUT_BYTES);
    int pidx = -1, psc = 0;
#define PUBLISH_PENDING do { if (pidx >= 0 && lane == 0) __hip_atomic_store(flagq + pidx, ((u64)call << 32) | (unsigned)psc, RLX_SYS); pidx = -1; } while (0)
    bool bad = false;
    if (wid > 0) {
        for (unsigned n = 0;; ++n) {
            if (lds_load(ctl) <= n) {
                WAIT_V(0); PUBLISH_PENDING;
                while (lds_load(ctl) <= n) __builtin_amdgcn_s_sleep(2);
            }
            FENCE;
            LAS unsigned* jw = ctl + 8 + 4 * (n & 3);
            const unsigned soff = lds_load(jw), fidx = lds_load(jw + 1), sc = lds_load(jw + 2), dq = lds_load(jw + 3);
            if (dq == 10u) break;
            if (dq >= 8u || dq == (unsigned)wid) {
                const LAS unsigned char* img = lds + (n & 3) * BLK + lane * 16;
#pragma unroll
                for (int g = 0; g < 4; ++g) {
                    u32x4 o[8];
#pragma unroll
                    for (int k = 0; k < 8; ++k) o[k] = *(const LAS u32x4*)(img + (g * 8 + k) * 1024);
                    if (g == 3) { WAIT_L(0); if (lane == 0) lds_store(ctl + wid, n + 1); }
#pragma unroll
                    for (int k = 0; k < 8; ++k) {
                        if (dq == 9u) __builtin_amdgcn_raw_buffer_store_b128(o[k], rsO, lane * 16, (int)(soff + (unsigned)(g * 8 + k) * (N * 4)), SLAB_AUX);
                        else __builtin_amdgcn_raw_buffer_store_b128(o[k], rsD, lane * 16, (int)(soff + (unsigned)(g * 8 + k) * 1024u), ST_AUX);
                        WAIT_WIN;
                        if (g == 0 && k == WIN - 1) PUBLISH_PENDING; }
                }
                if (dq != 9u) { pidx = (int)fidx; psc = (int)sc; }
            } else if (lane == 0) lds_store(ctl + wid, n + 1);
        }
        WAIT_V(0); PUBLISH_PENDING;
        if (own < 0 && lane == 0) __hip_atomic_store((gu32*)(p.ws[wid] + OFF_DONE + cw * 128) + ((8 - wid) & 7), call, RLX_SYS);
        return;
    }
    const unsigned tbase = (call - 1u) * DRAWS - (unsigned)(NSTAT * NCW);
    gu32* tick = (gu32*)(p.ws[0] + OFF_TICKET);
    const __amdgpu_buffer_rsrc_t rsX = make_rsrc((const char*)p.x, M * KS * 4), rsW = make_rsrc((const char*)p.w, KS * N * 4);
    const unsigned lA = (unsigned)(fr * KS + fq * 8) * 4u, lB = (unsigned)(fq * 8 * N + fr * 4) * 4u;
    unsigned n = 0;
    auto slot_free = [&]() {
        while (n >= 4u) { const unsigned c = __hip_atomic_load(ctl + 1 + (lane % 7), __ATOMIC_RELAXED, __HIP_MEMORY_SCOPE_WORKGROUP);
            if (__all(c + 3u >= n)) break; __builtin_amdgcn_s_sleep(2); }
        FENCE;
    };
    auto post = [&](unsigned soff, unsigned fidx, unsigned sc, unsigned dq) {
        if (lane == 0) { LAS unsigned* jw = ctl + 8 + 4 * (n & 3); jw[0] = soff; jw[1] = fidx; jw[2] = sc; jw[3] = dq; }
        WAIT_L(0);
        ++n; if (lane == 0) lds_store(ctl, n);
    };
#define LOAD_PIECE(dst, j, t) do { const bool _a = (j).panel < NPA; \
        const unsigned _b = _a ? (unsigned)((j).panel * 256 * KS + (j).u * 64) * 4u + ((t) & 3) * (32 * KS * 4) + ((t) >> 2) * (128 * KS * 4) \
                               : (unsigned)((j).u * 64 * N + ((j).panel - NPA) * 256) * 4u + ((t) & 3) * 256 + ((t) >> 2) * (32 * N * 4); \
        const unsigned _s0 = _a ? 16 : N * 4, _s1 = _a ? 128 : 2 * N * 4, _s2 = _a ? 16 * KS * 4 : 4 * N * 4; \
        _Pragma("unroll") for (int r = 0; r < 8; ++r) dst[r] = __builtin_bit_cast(f32x4, __builtin_amdgcn_raw_buffer_load_b128(_a ? rsX : rsW, (int)(_a ? lA : lB), \
            (int)(_b + (r & 1) * _s0 + ((r >> 1) & 1) * _s1 + (r >> 2) * _s2), SRC_AUX)); } while (0)
#define PIECE_MAX(mx, v) do { _Pragma("unroll") for (int r = 0; r < 8; ++r) { const u32x4 _u = __builtin_bit_cast(u32x4, v[r]) & 0x7fffffffu; \
        mx = max(max(mx, _u.x), _u.y); mx = max(max(mx, _u.z), _u.w); } asm volatile("" : "+v"(mx)); } while (0)
#define PUT_FRAG(o, d, loff) do { *(LAS u32x4*)(img + (d)) = o; __builtin_amdgcn_raw_buffer_store_b128(o, rsD, lane * 16, (int)((loff) + (d)), ST_AUX); } while (0)
#define CONV_PIECE(v, isA, t, s, loff) do { \
        if (isA) { _Pragma("unroll") for (int i = 0; i < 4; ++i) { const f32x4 a = v[2 * i], c = v[2 * i + 1]; u32x4 o; \
            o.x = pk16(a.x, a.y, s); o.y = pk16(a.z, a.w, s); o.z = pk16(c.x, c.y, s); o.w = pk16(c.z, c.w, s); PUT_FRAG(o, (t) * 4096 + i * 1024, loff); } } \
        else { _Pragma("unroll") for (int i = 0; i < 4; ++i) { u32x4 o; \
            o.x = pk16(v[0][i], v[1][i], s); o.y = pk16(v[2][i], v[3][i], s); o.z = pk16(v[4][i], v[5][i], s); o.w = pk16(v[6][i], v[7][i], s); \
            PUT_FRAG(o, ((t) & 3) * 4096 + ((t) >> 2) * 1024 + (i >> 1) * HTB + (i & 1) * 2048, loff); } } \
        FENCE; } while (0)
    unsigned t1 = own < 0 ? cw : 0, t2 = own < 0 ? NCW + cw : 1, drawn = 0;
    const unsigned jend = own < 0 ? J3 : 16;
    if (own >= 0) {
        const u64 t0 = __builtin_amdgcn_s_memrealtime();
        while ((unsigned)__builtin_amdgcn_readfirstlane(__hip_atomic_load(tick, RLX_AGENT)) - tbase < (unsigned)OWN_GO && !give_up(tmo, call, t0)) __builtin_amdgcn_s_sleep(32);
    }
    f32x4 v[4][8];
    Job cur = decode_job((int)t1, own, p.rank), nxt = cur;
    LOAD_PIECE(v[0], cur, 0); LOAD_PIECE(v[1], cur, 1); LOAD_PIECE(v[2], cur, 2); LOAD_PIECE(v[3], cur, 3);
#pragma unroll 1
    while (t1 < jend) {
        const bool dyn = own < 0 && n + 2u >= (unsigned)NSTAT;
        if (dyn && lane == 0) draw_async(drawn, tick);
        nxt = t2 < jend ? decode_job((int)t2, own, p.rank) : cur;
        const bool isA = cur.panel < NPA;
        const unsigned soff = (unsigned)cur.panel * PANEL + (unsigned)cur.u * 8u * BLK;
        const unsigned loff = cur.dq <= 0 ? soff : 0xf0000000u;
        slot_free();
        LAS unsigned char* img = lds + (n & 3) * BLK + lane * 16;
        unsigned mx = 0; PIECE_MAX(mx, v[0]);
        int s = scale_exp(wave_max_u32(mx));
#pragma unroll
        for (int t = 0; t < 8; ++t) {
            if (t) PIECE_MAX(mx, v[t & 3]);
            CONV_PIECE(v[t & 3], isA, t, s, loff);
            if (t < 4) LOAD_PIECE(v[t & 3], cur, t + 4); else LOAD_PIECE(v[t & 3], nxt, t - 4);
            FENCE;
            if (t == 4) PUBLISH_PENDING;
        }
        mx = wave_max_u32(mx);
        if (__builtin_expect(mx >= 0x7f800000u || scale_exp(mx) != s, 0)) {
            f32x4 w[8]; mx = 0;
#pragma unroll 1
            for (int t = 0; t < 8; ++t) { LOAD_PIECE(w, cur, t);
#pragma unroll
                for (int r = 0; r < 8; ++r) { const u32x4 u = __builtin_bit_cast(u32x4, w[r]) & 0x7fffffffu;
#pragma unroll
                    for (int c = 0; c < 4; ++c) { const unsigned b = u[c]; mx = max(mx, b < 0x7f800000u ? b : 0u); } } }
            mx = wave_max_u32(mx); s = scale_exp(mx);
            WAIT_V(0);
#pragma unroll 1
            for (int t = 0; t < 8; ++t) { LOAD_PIECE(w, cur, t); CONV_PIECE(w, isA, t, s, loff); }
        }
        const int sc = mx ? s : ZSENT, fidx = cur.panel * 64 + cur.u * 8;
        post(soff, (unsigned)fidx, (unsigned)sc, cur.dq < 0 ? 8u : (unsigned)cur.dq);
        STAMP(n);
        if (cur.dq <= 0) { pidx = fidx; psc = sc; }
        asm volatile("" : "+v"(drawn));
        t1 = t2; t2 = dyn ? __builtin_amdgcn_readfirstlane(drawn) - tbase : t2 + (own < 0 ? NCW : 1); cur = nxt;
    }
    WAIT_V(0); PUBLISH_PENDING;
    if (own >= 0) { slot_free(); post(0u, 0u, 0u, 10u); return; }
    STAMP(99);
    unsigned ready = 0;
    while (t1 < (unsigned)JTOT) {
        const bool pending = t2 < (unsigned)JTOT;
        if (pending && lane == 0) draw_async(drawn, tick);
        const int tile = (int)(t1 - J3) >> 3;
        if (!((ready >> tile) & 1u)) {
            gu32* cnt = (gu32*)(p.ws[0] + OFF_OWN + (lane & 31) * 128);
            const u64 t0 = __builtin_amdgcn_s_memrealtime();
            for (;;) {
                ready = (unsigned)__ballot(__hip_atomic_load(cnt, RLX_AGENT) >= 8u * call);
                if ((ready >> tile) & 1u) break;
                if (give_up(tmo, call, t0)) { bad = true; ready = ~0u; break; }
                __builtin_amdgcn_s_sleep(8);
            }
        }
        int pm, pn; own_tile(tile, p.rank, pm, pn);
        const unsigned off = (unsigned)((pm * 256 + (int)((t1 - J3) & 7) * 32) * (N * 4) + pn * 1024);
        slot_free();
        if (!bad) {
#pragma unroll
            for (int k = 0; k < 32; ++k) __builtin_amdgcn_raw_ptr_buffer_load_lds(rsO, (LAS void*)(lds + (n & 3) * BLK + k * 1024), 16, lane * 16, (int)(off + (unsigned)k * (N * 4)), 0, 18);
        } else {
            for (int k = 0; k < 32; ++k) *(LAS u32x4*)(lds + (n & 3) * BLK + k * 1024 + lane * 16) = (u32x4){0x7fc00000u, 0x7fc00000u, 0x7fc00000u, 0x7fc00000u};
        }
        WAIT_V(0); asm volatile("" : "+v"(drawn));
        post(off, 0u, 0u, 9u);
        STAMP(n);
        t1 = t2; t2 = pending ? __builtin_amdgcn_readfirstlane(drawn) - tbase : ~0u;
    }
    slot_free(); post(0u, 0u, 0u, 10u);
    STAMP(100);
    {
        const u64 t0 = __builtin_amdgcn_s_memrealtime();
        gu32* dn = (gu32*)(p.ws[0] + OFF_DONE + cw * 128) + (lane & 7);
        for (;;) {
            const unsigned val = __hip_atomic_load(dn, RLX_SYS);
            if (__all((lane & 7) == 0 || val >= call)) break;
            if (give_up(tmo, call, t0)) { bad = true; break; }
            __builtin_amdgcn_s_sleep(4);
        }
        __builtin_amdgcn_fence(__ATOMIC_ACQUIRE, "");
        STAMP(101);
        if (bad && lane == 0) ((float*)p.out[0])[(size_t)(NSH + 2 * p.rank) * 256 + cw] = __builtin_nanf("");
    }
}

__device__ __forceinline__ void gemm_role(const Params& p, LAS unsigned char* lds, int pm, int pn, int pmstep, int pnstep, int ntiles, int own_id, int pp0, unsigned call) {
    const int tid = threadIdx.x, wid = __builtin_amdgcn_readfirstlane(tid >> 6), lane = tid & 63, wr = wid >> 2, wc = wid & 3;
    const unsigned par = call & 1u;
    const unsigned voff = (unsigned)wid * 2048u + (unsigned)lane * 16u;
    const unsigned ldsw = (unsigned)wid * 2048u;
    const int aoff = wr * 8192 + lane * 16, boff = wc * 4096 + lane * 16;
    gu32* tmo = (gu32*)(p.ws[0] + OFF_TMO);
    const __amdgpu_buffer_rsrc_t rsS = make_rsrc(p.ws[0] + OFF_STAGE + (size_t)par * STAGE_PAR, STAGE_PAR);
    const __amdgpu_buffer_rsrc_t rsC = make_rsrc(p.out[0], OUT_BYTES);
    gu64* flags = (gu64*)(p.ws[0] + OFF_FLAG) + par * (NPAN * 64);
    bool bad = false;
#define SA_(b, h) (((b) * 2 + (h)) * HTB)
#define SB_(b, h) ((4 + (b) * 2 + (h)) * HTB)
#define STAGE(bufoff, soff) do { \
    __builtin_amdgcn_raw_ptr_buffer_load_lds(rsS, (LAS void*)(lds + (bufoff) + ldsw), 16, (int)voff, (int)(soff), 0, GLDS_AUX); \
    __builtin_amdgcn_raw_ptr_buffer_load_lds(rsS, (LAS void*)(lds + (bufoff) + ldsw), 16, (int)voff, (int)(soff), 1024, GLDS_AUX); } while (0)
#define LDA(dst, b, h) do { _Pragma("unroll") for (int m = 0; m < 4; ++m) _Pragma("unroll") for (int k = 0; k < 2; ++k) dst[m][k] = *(const LAS f16x8*)(lds + SA_(b, h) + aoff + m * 2048 + k * 1024); } while (0)
#define LDB(dst, b, h) do { _Pragma("unroll") for (int n = 0; n < 2; ++n) _Pragma("unroll") for (int k = 0; k < 2; ++k) dst[n][k] = *(const LAS f16x8*)(lds + SB_(b, h) + boff + n * 2048 + k * 1024); } while (0)
#define POLL_ACC
#define ZERO4 ((f32x4){0.f, 0.f, 0.f, 0.f})
#define MF2(Z, i, R, m, n, Bt) do { if (Z) asm volatile("v_mfma_f32_16x16x32_f16 %0, %1, %2, 0\n\tv_mfma_f32_16x16x32_f16 %0, %3, %4, %0" : "+{" R "}"(acc[i]) : "v"(At[m][0]), "v"(Bt[n][0]), "v"(At[m][1]), "v"(Bt[n][1])); \
      else asm volatile("v_mfma_f32_16x16x32_f16 %0, %1, %2, %0\n\tv_mfma_f32_16x16x32_f16 %0, %3, %4, %0" : "+{" R "}"(acc[i]) : "v"(At[m][0]), "v"(Bt[n][0]), "v"(At[m][1]), "v"(Bt[n][1])); } while (0)
#define MMA2_0(Z) do { __builtin_amdgcn_s_setprio(1); \
      MF2(Z, 0, "v[128:131]", 0, 0, B0); MF2(Z, 1, "v[132:135]", 0, 1, B0); MF2(Z, 8, "v[160:163]", 0, 0, B1); MF2(Z, 9, "v[164:167]", 0, 1, B1); \
      MF2(Z, 2, "v[136:139]", 1, 0, B0); MF2(Z, 3, "v[140:143]", 1, 1, B0); MF2(Z, 10, "v[168:171]", 1, 0, B1); MF2(Z, 11, "v[172:175]", 1, 1, B1); \
      MF2(Z, 4, "v[144:147]", 2, 0, B0); MF2(Z, 5, "v[148:151]", 2, 1, B0); MF2(Z, 12, "v[176:179]", 2, 0, B1); MF2(Z, 13, "v[180:183]", 2, 1, B1); \
      MF2(Z, 6, "v[152:155]", 3, 0, B0); MF2(Z, 7, "v[156:159]", 3, 1, B0); MF2(Z, 14, "v[184:187]", 3, 0, B1); MF2(Z, 15, "v[188:191]", 3, 1, B1); \
      __builtin_amdgcn_s_setprio(0); } while (0)
#define MMA2_1(Z) do { __builtin_amdgcn_s_setprio(1); \
      MF2(Z, 16, "v[192:195]", 0, 0, B0); MF2(Z, 17, "v[196:199]", 0, 1, B0); MF2(Z, 24, "v[224:227]", 0, 0, B1); MF2(Z, 25, "v[228:231]", 0, 1, B1); \
      MF2(Z, 18, "v[200:203]", 1, 0, B0); MF2(Z, 19, "v[204:207]", 1, 1, B0); MF2(Z, 26, "v[232:235]", 1, 0, B1); MF2(Z, 27, "v[236:239]", 1, 1, B1); \
      MF2(Z, 20, "v[208:211]", 2, 0, B0); MF2(Z, 21, "v[212:215]", 2, 1, B0); MF2(Z, 28, "v[240:243]", 2, 0, B1); MF2(Z, 29, "v[244:247]", 2, 1, B1); \
      MF2(Z, 22, "v[216:219]", 3, 0, B0); MF2(Z, 23, "v[220:223]", 3, 1, B0); MF2(Z, 30, "v[248:251]", 3, 0, B1); MF2(Z, 31, "v[252:255]", 3, 1, B1); \
      __builtin_amdgcn_s_setprio(0); } while (0)
#define MMA2(Z, ai) MMA2_##ai(Z)
#define STORE_ROWS(cbase, ex, ai, m) do { _Pragma("unroll") for (int j = 0; j < 4; ++j) { \
      f32x4 o = {__builtin_ldexpf(acc[(ai) * 16 + (m) * 2][j], ex), __builtin_ldexpf(acc[(ai) * 16 + (m) * 2 + 1][j], ex), __builtin_ldexpf(acc[(ai) * 16 + 8 + (m) * 2][j], ex), __builtin_ldexpf(acc[(ai) * 16 + 8 + (m) * 2 + 1][j], ex)}; \
      if (own_id >= 0) __builtin_amdgcn_raw_buffer_store_b128(__builtin_bit_cast(u32x4, o), rsC, (int)coff, (int)((cbase) + (unsigned)((ai) * 128 + (m) * 16 + j) * (N * 4)), OWN_AUX); \
      else __builtin_amdgcn_raw_buffer_store_b128(__builtin_bit_cast(u32x4, o), rsC, (int)coff, (int)((cbase) + (unsigned)((ai) * 128 + (m) * 16 + j) * (N * 4)), C_AUX); } } while (0)
#define POLL_SLEEP 16
#define POLL(a, b, pos2, maskv, expv) do { const u64 _t0 = __builtin_amdgcn_s_memrealtime(); bool _ok = false; \
      for (int _n = 0;; ++_n) { \
        if (_n == 0 || (lane >> 4) == ((pos2) >> 3)) {     \
          const u64 _fa = __hip_atomic_load(flags + (a) * 64 + lane, RLX_SYS), _fb = __hip_atomic_load(flags + (NPA + (b)) * 64 + lane, RLX_SYS); \
          _ok = (unsigned)(_fa >> 32) == call && (unsigned)(_fb >> 32) == call; \
          expv = ((int)_fa == ZSENT || (int)_fb == ZSENT) ? ZSENT : (int)_fa + (int)_fb; } \
        maskv = __ballot(_ok); \
        if (((maskv >> (2 * (pos2))) & 3ull) == 3ull) break; \
        if (bad || ((_n & 63) == 63 && give_up(tmo, call, _t0))) { bad = true; maskv = ~0ull; expv = ZSENT; break; } \
        __builtin_amdgcn_s_sleep(POLL_SLEEP); } POLL_ACC; } while (0)
#define SET_CMASK cmask = __ballot(ecur != esc && ecur != ZSENT)
#define RESCALE(pos_) do { int _d; asm volatile( \
        "s_bitcmp1_b64 %[cm], %[pos]\n\t" \
        "s_cbranch_scc0 1f\n\t" \
        "s_nop 15\n\t" \
        "v_readlane_b32 %[d], %[ec], %[pos]\n\t" \
        "s_nop 3\n\t" \
        "v_cmp_ne_u32 %[cm], %[d], %[ec]\n\t" \
        "v_cmp_ne_u32 vcc, 2.0, %[ec]\n\t" \
        "s_xor_b32 %[es], %[es], %[d]\n\t" \
        "s_xor_b32 %[d], %[es], %[d]\n\t" \
        "s_xor_b32 %[es], %[es], %[d]\n\t" \
        "s_sub_i32 %[d], %[es], %[d]\n\t" \
        "s_and_b64 %[cm], %[cm], vcc\n\t" \
        ".set gr_, 128\n\t" \
        ".rept 128\n\t" \
        "v_ldexp_f32 v[gr_], v[gr_], %[d]\n\t" \
        ".set gr_, gr_ + 1\n\t" \
        ".endr\n\t" \
        "s_nop 3\n\t" \
        "1:\n\t" \
        : [cm] "+s"(cmask), [es] "+s"(esc), [d] "=&s"(_d), \
          "+{v[128:131]}"(acc[0]), "+{v[132:135]}"(acc[1]), "+{v[136:139]}"(acc[2]), "+{v[140:143]}"(acc[3]), "+{v[144:147]}"(acc[4]), "+{v[148:151]}"(acc[5]), "+{v[152:155]}"(acc[6]), "+{v[156:159]}"(acc[7]), \
          "+{v[160:163]}"(acc[8]), "+{v[164:167]}"(acc[9]), "+{v[168:171]}"(acc[10]), "+{v[172:175]}"(acc[11]), "+{v[176:179]}"(acc[12]), "+{v[180:183]}"(acc[13]), "+{v[184:187]}"(acc[14]), "+{v[188:191]}"(acc[15]), \
          "+{v[192:195]}"(acc[16]), "+{v[196:199]}"(acc[17]), "+{v[200:203]}"(acc[18]), "+{v[204:207]}"(acc[19]), "+{v[208:211]}"(acc[20]), "+{v[212:215]}"(acc[21]), "+{v[216:219]}"(acc[22]), "+{v[220:223]}"(acc[23]), \
          "+{v[224:227]}"(acc[24]), "+{v[228:231]}"(acc[25]), "+{v[232:235]}"(acc[26]), "+{v[236:239]}"(acc[27]), "+{v[240:243]}"(acc[28]), "+{v[244:247]}"(acc[29]), "+{v[248:251]}"(acc[30]), "+{v[252:255]}"(acc[31]) \
        : [pos] "s"(pos_), [ec] "v"(ecur) : "vcc", "scc"); } while (0)
#define BODY(FIRST, LAST, EXTRA) do { \
      const int pp = (it + pp0) & 31, ppn = (pp + 1) & 31; \
      const unsigned a1 = cA + (unsigned)(pp * 2 + 1) * BLK; \
      const unsigned a2 = ((LAST) ? nA : cA) + (unsigned)(ppn * 2) * BLK, b2 = ((LAST) ? nB : cB) + (unsigned)(ppn * 2) * BLK; \
      const unsigned a3 = a2 + BLK, b3 = b2 + BLK; \
      LDB(B0, 0, 0); LDB(B1, 0, 1); SCHED; LDA(At, 0, 0); STAGE(SA_(1, 1), a1 + HTB); FENCE; \
      if (FIRST) { const int e = __builtin_amdgcn_readlane(ecur, 2 * pp); esc = e == ZSENT ? 0 : e; SET_CMASK; } else RESCALE(2 * pp); \
      if ((FIRST) && ui > 0) { STORE_ROWS(cPrev, eprev, 0, 3); STORE_ROWS(cPrev, eprev, 1, 0); STORE_ROWS(cPrev, eprev, 1, 1); WAIT_V(32); } else WAIT_V(8); \
      WAIT_L(0); BAR; MMA2(FIRST, 0); BAR; SCHED; \
      LDA(At, 0, 1); FENCE; \
      if (LAST) { if (has_next) {     \
          asm volatile("" : "+v"(nfa), "+v"(nfb)); \
          rmask = __ballot((unsigned)(nfa >> 32) == call && (unsigned)(nfb >> 32) == call); \
          enext = ((int)nfa == ZSENT || (int)nfb == ZSENT) ? ZSENT : (int)nfa + (int)nfb; \
          if (((rmask >> (2 * ppn)) & 3ull) != 3ull) POLL(pm + pmstep, pn + pnstep, ppn, rmask, enext); } } \
      else if (__builtin_expect(((rmask >> (2 * ppn)) & 3ull) != 3ull, 0)) { POLL(pm, pn, ppn, rmask, ecur); SET_CMASK; } \
      STAGE(SB_(0, 0), b2); STAGE(SB_(0, 1), b2 + HTB); STAGE(SA_(0, 0), a2); \
      if ((FIRST) && ui > 0) { FENCE; STORE_ROWS(cPrev, eprev, 1, 2); STORE_ROWS(cPrev, eprev, 1, 3); WAIT_V(40); } else WAIT_V(8); \
      WAIT_L(0); BAR; MMA2(FIRST, 1); BAR; SCHED; \
      LDB(B0, 1, 0); LDB(B1, 1, 1); SCHED; LDA(At, 1, 0); FENCE; EXTRA; STAGE(SA_(0, 1), a2 + HTB); FENCE; \
      RESCALE(2 * pp + 1); \
      if ((FIRST) && ui > 0) WAIT_V(28); else WAIT_V(8); \
      WAIT_L(0); BAR; MMA2(0, 0); BAR; SCHED; \
      LDA(At, 1, 1); STAGE(SB_(1, 0), b3); STAGE(SB_(1, 1), b3 + HTB); STAGE(SA_(1, 0), a3); \
      if (LAST) { FENCE; STORE_ROWS(cC, -esc, 0, 0); STORE_ROWS(cC, -esc, 0, 1); STORE_ROWS(cC, -esc, 0, 2); WAIT_V(20); } else if ((FIRST) && ui > 0) WAIT_V(16); else WAIT_V(8); \
      WAIT_L(0); BAR; MMA2(0, 1); BAR; SCHED; } while (0)
    f32x4 acc[32];
#pragma unroll
    for (int i = 0; i < 32; ++i) acc[i] = ZERO4;
    f16x8 At[4][2], B0[2][2], B1[2][2];
    unsigned cA = (unsigned)pm * PANEL, cB = (unsigned)(NPA + pn) * PANEL;
    const unsigned coff = (unsigned)((wr * 64 + (lane >> 4) * 4) * (N * 4) + (wc * 64 + (lane & 15) * 4) * 4);
    unsigned cC = (unsigned)pm * 256u * (N * 4) + (unsigned)pn * 1024u, cPrev = cC;
    u64 rmask, cmask = 0, nfa = 0, nfb = 0; int ecur, enext = 0, esc = 0, eprev = 0;
    POLL(pm, pn, pp0, rmask, ecur);
    { const unsigned a0 = cA + (unsigned)pp0 * 2u * BLK, b0 = cB + (unsigned)pp0 * 2u * BLK;
      STAGE(SB_(0, 0), b0); STAGE(SB_(0, 1), b0 + HTB); STAGE(SA_(0, 0), a0); STAGE(SA_(0, 1), a0 + HTB);
      if (wr == 1) BAR;
      WAIT_V(2); BAR;
      STAGE(SB_(1, 0), b0 + BLK); STAGE(SA_(1, 0), a0 + BLK); STAGE(SB_(1, 1), b0 + BLK + HTB);
      WAIT_V(6); BAR; }
    STAMP(sb + 1);
#pragma unroll 1
    for (int ui = 0; ui < ntiles; ++ui) {
        const bool has_next = ui + 1 < ntiles;
        const unsigned nA = has_next ? cA + (unsigned)pmstep * PANEL : cA, nB = has_next ? cB + (unsigned)pnstep * PANEL : cB;
        int it = 0;
        BODY(1, 0, {});
        STAMPV(2 * sb + 60 + ui, __builtin_amdgcn_s_memtime() - cya);
#pragma unroll 1
        for (it = 1; it < 31; ++it) {
            BODY(0, 0, {
                if (it == 30) {
                    const gu64* fa = flags + (has_next ? pm + pmstep : pm) * 64 + lane; const gu64* fb = flags + (NPA + (has_next ? pn + pnstep : pn)) * 64 + lane;
                    asm volatile("global_load_dwordx2 %0, %2, off sc0 sc1\n\tglobal_load_dwordx2 %1, %3, off sc0 sc1" : "=&v"(nfa), "=&v"(nfb) : "v"(fa), "v"(fb) : "memory");
                    if (own_id >= 0 && ui > 0 && lane == 0)
                        __hip_atomic_fetch_add((gu32*)(p.ws[0] + OFF_OWN + ((ui - 1) * NOWN + own_id) * 128), 1u, RLX_AGENT);
                } });
        }
        STAMPV(2 * sb + 70 + ui, __builtin_amdgcn_s_memtime() - cya);
        BODY(0, 1, {});
        STAMPV(2 * sb + 80 + ui, __builtin_amdgcn_s_memtime() - cya);
        STAMP(sb + 2 + ui); STAMPV(2 * sb + 20 + ui, pollt); STAMPV(2 * sb + 40 + ui, __builtin_amdgcn_s_memtime() - cyc0);
        cPrev = cC; eprev = -esc; ecur = enext;
        cC += (unsigned)pmstep * 256u * (N * 4) + (unsigned)pnstep * 1024u; cA = nA; cB = nB; pm += pmstep; pn += pnstep;
    }
    asm volatile("s_nop 15" ::: "memory"); SCHED;
    STORE_ROWS(cPrev, eprev, 0, 3); STORE_ROWS(cPrev, eprev, 1, 0); STORE_ROWS(cPrev, eprev, 1, 1); STORE_ROWS(cPrev, eprev, 1, 2); STORE_ROWS(cPrev, eprev, 1, 3);
    WAIT_V(0);
    STAMP(sb + 12);
    if (bad) { *(float*)(p.out[0] + cPrev + coff) = __builtin_nanf(""); WAIT_V(0); }
    if (own_id >= 0) {
        if (lane == 0) __hip_atomic_fetch_add((gu32*)(p.ws[0] + OFF_OWN + ((ntiles - 1) * NOWN + own_id) * 128), 1u, RLX_AGENT);
    }
    if (wr == 0) BAR;
    BAR;
}

__global__ __launch_bounds__(512, 2) void fused_kernel(Params p) {
    extern __shared__ __attribute__((aligned(16))) unsigned char shm[];
    LAS unsigned char* lds = (LAS unsigned char*)shm;
    gu32* cc = (gu32*)(p.ws[0] + OFF_CALL + blockIdx.x * 64);
    const unsigned call = (unsigned)__builtin_amdgcn_readfirstlane(__hip_atomic_load(cc, RLX_AGENT)) + 1u;
    WAIT_V(0); BAR;
    if (threadIdx.x == 0) __hip_atomic_store(cc, call, RLX_AGENT);
    STAMP(0);
    const int x = blockIdx.x & 7, o = blockIdx.x >> 3;
    int pm, pn, pmstep, pnstep, nt, pp0, own = -1;
    static_assert(NOWN == 8, "one own-tile workgroup per XCD");
    if (o < 4) {
        own = (int)blockIdx.x - (32 - NOWN);
        if ((int)blockIdx.x < NCW || own >= 0) comm_role(p, lds, blockIdx.x, own, call);
        if (own < 0) return;
        WAIT_L(0); BAR; FENCE;
    }
    const int g = (o - 4) >> 2, rg = x & 3, strm = (x >> 2) * NGRP + g;
    if (own >= 0) { own_tile(own, p.rank, pm, pn); pmstep = NOWN / 8; pnstep = 0; nt = OWN_TILES; pp0 = 0; }
    else { pm = rg * 4 + (o & 3); pn = strm; pmstep = 0; pnstep = NSTR; nt = ROUNDS; pp0 = g * STAG; }
    gemm_role(p, lds, pm, pn, pmstep, pnstep, nt, own, pp0, call);
}

}

extern "C" void kernel_launch(int rank, int nranks, ncclComm_t comm, hipStream_t stream,
                              void* const* d_in, const int* in_sizes, int n_in,
                              void* d_out, int out_size, const int* in_dtypes, int out_dtype,
                              void* const* d_out_all, void* const* d_ws_all, size_t ws_bytes) {
    if (nranks != NR || (unsigned)rank >= (unsigned)NR || ws_bytes < WS_NEED || n_in < 2 ||
        in_sizes[0] != M * KS || in_sizes[1] != KS * N || out_size != M * N) {
        fprintf(stderr, "kernel_launch: unexpected problem shape\n");
        return;
    }
    Params p;
    p.x = (const float*)d_in[0]; p.w = (const float*)d_in[1]; p.rank = rank;
    for (int q = 0; q < NR; ++q) { p.ws[q] = (char*)d_ws_all[(rank + q) % NR]; p.out[q] = (char*)d_out_all[(rank + q) % NR]; }
    p.out[0] = (char*)d_out;
    HIP_CHECK(hipFuncSetAttribute((const void*)fused_kernel, hipFuncAttributeMaxDynamicSharedMemorySize, LDS_BYTES));
    hipLaunchKernelGGL(fused_kernel, dim3(256), dim3(512), LDS_BYTES, stream, p);
    HIP_CHECK(hipGetLastError());
}
```

```cpp
#include <hip/hip_runtime.h>
#include <rccl/rccl.h>
#include <cstdio>

#define HIP_CHECK(expr) do { hipError_t _e = (expr); if (_e != hipSuccess) { \
    fprintf(stderr, "HIP error %s at %s:%d\n", hipGetErrorString(_e), __FILE__, __LINE__); \
    return; } } while (0)

namespace {

#define LAS __attribute__((address_space(3)))
#define GAS __attribute__((address_space(1)))
typedef _Float16 f16x8 __attribute__((ext_vector_type(8)));
typedef _Float16 f16x2 __attribute__((ext_vector_type(2)));
typedef float f32x4 __attribute__((ext_vector_type(4)));
typedef unsigned u32x4 __attribute__((ext_vector_type(4)));
typedef unsigned long long u64;
typedef GAS unsigned gu32;
typedef GAS u64 gu64;

constexpr int M = 4096, N = 32768, KS = 512, NR = 8;
constexpr int NPA = 16, NPB = 128, NPAN = NPA + NPB;
constexpr unsigned BLK = 32768, PANEL = 64 * BLK, HTB = 16384;
#define DX 1
constexpr int ROUNDS = 8, NSTR = 14 - DX, NSH = NSTR * ROUNDS + 4 * DX;
constexpr int NOWN = 8, OWN_TILES = 4 + DX;
#define NCW 24
#define WIN 2
constexpr int LDS_BYTES = 8 * HTB + 256;
constexpr unsigned OFF_CALL = 0;
constexpr unsigned OFF_TMO = 16384;
constexpr unsigned OFF_OWN = 20480;
constexpr unsigned OFF_TICKET = 28672;
constexpr unsigned OFF_DONE = 32768;
constexpr unsigned OFF_FLAG = 65536;
constexpr unsigned OFF_STAGE = 1u << 20;
constexpr unsigned STAGE_PAR = NPAN * PANEL;
constexpr size_t WS_NEED = (size_t)OFF_STAGE + 2ull * STAGE_PAR;
constexpr unsigned OUT_BYTES = (unsigned)((size_t)M * N * 4);
constexpr u64 DEADLINE = 100000000ull;
constexpr int ZSENT = 0x40000000;
constexpr int J0 = 8 * (NPA + 14);
constexpr int J3 = 8 * (NPA + NSH);
constexpr int JTOT = J3 + NOWN * OWN_TILES * 8;
#define OWN_GO (J0 + 8 * NCW)
#define GLDS_AUX 17
#define SRC_AUX 2
#define SLAB_AUX 19
#define OWN_AUX 18
#define C_AUX 2
#define ST_AUX 17

struct Params {
    const float* x; const float* w;
    char* ws[NR];
    char* out[NR];
    int rank;
};

#define STAMP(idx) do {} while (0)
#define STAMPV(idx, v) do {} while (0)
#define RLX_SYS __ATOMIC_RELAXED, __HIP_MEMORY_SCOPE_SYSTEM
#define RLX_AGENT __ATOMIC_RELAXED, __HIP_MEMORY_SCOPE_AGENT
#define WAIT_V(n) asm volatile("s_waitcnt vmcnt(" #n ")" ::: "memory")
#define WAIT_L(n) do { static_assert((n) == 0, ""); __builtin_amdgcn_s_waitcnt(0xC07F); asm volatile("" ::: "memory"); } while (0)
#define BAR __builtin_amdgcn_s_barrier()
#define SCHED __builtin_amdgcn_sched_barrier(0)
#define FENCE do { asm volatile("" ::: "memory"); SCHED; } while (0)

__device__ __forceinline__ unsigned pk16(float a, float b, int s) {
    f16x2 v; v.x = (_Float16)__builtin_ldexpf(a, s); v.y = (_Float16)__builtin_ldexpf(b, s);
    return __builtin_bit_cast(unsigned, v);
}
__device__ __forceinline__ __amdgpu_buffer_rsrc_t make_rsrc(const char* base, unsigned bytes) {
    const u64 a = (u64)base;
    const u64 u = ((u64)(unsigned)__builtin_amdgcn_readfirstlane((int)(a >> 32)) << 32) | (unsigned)__builtin_amdgcn_readfirstlane((int)a);
    return __builtin_amdgcn_make_buffer_rsrc((void*)u, 0, __builtin_amdgcn_readfirstlane((int)bytes), 0x00020000);
}
__device__ __forceinline__ bool give_up(gu32* tmo, unsigned call, u64 t0) {
    const bool bad = (unsigned)__builtin_amdgcn_readfirstlane(__hip_atomic_load(tmo, RLX_AGENT)) == call ||
                     __builtin_amdgcn_s_memrealtime() - t0 > DEADLINE;
    if (bad) __hip_atomic_store(tmo, call, RLX_AGENT);
    return bad;
}

__device__ __forceinline__ void draw_async(unsigned& t, gu32* ctr) {
    asm volatile("global_atomic_add %0, %1, %2, off sc0" : "=v"(t) : "v"(ctr), "v"(1u) : "memory");
}

__device__ __forceinline__ int own_panel(int h, int rank) { return h < 2 ? NSH + 2 * rank + h : NSH + 16 + (rank >> 1); }
struct Job { int panel, u, dq; };
__device__ __forceinline__ Job decode_job(int n, int own, int rank) {
    Job j; j.dq = -1;
    if (own >= 0) { j.u = n / (2 + DX); j.dq = (own + n + 1) & 7; j.panel = NPA + own_panel(n % (2 + DX), (rank + j.dq) & 7); return j; }
    int c, k;
    if (n < J0) { c = 0; j.u = n / (NPA + 14); k = n % (NPA + 14) - NPA; if (k < 0) { j.panel = k + NPA; return j; } }
    else { const int m = n - J0, pr = m / (8 * (NSTR + 14)), r = m % (8 * (NSTR + 14));
        if (r < 8 * NSTR) { c = 2 * pr + 1; j.u = r / NSTR; k = r % NSTR; } else { c = 2 * pr + 2; j.u = (r - 8 * NSTR) / 14; k = (r - 8 * NSTR) % 14; } }
    j.panel = NPA + (k < NSTR ? c * NSTR + k : NSTR * ROUNDS + (c >> 1));
    return j;
}
__device__ __forceinline__ void tile_of(int kind, int id, int row, int c, int rank, int& pm, int& pn) {
    if (kind == 0) { pm = row; pn = c * NSTR + id; }
    else if (kind == 1) { pm = row + (c & 1); pn = NSTR * ROUNDS + (c >> 1); }
    else if (c < 4) { pm = (id >> 1) * 4 + c; pn = own_panel(id & 1, rank); }
    else { pm = (rank & 1) * 8 + id; pn = own_panel(2, rank); }
}

#define NSTAT (J0 / NCW)
constexpr unsigned DRAWS = JTOT - NSTAT * NCW + NCW;
#define STR2(x) #x
#define STR(x) STR2(x)
#define WAIT_WIN asm volatile("s_waitcnt vmcnt(" STR(WIN) ")" ::: "memory")

__device__ __forceinline__ unsigned wave_max_u32(unsigned mx) {
#pragma unroll
    for (int k = 32; k >= 1; k >>= 1) mx = max(mx, (unsigned)__shfl_xor((int)mx, k));
    return __builtin_amdgcn_readfirstlane(mx);
}
__device__ __forceinline__ int scale_exp(unsigned mx) { return ((14 - ((int)(mx >> 23) - 127)) >> 4) * 16; }
__device__ __forceinline__ unsigned lds_load(LAS unsigned* w) { return __builtin_amdgcn_readfirstlane(__hip_atomic_load(w, __ATOMIC_RELAXED, __HIP_MEMORY_SCOPE_WORKGROUP)); }
__device__ __forceinline__ void lds_store(LAS unsigned* w, unsigned v) { __hip_atomic_store(w, v, __ATOMIC_RELAXED, __HIP_MEMORY_SCOPE_WORKGROUP); }

constexpr int NCONV = NPAN * 8;
constexpr unsigned CDRAWS = NCONV + (256 - NCW);

__device__ __forceinline__ void block_loads(const Params& p, const Job& j, int wid, int lane, f32x4 (&v)[8]) {
    const bool a = j.panel < NPA;
    const __amdgpu_buffer_rsrc_t rs = make_rsrc(a ? (const char*)p.x : (const char*)p.w, a ? M * KS * 4 : KS * N * 4);
    const unsigned l = a ? (unsigned)((lane & 15) * KS + (lane >> 4) * 8) * 4u : (unsigned)((lane >> 4) * 8 * N + (lane & 15) * 4) * 4u;
    const unsigned b = a ? (unsigned)(j.panel * 256 * KS + j.u * 64) * 4u + (wid & 3) * (32 * KS * 4) + (wid >> 2) * (128 * KS * 4)
                         : (unsigned)(j.u * 64 * N + (j.panel - NPA) * 256) * 4u + (wid & 3) * 256 + (wid >> 2) * (32 * N * 4);
    const unsigned s0 = a ? 16 : N * 4, s1 = a ? 128 : 2 * N * 4, s2 = a ? 16 * KS * 4 : 4 * N * 4;
#pragma unroll
    for (int r = 0; r < 8; ++r) v[r] = __builtin_bit_cast(f32x4, __builtin_amdgcn_raw_buffer_load_b128(rs, (int)l, (int)(b + (r & 1) * s0 + ((r >> 1) & 1) * s1 + (r >> 2) * s2), SRC_AUX));
}
__device__ __forceinline__ unsigned piece_max(const f32x4 (&v)[8]) {
    unsigned mx = 0;
#pragma unroll
    for (int r = 0; r < 8; ++r) { const u32x4 u = __builtin_bit_cast(u32x4, v[r]) & 0x7fffffffu;
#pragma unroll
        for (int c = 0; c < 4; ++c) { const unsigned b = u[c]; mx = max(mx, b < 0x7f800000u ? b : 0u); } }
    return wave_max_u32(mx);
}
__device__ __forceinline__ unsigned block_max(const LAS unsigned* red, int lane) {
    unsigned mx = red[lane & 7];
#pragma unroll
    for (int k = 4; k >= 1; k >>= 1) mx = max(mx, (unsigned)__shfl_xor((int)mx, k));
    return __builtin_amdgcn_readfirstlane(mx);
}
template <class Put> __device__ __forceinline__ void block_pack(const Job& j, int wid, const f32x4 (&v)[8], int s, Put put) {
    if (j.panel < NPA) {
#pragma unroll
        for (int i = 0; i < 4; ++i) { const f32x4 a = v[2 * i], c = v[2 * i + 1]; u32x4 o;
            o.x = pk16(a.x, a.y, s); o.y = pk16(a.z, a.w, s); o.z = pk16(c.x, c.y, s); o.w = pk16(c.z, c.w, s); put(o, wid * 4096 + i * 1024); }
    } else {
#pragma unroll
        for (int i = 0; i < 4; ++i) { u32x4 o;
            o.x = pk16(v[0][i], v[1][i], s); o.y = pk16(v[2][i], v[3][i], s); o.z = pk16(v[4][i], v[5][i], s); o.w = pk16(v[6][i], v[7][i], s);
            put(o, (wid & 3) * 4096 + (wid >> 2) * 1024 + (i >> 1) * HTB + (i & 1) * 2048); }
    }
}

__device__ __forceinline__ void convert_phase(const Params& p, LAS unsigned char* lds, unsigned drawn, unsigned call) {
    const int tid = threadIdx.x, wid = __builtin_amdgcn_readfirstlane(tid >> 6), lane = tid & 63;
    const unsigned par = call & 1u, tbase = (call - 1u) * CDRAWS;
    gu32* tick = (gu32*)(p.ws[0] + OFF_TICKET + 128);
    gu64* flags = (gu64*)(p.ws[0] + OFF_FLAG) + par * (NPAN * 64);
    LAS unsigned* red = (LAS unsigned*)lds;
    const __amdgpu_buffer_rsrc_t rsD = make_rsrc(p.ws[0] + OFF_STAGE + (size_t)par * STAGE_PAR, STAGE_PAR);
    f32x4 v[8];
    auto job = [&](unsigned t) { Job j; if (t < (unsigned)J3) j = decode_job((int)t, -1, 0); else { j.u = (int)(t - J3) & 7; j.panel = NPA + NSH + ((int)(t - J3) >> 3); j.dq = -1; } return j; };
    if (tid == 0) red[8] = drawn;
    WAIT_L(0); BAR; FENCE;
    unsigned t = __builtin_amdgcn_readfirstlane(red[8]) - tbase;
    if (t < (unsigned)NCONV) {
        Job cur = job(t);
        if (tid == 0) draw_async(drawn, tick);
        block_loads(p, cur, wid, lane, v);
        for (;;) {
            unsigned mx = piece_max(v);
            asm volatile("" : "+v"(drawn));
            if (lane == 0) red[wid] = mx;
            if (tid == 0) red[9] = drawn;
            WAIT_L(0); BAR; FENCE;
            mx = block_max(red, lane);
            t = __builtin_amdgcn_readfirstlane(red[9]) - tbase;
            const int s = scale_exp(mx);
            const unsigned off = (unsigned)cur.panel * PANEL + (unsigned)cur.u * BLK;
            block_pack(cur, wid, v, s, [&](u32x4 o, int d) { __builtin_amdgcn_raw_buffer_store_b128(o, rsD, lane * 16, (int)(off + d), ST_AUX); });
            FENCE;
            const int fidx = cur.panel * 64 + cur.u;
            const bool more = t < (unsigned)NCONV;
            if (more) { cur = job(t); if (tid == 0) draw_async(drawn, tick); block_loads(p, cur, wid, lane, v); WAIT_V(8); } else WAIT_V(0);
            WAIT_L(0); BAR; FENCE;
            if (tid == 0) __hip_atomic_store(flags + fidx, ((u64)call << 32) | (unsigned)(mx ? s : ZSENT), RLX_SYS);
            if (!more) break;
        }
    }
    WAIT_L(0); BAR; FENCE;
}

__device__ __forceinline__ void comm_role(const Params& p, LAS unsigned char* lds, int cw, int own, unsigned call) {
    const int tid = threadIdx.x, wid = __builtin_amdgcn_readfirstlane(tid >> 6), lane = tid & 63;
    const unsigned par = call & 1u;
    gu32* tmo = (gu32*)(p.ws[0] + OFF_TMO);
    LAS unsigned* ctl = (LAS unsigned*)(lds + 4 * BLK);
    if (tid < 8) ctl[tid] = tid == 0 && own < 0 ? 1u : 0u;
    WAIT_L(0); BAR; FENCE;
    const int pos0 = wid == 0 ? 0 : 15 - wid;
    gu64* flagq = (gu64*)(p.ws[wid] + OFF_FLAG) + par * (NPAN * 64) + pos0;
    const __amdgpu_buffer_rsrc_t rsD = make_rsrc(p.ws[wid] + OFF_STAGE + (size_t)par * STAGE_PAR + pos0 * BLK, STAGE_PAR - pos0 * BLK);
    const __amdgpu_buffer_rsrc_t rsO = make_rsrc(p.out[wid], OUT_BYTES);
    int pidx = -1, psc = 0;
#define PUBLISH_PENDING do { if (pidx >= 0 && lane == 0) __hip_atomic_store(flagq + pidx, ((u64)call << 32) | (unsigned)psc, RLX_SYS); pidx = -1; } while (0)
    bool bad = false;
    if (wid > 0) {
        for (unsigned n = 0;; ++n) {
            if (lds_load(ctl) <= n) {
                WAIT_V(0); PUBLISH_PENDING;
                while (lds_load(ctl) <= n) __builtin_amdgcn_s_sleep(2);
            }
            FENCE;
            LAS unsigned* jw = ctl + 8 + 4 * (n & 3);
            const unsigned soff = lds_load(jw), fidx = lds_load(jw + 1), sc = lds_load(jw + 2), dq = lds_load(jw + 3);
            if (dq == 10u) break;
            if (dq >= 8u || dq == (unsigned)wid) {
                const LAS unsigned char* img = lds + (n & 3) * BLK + lane * 16;
#pragma unroll
                for (int g = 0; g < 4; ++g) {
                    u32x4 o[8];
#pragma unroll
                    for (int k = 0; k < 8; ++k) o[k] = *(const LAS u32x4*)(img + (g * 8 + k) * 1024);
                    if (g == 3) { WAIT_L(0); if (lane == 0) lds_store(ctl + wid, n + 1); }
#pragma unroll
                    for (int k = 0; k < 8; ++k) {
                        if (dq == 9u) __builtin_amdgcn_raw_buffer_store_b128(o[k], rsO, lane * 16, (int)(soff + (unsigned)(g * 8 + k) * (N * 4)), SLAB_AUX);
                        else __builtin_amdgcn_raw_buffer_store_b128(o[k], rsD, lane * 16, (int)(soff + (unsigned)(g * 8 + k) * 1024u), ST_AUX);
                        WAIT_WIN;
                        if (g == 0 && k == WIN - 1) PUBLISH_PENDING; }
                }
                if (dq != 9u) { pidx = (int)fidx; psc = (int)sc; }
            } else if (lane == 0) lds_store(ctl + wid, n + 1);
        }
        WAIT_V(0); PUBLISH_PENDING;
        if (own < 0 && lane == 0) __hip_atomic_store((gu32*)(p.ws[wid] + OFF_DONE + cw * 128) + ((8 - wid) & 7), call, RLX_SYS);
        return;
    }
    const unsigned tbase = (call - 1u) * DRAWS - (unsigned)(NSTAT * NCW);
    gu32* tick = (gu32*)(p.ws[0] + OFF_TICKET);
    unsigned n = own < 0 ? 1u : 0u;
    auto slot_free = [&]() {
        while (n >= 4u) { const unsigned c = __hip_atomic_load(ctl + 1 + (lane % 7), __ATOMIC_RELAXED, __HIP_MEMORY_SCOPE_WORKGROUP);
            if (__all(c + 3u >= n)) break; __builtin_amdgcn_s_sleep(2); }
        FENCE;
    };
    auto post = [&](unsigned soff, unsigned fidx, unsigned sc, unsigned dq) {
        if (lane == 0) { LAS unsigned* jw = ctl + 8 + 4 * (n & 3); jw[0] = soff; jw[1] = fidx; jw[2] = sc; jw[3] = dq; }
        WAIT_L(0);
        ++n; if (lane == 0) lds_store(ctl, n);
    };
    unsigned t1 = own < 0 ? NCW + cw : 0, t2 = own < 0 ? 2 * NCW + cw : 1, drawn = 0;
    const unsigned jend = own < 0 ? J3 : 8 * (2 + DX);
    if (own >= 0) {
        const u64 t0 = __builtin_amdgcn_s_memrealtime();
        while ((unsigned)__builtin_amdgcn_readfirstlane(__hip_atomic_load(tick, RLX_AGENT)) - tbase < (unsigned)OWN_GO && !give_up(tmo, call, t0)) __builtin_amdgcn_s_sleep(32);
    }
#pragma unroll 1
    while (t1 < jend) {
        const bool dyn = own < 0 && n + 2u >= (unsigned)NSTAT;
        if (dyn && lane == 0) draw_async(drawn, tick);
        const Job cur = decode_job((int)t1, own, p.rank);
        if (cur.dq != 0) {
            u64 fl; const u64 t0 = __builtin_amdgcn_s_memrealtime();
            for (;;) {
                fl = __hip_atomic_load(flagq + cur.panel * 64 + cur.u, RLX_SYS);
                if ((unsigned)__builtin_amdgcn_readfirstlane((int)(fl >> 32)) == call) break;
                if (bad || give_up(tmo, call, t0)) { bad = true; break; }
                __builtin_amdgcn_s_sleep(2);
            }
            slot_free();
            const unsigned off = (unsigned)cur.panel * PANEL + (unsigned)cur.u * BLK;
#pragma unroll
            for (int k = 0; k < 32; ++k) __builtin_amdgcn_raw_ptr_buffer_load_lds(rsD, (LAS void*)(lds + (n & 3) * BLK + k * 1024), 16, lane * 16, (int)(off + k * 1024), 0, 17);
            WAIT_V(0);
            if (!bad) post((unsigned)cur.panel * PANEL + (unsigned)cur.u * 7u * BLK, (unsigned)(cur.panel * 64 + cur.u * 7), (unsigned)fl, cur.dq < 0 ? 8u : (unsigned)cur.dq);
            STAMP(n);
        }
        WAIT_V(0); asm volatile("" : "+v"(drawn));
        t1 = t2; t2 = dyn ? __builtin_amdgcn_readfirstlane(drawn) - tbase : t2 + (own < 0 ? NCW : 1);
    }
    if (own >= 0) { slot_free(); post(0u, 0u, 0u, 10u); return; }
    STAMP(99);
    u64 ready = 0;
    while (t1 < (unsigned)JTOT) {
        const bool pending = t2 < (unsigned)JTOT;
        if (pending && lane == 0) draw_async(drawn, tick);
        const int tile = (int)(t1 - J3) >> 3;
        if (!((ready >> tile) & 1u)) {
            gu32* cnt = (gu32*)(p.ws[0] + OFF_OWN + lane * 128);
            const u64 t0 = __builtin_amdgcn_s_memrealtime();
            for (;;) {
                ready = __ballot(__hip_atomic_load(cnt, RLX_AGENT) >= 8u * call);
                if ((ready >> tile) & 1u) break;
                if (give_up(tmo, call, t0)) { bad = true; ready = ~0ull; break; }
                __builtin_amdgcn_s_sleep(8);
            }
        }
        int pm, pn; tile_of(2, tile & 7, 0, tile >> 3, p.rank, pm, pn);
        const unsigned off = (unsigned)((pm * 256 + (int)((t1 - J3) & 7) * 32) * (N * 4) + pn * 1024);
        slot_free();
        if (!bad) {
#pragma unroll
            for (int k = 0; k < 32; ++k) __builtin_amdgcn_raw_ptr_buffer_load_lds(rsO, (LAS void*)(lds + (n & 3) * BLK + k * 1024), 16, lane * 16, (int)(off + (unsigned)k * (N * 4)), 0, 18);
        } else {
            for (int k = 0; k < 32; ++k) *(LAS u32x4*)(lds + (n & 3) * BLK + k * 1024 + lane * 16) = (u32x4){0x7fc00000u, 0x7fc00000u, 0x7fc00000u, 0x7fc00000u};
        }
        WAIT_V(0); asm volatile("" : "+v"(drawn));
        post(off, 0u, 0u, 9u);
        STAMP(n);
        t1 = t2; t2 = pending ? __builtin_amdgcn_readfirstlane(drawn) - tbase : ~0u;
    }
    slot_free(); post(0u, 0u, 0u, 10u);
    STAMP(100);
    {
        const u64 t0 = __builtin_amdgcn_s_memrealtime();
        gu32* dn = (gu32*)(p.ws[0] + OFF_DONE + cw * 128) + (lane & 7);
        for (;;) {
            const unsigned val = __hip_atomic_load(dn, RLX_SYS);
            if (__all((lane & 7) == 0 || val >= call)) break;
            if (give_up(tmo, call, t0)) { bad = true; break; }
            __builtin_amdgcn_s_sleep(4);
        }
        __builtin_amdgcn_fence(__ATOMIC_ACQUIRE, "");
        STAMP(101);
        if (bad && lane == 0) ((float*)p.out[0])[(size_t)(NSH + 2 * p.rank) * 256 + cw] = __builtin_nanf("");
    }
}

__device__ __forceinline__ void gemm_role(const Params& p, LAS unsigned char* lds, int kind, int id, int row, unsigned call) {
    const int own_id = kind == 2 ? row * NOWN + id : -1, ntiles = kind < 2 ? ROUNDS : row ? 1 : 4;
    int pm, pn; tile_of(kind, id, row, kind == 2 ? row : 0, p.rank, pm, pn);
    const int tid = threadIdx.x, wid = __builtin_amdgcn_readfirstlane(tid >> 6), lane = tid & 63, wr = wid >> 2, wc = wid & 3;
    const unsigned par = call & 1u;
    const unsigned voff = (unsigned)wid * 2048u + (unsigned)lane * 16u;
    const unsigned ldsw = (unsigned)wid * 2048u;
    const int aoff = wr * 8192 + lane * 16, boff = wc * 4096 + lane * 16;
    gu32* tmo = (gu32*)(p.ws[0] + OFF_TMO);
    const __amdgpu_buffer_rsrc_t rsS = make_rsrc(p.ws[0] + OFF_STAGE + (size_t)par * STAGE_PAR, STAGE_PAR);
    const __amdgpu_buffer_rsrc_t rsC = make_rsrc(p.out[0], OUT_BYTES);
    gu64* flags = (gu64*)(p.ws[0] + OFF_FLAG) + par * (NPAN * 64);
    bool bad = false;
#define SA_(b, h) (((b) * 2 + (h)) * HTB)
#define SB_(b, h) ((4 + (b) * 2 + (h)) * HTB)
#define STAGE(bufoff, soff) do { \
    __builtin_amdgcn_raw_ptr_buffer_load_lds(rsS, (LAS void*)(lds + (bufoff) + ldsw), 16, (int)voff, (int)(soff), 0, GLDS_AUX); \
    __builtin_amdgcn_raw_ptr_buffer_load_lds(rsS, (LAS void*)(lds + (bufoff) + ldsw), 16, (int)voff, (int)(soff), 1024, GLDS_AUX); } while (0)
#define LDA(dst, b, h) do { _Pragma("unroll") for (int m = 0; m < 4; ++m) _Pragma("unroll") for (int k = 0; k < 2; ++k) dst[m][k] = *(const LAS f16x8*)(lds + SA_(b, h) + aoff + m * 2048 + k * 1024); } while (0)
#define LDB(dst, b, h) do { _Pragma("unroll") for (int n = 0; n < 2; ++n) _Pragma("unroll") for (int k = 0; k < 2; ++k) dst[n][k] = *(const LAS f16x8*)(lds + SB_(b, h) + boff + n * 2048 + k * 1024); } while (0)
#define POLL_ACC
#define ZERO4 ((f32x4){0.f, 0.f, 0.f, 0.f})
#define MF2(Z, i, R, m, n, Bt) do { if (Z) asm volatile("v_mfma_f32_16x16x32_f16 %0, %1, %2, 0\n\tv_mfma_f32_16x16x32_f16 %0, %3, %4, %0" : "+{" R "}"(acc[i]) : "v"(At[m][0]), "v"(Bt[n][0]), "v"(At[m][1]), "v"(Bt[n][1])); \
      else asm volatile("v_mfma_f32_16x16x32_f16 %0, %1, %2, %0\n\tv_mfma_f32_16x16x32_f16 %0, %3, %4, %0" : "+{" R "}"(acc[i]) : "v"(At[m][0]), "v"(Bt[n][0]), "v"(At[m][1]), "v"(Bt[n][1])); } while (0)
#define MMA2_0(Z) do { __builtin_amdgcn_s_setprio(1); \
      MF2(Z, 0, "v[128:131]", 0, 0, B0); MF2(Z, 1, "v[132:135]", 0, 1, B0); MF2(Z, 8, "v[160:163]", 0, 0, B1); MF2(Z, 9, "v[164:167]", 0, 1, B1); \
      MF2(Z, 2, "v[136:139]", 1, 0, B0); MF2(Z, 3, "v[140:143]", 1, 1, B0); MF2(Z, 10, "v[168:171]", 1, 0, B1); MF2(Z, 11, "v[172:175]", 1, 1, B1); \
      MF2(Z, 4, "v[144:147]", 2, 0, B0); MF2(Z, 5, "v[148:151]", 2, 1, B0); MF2(Z, 12, "v[176:179]", 2, 0, B1); MF2(Z, 13, "v[180:183]", 2, 1, B1); \
      MF2(Z, 6, "v[152:155]", 3, 0, B0); MF2(Z, 7, "v[156:159]", 3, 1, B0); MF2(Z, 14, "v[184:187]", 3, 0, B1); MF2(Z, 15, "v[188:191]", 3, 1, B1); \
      __builtin_amdgcn_s_setprio(0); } while (0)
#define MMA2_1(Z) do { __builtin_amdgcn_s_setprio(1); \
      MF2(Z, 16, "v[192:195]", 0, 0, B0); MF2(Z, 17, "v[196:199]", 0, 1, B0); MF2(Z, 24, "v[224:227]", 0, 0, B1); MF2(Z, 25, "v[228:231]", 0, 1, B1); \
      MF2(Z, 18, "v[200:203]", 1, 0, B0); MF2(Z, 19, "v[204:207]", 1, 1, B0); MF2(Z, 26, "v[232:235]", 1, 0, B1); MF2(Z, 27, "v[236:239]", 1, 1, B1); \
      MF2(Z, 20, "v[208:211]", 2, 0, B0); MF2(Z, 21, "v[212:215]", 2, 1, B0); MF2(Z, 28, "v[240:243]", 2, 0, B1); MF2(Z, 29, "v[244:247]", 2, 1, B1); \
      MF2(Z, 22, "v[216:219]", 3, 0, B0); MF2(Z, 23, "v[220:223]", 3, 1, B0); MF2(Z, 30, "v[248:251]", 3, 0, B1); MF2(Z, 31, "v[252:255]", 3, 1, B1); \
      __builtin_amdgcn_s_setprio(0); } while (0)
#define MMA2(Z, ai) MMA2_##ai(Z)
#define STORE_ROWS(cbase, ex, ai, m) do { _Pragma("unroll") for (int j = 0; j < 4; ++j) { \
      f32x4 o = {__builtin_ldexpf(acc[(ai) * 16 + (m) * 2][j], ex), __builtin_ldexpf(acc[(ai) * 16 + (m) * 2 + 1][j], ex), __builtin_ldexpf(acc[(ai) * 16 + 8 + (m) * 2][j], ex), __builtin_ldexpf(acc[(ai) * 16 + 8 + (m) * 2 + 1][j], ex)}; \
      if (own_id >= 0) __builtin_amdgcn_raw_buffer_store_b128(__builtin_bit_cast(u32x4, o), rsC, (int)coff, (int)((cbase) + (unsigned)((ai) * 128 + (m) * 16 + j) * (N * 4)), OWN_AUX); \
      else __builtin_amdgcn_raw_buffer_store_b128(__builtin_bit_cast(u32x4, o), rsC, (int)coff, (int)((cbase) + (unsigned)((ai) * 128 + (m) * 16 + j) * (N * 4)), C_AUX); } } while (0)
#define POLL_SLEEP 16
#define POLL(a, b, pos2, maskv, expv) do { const u64 _t0 = __builtin_amdgcn_s_memrealtime(); bool _ok = false; \
      for (int _n = 0;; ++_n) { \
        if (_n == 0 || (lane >> 4) == ((pos2) >> 3)) {     \
          const u64 _fa = __hip_atomic_load(flags + (a) * 64 + lane, RLX_SYS), _fb = __hip_atomic_load(flags + (NPA + (b)) * 64 + lane, RLX_SYS); \
          _ok = (unsigned)(_fa >> 32) == call && (unsigned)(_fb >> 32) == call; \
          expv = ((int)_fa == ZSENT || (int)_fb == ZSENT) ? ZSENT : (int)_fa + (int)_fb; } \
        maskv = __ballot(_ok); \
        if (((maskv >> (2 * (pos2))) & 3ull) == 3ull) break; \
        if (bad || ((_n & 63) == 63 && give_up(tmo, call, _t0))) { bad = true; maskv = ~0ull; expv = ZSENT; break; } \
        __builtin_amdgcn_s_sleep(POLL_SLEEP); } POLL_ACC; WAIT_L(0); } while (0)
#define SET_CMASK cmask = __ballot(ecur != esc && ecur != ZSENT)
#define RESCALE(pos_) do { int _d; asm volatile( \
        "s_bitcmp1_b64 %[cm], %[pos]\n\t" \
        "s_cbranch_scc0 1f\n\t" \
        "s_nop 15\n\t" \
        "v_readlane_b32 %[d], %[ec], %[pos]\n\t" \
        "s_nop 3\n\t" \
        "v_cmp_ne_u32 %[cm], %[d], %[ec]\n\t" \
        "v_cmp_ne_u32 vcc, 2.0, %[ec]\n\t" \
        "s_xor_b32 %[es], %[es], %[d]\n\t" \
        "s_xor_b32 %[d], %[es], %[d]\n\t" \
        "s_xor_b32 %[es], %[es], %[d]\n\t" \
        "s_sub_i32 %[d], %[es], %[d]\n\t" \
        "s_and_b64 %[cm], %[cm], vcc\n\t" \
        ".set gr_, 128\n\t" \
        ".rept 128\n\t" \
        "v_ldexp_f32 v[gr_], v[gr_], %[d]\n\t" \
        ".set gr_, gr_ + 1\n\t" \
        ".endr\n\t" \
        "s_nop 3\n\t" \
        "1:\n\t" \
        : [cm] "+s"(cmask), [es] "+s"(esc), [d] "=&s"(_d), \
          "+{v[128:131]}"(acc[0]), "+{v[132:135]}"(acc[1]), "+{v[136:139]}"(acc[2]), "+{v[140:143]}"(acc[3]), "+{v[144:147]}"(acc[4]), "+{v[148:151]}"(acc[5]), "+{v[152:155]}"(acc[6]), "+{v[156:159]}"(acc[7]), \
          "+{v[160:163]}"(acc[8]), "+{v[164:167]}"(acc[9]), "+{v[168:171]}"(acc[10]), "+{v[172:175]}"(acc[11]), "+{v[176:179]}"(acc[12]), "+{v[180:183]}"(acc[13]), "+{v[184:187]}"(acc[14]), "+{v[188:191]}"(acc[15]), \
          "+{v[192:195]}"(acc[16]), "+{v[196:199]}"(acc[17]), "+{v[200:203]}"(acc[18]), "+{v[204:207]}"(acc[19]), "+{v[208:211]}"(acc[20]), "+{v[212:215]}"(acc[21]), "+{v[216:219]}"(acc[22]), "+{v[220:223]}"(acc[23]), \
          "+{v[224:227]}"(acc[24]), "+{v[228:231]}"(acc[25]), "+{v[232:235]}"(acc[26]), "+{v[236:239]}"(acc[27]), "+{v[240:243]}"(acc[28]), "+{v[244:247]}"(acc[29]), "+{v[248:251]}"(acc[30]), "+{v[252:255]}"(acc[31]) \
        : [pos] "s"(pos_), [ec] "v"(ecur) : "vcc", "scc"); } while (0)
#define SET_NRDY do { nrdy = ~rmask ? __builtin_ctzll(~rmask) >> 1 : 32; evt = min(min(30, nrdy - 2), pubit); } while (0)
#define FETCH_FLAGS(a, b) do { const gu64* _pa = flags + (a) * 64 + lane; const gu64* _pb = flags + (NPA + (b)) * 64 + lane; \
      asm volatile("global_load_dwordx2 %0, %2, off sc0 sc1\n\tglobal_load_dwordx2 %1, %3, off sc0 sc1" : "=&v"(nfa), "=&v"(nfb) : "v"(_pa), "v"(_pb) : "memory"); } while (0)
#define BODY(FIRST, LAST, EXTRA) do { \
      const int pp = it, ppn = (pp + 1) & 31; \
      const unsigned a1 = cA + (unsigned)(pp * 2 + 1) * BLK; \
      const unsigned a2 = ((LAST) ? nA : cA) + (unsigned)(ppn * 2) * BLK, b2 = ((LAST) ? nB : cB) + (unsigned)(ppn * 2) * BLK; \
      const unsigned a3 = a2 + BLK, b3 = b2 + BLK; \
      LDB(B0, 0, 0); LDB(B1, 0, 1); SCHED; LDA(At, 0, 0); STAGE(SA_(1, 1), a1 + HTB); FENCE; \
      if (FIRST) { const int e = __builtin_amdgcn_readlane(ecur, 2 * pp); esc = e == ZSENT ? 0 : e; SET_CMASK; } else RESCALE(2 * pp); \
      if ((FIRST) && ui > 0) { STORE_ROWS(cPrev, eprev, 0, 3); STORE_ROWS(cPrev, eprev, 1, 0); STORE_ROWS(cPrev, eprev, 1, 1); WAIT_V(32); } else WAIT_V(8); \
      WAIT_L(0); BAR; MMA2(FIRST, 0); BAR; SCHED; \
      LDA(At, 0, 1); FENCE; \
      if (LAST) { if (has_next) {     \
          asm volatile("" : "+v"(nfa), "+v"(nfb)); \
          rmask = __ballot((unsigned)(nfa >> 32) == call && (unsigned)(nfb >> 32) == call); \
          enext = ((int)nfa == ZSENT || (int)nfb == ZSENT) ? ZSENT : (int)nfa + (int)nfb; \
          if (((rmask >> (2 * ppn)) & 3ull) != 3ull) POLL(npm, npn, ppn, rmask, enext); SET_NRDY; } } \
      else if (__builtin_expect(ppn >= nrdy, 0)) { \
          if (!(FIRST)) {             \
            asm volatile("" : "+v"(nfa), "+v"(nfb)); \
            rmask = __ballot((unsigned)(nfa >> 32) == call && (unsigned)(nfb >> 32) == call); \
            ecur = ((int)nfa == ZSENT || (int)nfb == ZSENT) ? ZSENT : (int)nfa + (int)nfb; } \
          if (((rmask >> (2 * ppn)) & 3ull) != 3ull) POLL(pm, pn, ppn, rmask, ecur); \
          SET_CMASK; SET_NRDY; } \
      STAGE(SB_(0, 0), b2); STAGE(SB_(0, 1), b2 + HTB); STAGE(SA_(0, 0), a2); \
      if ((FIRST) && ui > 0) { FENCE; STORE_ROWS(cPrev, eprev, 1, 2); STORE_ROWS(cPrev, eprev, 1, 3); WAIT_V(40); } else WAIT_V(8); \
      WAIT_L(0); BAR; MMA2(FIRST, 1); BAR; SCHED; \
      LDB(B0, 1, 0); LDB(B1, 1, 1); SCHED; LDA(At, 1, 0); STAGE(SA_(0, 1), a2 + HTB); FENCE; \
      RESCALE(2 * pp + 1); \
      if ((FIRST) && ui > 0) WAIT_V(28); else WAIT_V(8); \
      WAIT_L(0); BAR; MMA2(0, 0); BAR; SCHED; \
      LDA(At, 1, 1); FENCE; EXTRA; STAGE(SB_(1, 0), b3); STAGE(SB_(1, 1), b3 + HTB); STAGE(SA_(1, 0), a3); \
      if (LAST) { FENCE; STORE_ROWS(cC, -esc, 0, 0); STORE_ROWS(cC, -esc, 0, 1); STORE_ROWS(cC, -esc, 0, 2); WAIT_V(20); } else if ((FIRST) && ui > 0) WAIT_V(16); else WAIT_V(8); \
      WAIT_L(0); BAR; MMA2(0, 1); BAR; SCHED; } while (0)
    f32x4 acc[32];
#pragma unroll
    for (int i = 0; i < 32; ++i) acc[i] = ZERO4;
    f16x8 At[4][2], B0[2][2], B1[2][2];
    unsigned cA = (unsigned)pm * PANEL, cB = (unsigned)(NPA + pn) * PANEL;
    const unsigned coff = (unsigned)((wr * 64 + (lane >> 4) * 4) * (N * 4) + (wc * 64 + (lane & 15) * 4) * 4);
    unsigned cC = (unsigned)pm * 256u * (N * 4) + (unsigned)pn * 1024u, cPrev = cC;
    u64 rmask, cmask = 0, nfa = 0, nfb = 0; int ecur, enext = 0, esc = 0, eprev = 0;
    int nrdy, evt, pubit = 64;
    POLL(pm, pn, 0, rmask, ecur); SET_NRDY;
    STAGE(SB_(0, 0), cB); STAGE(SB_(0, 1), cB + HTB); STAGE(SA_(0, 0), cA); STAGE(SA_(0, 1), cA + HTB);
    if (wr == 1) BAR;
    WAIT_V(2); BAR;
    STAGE(SB_(1, 0), cB + BLK); STAGE(SA_(1, 0), cA + BLK); STAGE(SB_(1, 1), cB + BLK + HTB);
    WAIT_V(6); BAR;
    STAMP(sb + 1);
#pragma unroll 1
    for (int ui = 0; ui < ntiles; ++ui) {
        const bool has_next = ui + 1 < ntiles;
        int npm = pm, npn = pn;
        if (has_next) tile_of(kind, id, row, ui + 1, p.rank, npm, npn);
        const unsigned nA = (unsigned)npm * PANEL, nB = (unsigned)(NPA + npn) * PANEL;
        int it = 0;
        BODY(1, 0, { if (nrdy <= 2) FETCH_FLAGS(pm, pn); });
        STAMPV(2 * sb + 60 + ui, __builtin_amdgcn_s_memtime() - cya);
#pragma unroll 1
        for (it = 1; it < 31; ++it) {
            BODY(0, 0, {
                if (__builtin_expect(it >= evt, 0)) {
                    if (it == 30) { FETCH_FLAGS(npm, npn); pubit = own_id >= 0 ? 1 : 64; }
                    else if (it + 2 >= nrdy) FETCH_FLAGS(pm, pn);
                    if (it == pubit) {
                        if (lane == 0) __hip_atomic_fetch_add((gu32*)(p.ws[0] + OFF_OWN + ((ui - 1) * NOWN + own_id) * 128), 1u, RLX_AGENT);
                        pubit = 64; evt = min(30, nrdy - 2);
                    }
                } });
        }
        STAMPV(2 * sb + 70 + ui, __builtin_amdgcn_s_memtime() - cya);
        BODY(0, 1, {});
        STAMPV(2 * sb + 80 + ui, __builtin_amdgcn_s_memtime() - cya);
        STAMP(sb + 2 + ui); STAMPV(2 * sb + 20 + ui, pollt); STAMPV(2 * sb + 40 + ui, __builtin_amdgcn_s_memtime() - cyc0);
        cPrev = cC; eprev = -esc; ecur = enext;
        cA = nA; cB = nB; pm = npm; pn = npn;
        cC = (unsigned)pm * 256u * (N * 4) + (unsigned)pn * 1024u;
    }
    asm volatile("s_nop 15" ::: "memory"); SCHED;
    STORE_ROWS(cPrev, eprev, 0, 3); STORE_ROWS(cPrev, eprev, 1, 0); STORE_ROWS(cPrev, eprev, 1, 1); STORE_ROWS(cPrev, eprev, 1, 2); STORE_ROWS(cPrev, eprev, 1, 3);
    WAIT_V(0);
    STAMP(sb + 12);
    if (bad) { *(float*)(p.out[0] + cPrev + coff) = __builtin_nanf(""); WAIT_V(0); }
    if (own_id >= 0) {
        if (lane == 0) __hip_atomic_fetch_add((gu32*)(p.ws[0] + OFF_OWN + ((ntiles - 1) * NOWN + own_id) * 128), 1u, RLX_AGENT);
    }
    if (wr == 0) BAR;
    BAR;
}

__global__ __launch_bounds__(512, 2) void fused_kernel(Params p) {
    extern __shared__ __attribute__((aligned(16))) unsigned char shm[];
    LAS unsigned char* lds = (LAS unsigned char*)shm;
    STAMP(0);
    gu32* cc = (gu32*)(p.ws[0] + OFF_CALL + blockIdx.x * 64);
    const int x = blockIdx.x & 7, o = blockIdx.x >> 3;
    unsigned call;
    if ((int)blockIdx.x < NCW) {
        const int wid = __builtin_amdgcn_readfirstlane(threadIdx.x >> 6), lane = threadIdx.x & 63;
        LAS unsigned* ctl = (LAS unsigned*)(lds + 4 * BLK);
        const Job j = decode_job(blockIdx.x, -1, 0);
        f32x4 v[8];
        block_loads(p, j, wid, lane, v);
        call = (unsigned)__builtin_amdgcn_readfirstlane(__hip_atomic_load(cc, RLX_AGENT)) + 1u;
        const unsigned pmx = piece_max(v);
        if (lane == 0) ctl[32 + wid] = pmx;
        WAIT_V(0); WAIT_L(0); BAR; FENCE;
        const unsigned mx = block_max(ctl + 32, lane);
        const int s = scale_exp(mx);
        block_pack(j, wid, v, s, [&](u32x4 o, int d) { *(LAS u32x4*)(lds + d + lane * 16) = o; });
        if (threadIdx.x == 0) { ctl[8] = (unsigned)j.panel * PANEL + (unsigned)j.u * 7u * BLK; ctl[9] = (unsigned)(j.panel * 64 + j.u * 7); ctl[10] = (unsigned)(mx ? s : ZSENT); ctl[11] = 8u; }
        if (threadIdx.x == 0) __hip_atomic_store(cc, call, RLX_AGENT);
        STAMP(1);
    } else {
        unsigned drawn = 0;
        if (threadIdx.x == 0) draw_async(drawn, (gu32*)(p.ws[0] + OFF_TICKET + 128));
        call = (unsigned)__builtin_amdgcn_readfirstlane(__hip_atomic_load(cc, RLX_AGENT)) + 1u;
        WAIT_V(0); asm volatile("" : "+v"(drawn)); BAR;
        if (threadIdx.x == 0) __hip_atomic_store(cc, call, RLX_AGENT);
        convert_phase(p, lds, drawn, call);
    }
    STAMP(19);
    call = (unsigned)__builtin_amdgcn_readfirstlane((int)call);
    int kind = 0, id = (x & 1) * 7 + ((o - 4) >> 2), row = (x >> 1) * 4 + (o & 3), snd = -1;
    if (o < 3) kind = -1;
    else if (o == 3) { kind = 2; id = x; row = 0; snd = DX ? -1 : x; }
    else if (id >= NSTR) { if (o & 1) { kind = 2; id = (x & 6) + ((o >> 1) & 1); row = 4; snd = id; } else kind = 1; }
    if ((int)blockIdx.x < NCW || snd >= 0) {
        comm_role(p, lds, blockIdx.x, snd, call);
        WAIT_L(0); BAR; FENCE;
    }
    if (kind < 0) return;
    gemm_role(p, lds, kind, id, row, call);
}

}

extern "C" void kernel_launch(int rank, int nranks, ncclComm_t comm, hipStream_t stream,
                              void* const* d_in, const int* in_sizes, int n_in,
                              void* d_out, int out_size, const int* in_dtypes, int out_dtype,
                              void* const* d_out_all, void* const* d_ws_all, size_t ws_bytes) {
    if (nranks != NR || (unsigned)rank >= (unsigned)NR || ws_bytes < WS_NEED || n_in < 2 ||
        in_sizes[0] != M * KS || in_sizes[1] != KS * N || out_size != M * N) {
        fprintf(stderr, "kernel_launch: unexpected problem shape\n");
        return;
    }
    Params p;
    p.x = (const float*)d_in[0]; p.w = (const float*)d_in[1]; p.rank = rank;
    for (int q = 0; q < NR; ++q) { p.ws[q] = (char*)d_ws_all[(rank + q) % NR]; p.out[q] = (char*)d_out_all[(rank + q) % NR]; }
    p.out[0] = (char*)d_out;
    HIP_CHECK(hipFuncSetAttribute((const void*)fused_kernel, hipFuncAttributeMaxDynamicSharedMemorySize, LDS_BYTES));
    hipLaunchKernelGGL(fused_kernel, dim3(256), dim3(512), LDS_BYTES, stream, p);
    HIP_CHECK(hipGetLastError());
}
```

```cpp
#include <hip/hip_runtime.h>
#include <rccl/rccl.h>
#include <cstdio>

#define HIP_CHECK(expr) do { hipError_t _e = (expr); if (_e != hipSuccess) { \
    fprintf(stderr, "HIP error %s at %s:%d\n", hipGetErrorString(_e), __FILE__, __LINE__); \
    return; } } while (0)

namespace {

#define LAS __attribute__((address_space(3)))
#define GAS __attribute__((address_space(1)))
typedef _Float16 f16x8 __attribute__((ext_vector_type(8)));
typedef _Float16 f16x2 __attribute__((ext_vector_type(2)));
typedef float f32x4 __attribute__((ext_vector_type(4)));
typedef unsigned u32x4 __attribute__((ext_vector_type(4)));
typedef unsigned long long u64;
typedef GAS unsigned gu32;
typedef GAS u64 gu64;

constexpr int M = 4096, N = 32768, KS = 512, NR = 8;
constexpr int NPA = 16, NPB = 128, NPAN = NPA + NPB;
constexpr unsigned BLK = 32768, PANEL = 64 * BLK, HTB = 16384;
constexpr int DX = 1;
constexpr int ROUNDS = 8, NSTR = 14 - DX, NSH = NSTR * ROUNDS + 4 * DX;
constexpr int NOWN = 8, OWN_TILES = 4 + DX;
constexpr int NCW = 24;
#define WIN 2
constexpr int LDS_BYTES = 8 * HTB + 256;
constexpr unsigned OFF_CALL = 0;
constexpr unsigned OFF_TMO = 16384;
constexpr unsigned OFF_OWN = 20480;
constexpr unsigned OFF_TICKET = 28672;
constexpr unsigned OFF_DONE = 32768;
constexpr unsigned OFF_FLAG = 65536;
constexpr unsigned OFF_STAGE = 1u << 20;
constexpr unsigned STAGE_PAR = NPAN * PANEL;
constexpr size_t WS_NEED = (size_t)OFF_STAGE + 2ull * STAGE_PAR;
constexpr unsigned OUT_BYTES = (unsigned)((size_t)M * N * 4);
constexpr u64 DEADLINE = 100000000ull;
constexpr int ZSENT = 0x40000000;
constexpr int J0 = 8 * (NPA + 14);
constexpr int J3 = 8 * (NPA + NSH);
constexpr int JTOT = J3 + NOWN * OWN_TILES * 8;
constexpr int OWN_GO = J0 + 8 * NCW;
constexpr int GLDS_AUX = 17, ST_AUX = 17, SRC_AUX = 2, SLAB_AUX = 19;

struct Params {
    const float* x; const float* w;
    char* ws[NR];
    char* out[NR];
    int rank;
};

#define RLX_SYS __ATOMIC_RELAXED, __HIP_MEMORY_SCOPE_SYSTEM
#define RLX_AGENT __ATOMIC_RELAXED, __HIP_MEMORY_SCOPE_AGENT
#define WAIT_V(n) asm volatile("s_waitcnt vmcnt(" #n ")" ::: "memory")
#define WAIT_L(n) do { static_assert((n) == 0, ""); __builtin_amdgcn_s_waitcnt(0xC07F); asm volatile("" ::: "memory"); } while (0)
#define BAR __builtin_amdgcn_s_barrier()
#define SCHED __builtin_amdgcn_sched_barrier(0)
#define FENCE do { asm volatile("" ::: "memory"); SCHED; } while (0)

__device__ __forceinline__ unsigned pk16(float a, float b, int s) {
    f16x2 v; v.x = (_Float16)__builtin_ldexpf(a, s); v.y = (_Float16)__builtin_ldexpf(b, s);
    return __builtin_bit_cast(unsigned, v);
}
__device__ __forceinline__ __amdgpu_buffer_rsrc_t make_rsrc(const char* base, unsigned bytes) {
    const u64 a = (u64)base;
    const u64 u = ((u64)(unsigned)__builtin_amdgcn_readfirstlane((int)(a >> 32)) << 32) | (unsigned)__builtin_amdgcn_readfirstlane((int)a);
    return __builtin_amdgcn_make_buffer_rsrc((void*)u, 0, __builtin_amdgcn_readfirstlane((int)bytes), 0x00020000);
}
__device__ __forceinline__ bool give_up(gu32* tmo, unsigned call, u64 t0) {
    const bool bad = (unsigned)__builtin_amdgcn_readfirstlane(__hip_atomic_load(tmo, RLX_AGENT)) == call ||
                     __builtin_amdgcn_s_memrealtime() - t0 > DEADLINE;
    if (bad) __hip_atomic_store(tmo, call, RLX_AGENT);
    return bad;
}

__device__ __forceinline__ void draw_async(unsigned& t, gu32* ctr) {
    asm volatile("global_atomic_add %0, %1, %2, off sc0" : "=v"(t) : "v"(ctr), "v"(1u) : "memory");
}

__device__ __forceinline__ int own_panel(int h, int rank) { return h < 2 ? NSH + 2 * rank + h : NSH + 16 + (rank >> 1); }
struct Job { int panel, u, dq; };
__device__ __forceinline__ Job decode_job(int n, int own, int rank) {
    Job j; j.dq = -1;
    if (own >= 0) { j.u = n / (2 + DX); j.dq = (own + n + 1) & 7; j.panel = NPA + own_panel(n % (2 + DX), (rank + j.dq) & 7); return j; }
    int c, k;
    if (n < J0) { c = 0; j.u = n / (NPA + 14); k = n % (NPA + 14) - NPA; if (k < 0) { j.panel = k + NPA; return j; } }
    else { const int m = n - J0, pr = m / (8 * (NSTR + 14)), r = m % (8 * (NSTR + 14));
        if (r < 8 * NSTR) { c = 2 * pr + 1; j.u = r / NSTR; k = r % NSTR; } else { c = 2 * pr + 2; j.u = (r - 8 * NSTR) / 14; k = (r - 8 * NSTR) % 14; } }
    j.panel = NPA + (k < NSTR ? c * NSTR + k : NSTR * ROUNDS + (c >> 1));
    return j;
}
__device__ __forceinline__ void tile_of(int kind, int id, int row, int c, int rank, int& pm, int& pn) {
    if (kind == 0) { pm = row; pn = c * NSTR + id; }
    else if (kind == 1) { pm = row + (c & 1); pn = NSTR * ROUNDS + (c >> 1); }
    else if (c < 4) { pm = (id >> 1) * 4 + c; pn = own_panel(id & 1, rank); }
    else { pm = (rank & 1) * 8 + id; pn = own_panel(2, rank); }
}

constexpr int NSTAT = J0 / NCW;
constexpr unsigned DRAWS = JTOT - NSTAT * NCW + NCW;
#define STR2(x) #x
#define STR(x) STR2(x)
#define WAIT_WIN asm volatile("s_waitcnt vmcnt(" STR(WIN) ")" ::: "memory")

__device__ __forceinline__ unsigned wave_max_u32(unsigned mx) {
#pragma unroll
    for (int k = 32; k >= 1; k >>= 1) mx = max(mx, (unsigned)__shfl_xor((int)mx, k));
    return __builtin_amdgcn_readfirstlane(mx);
}
__device__ __forceinline__ int scale_exp(unsigned mx) { return ((14 - ((int)(mx >> 23) - 127)) >> 4) * 16; }
__device__ __forceinline__ unsigned lds_load(LAS unsigned* w) { return __builtin_amdgcn_readfirstlane(__hip_atomic_load(w, __ATOMIC_RELAXED, __HIP_MEMORY_SCOPE_WORKGROUP)); }
__device__ __forceinline__ void lds_store(LAS unsigned* w, unsigned v) { __hip_atomic_store(w, v, __ATOMIC_RELAXED, __HIP_MEMORY_SCOPE_WORKGROUP); }

constexpr int NCONV = NPAN * 8;
constexpr unsigned CDRAWS = NCONV + (256 - NCW);

__device__ __forceinline__ void block_loads(const Params& p, const Job& j, int wid, int lane, f32x4 (&v)[8]) {
    const bool a = j.panel < NPA;
    const __amdgpu_buffer_rsrc_t rs = make_rsrc(a ? (const char*)p.x : (const char*)p.w, a ? M * KS * 4 : KS * N * 4);
    const unsigned l = a ? (unsigned)((lane & 15) * KS + (lane >> 4) * 8) * 4u : (unsigned)((lane >> 4) * 8 * N + (lane & 15) * 4) * 4u;
    const unsigned b = a ? (unsigned)(j.panel * 256 * KS + j.u * 64) * 4u + (wid & 3) * (32 * KS * 4) + (wid >> 2) * (128 * KS * 4)
                         : (unsigned)(j.u * 64 * N + (j.panel - NPA) * 256) * 4u + (wid & 3) * 256 + (wid >> 2) * (32 * N * 4);
    const unsigned s0 = a ? 16 : N * 4, s1 = a ? 128 : 2 * N * 4, s2 = a ? 16 * KS * 4 : 4 * N * 4;
#pragma unroll
    for (int r = 0; r < 8; ++r) v[r] = __builtin_bit_cast(f32x4, __builtin_amdgcn_raw_buffer_load_b128(rs, (int)l, (int)(b + (r & 1) * s0 + ((r >> 1) & 1) * s1 + (r >> 2) * s2), SRC_AUX));
}
__device__ __forceinline__ unsigned piece_max(const f32x4 (&v)[8]) {
    unsigned mx = 0;
#pragma unroll
    for (int r = 0; r < 8; ++r) { const u32x4 u = __builtin_bit_cast(u32x4, v[r]) & 0x7fffffffu;
#pragma unroll
        for (int c = 0; c < 4; ++c) { const unsigned b = u[c]; mx = max(mx, b < 0x7f800000u ? b : 0u); } }
    return wave_max_u32(mx);
}
__device__ __forceinline__ unsigned block_max(const LAS unsigned* red, int lane) {
    unsigned mx = red[lane & 7];
#pragma unroll
    for (int k = 4; k >= 1; k >>= 1) mx = max(mx, (unsigned)__shfl_xor((int)mx, k));
    return __builtin_amdgcn_readfirstlane(mx);
}
template <class Put> __device__ __forceinline__ void block_pack(const Job& j, int wid, const f32x4 (&v)[8], int s, Put put) {
    if (j.panel < NPA) {
#pragma unroll
        for (int i = 0; i < 4; ++i) { const f32x4 a = v[2 * i], c = v[2 * i + 1]; u32x4 o;
            o.x = pk16(a.x, a.y, s); o.y = pk16(a.z, a.w, s); o.z = pk16(c.x, c.y, s); o.w = pk16(c.z, c.w, s); put(o, wid * 4096 + i * 1024); }
    } else {
#pragma unroll
        for (int i = 0; i < 4; ++i) { u32x4 o;
            o.x = pk16(v[0][i], v[1][i], s); o.y = pk16(v[2][i], v[3][i], s); o.z = pk16(v[4][i], v[5][i], s); o.w = pk16(v[6][i], v[7][i], s);
            put(o, (wid & 3) * 4096 + (wid >> 2) * 1024 + (i >> 1) * HTB + (i & 1) * 2048); }
    }
}

__device__ __forceinline__ void convert_phase(const Params& p, LAS unsigned char* lds, unsigned drawn, unsigned call) {
    const int tid = threadIdx.x, wid = __builtin_amdgcn_readfirstlane(tid >> 6), lane = tid & 63;
    const unsigned par = call & 1u, tbase = (call - 1u) * CDRAWS;
    gu32* tick = (gu32*)(p.ws[0] + OFF_TICKET + 128);
    gu64* flags = (gu64*)(p.ws[0] + OFF_FLAG) + par * (NPAN * 64);
    LAS unsigned* red = (LAS unsigned*)lds;
    const __amdgpu_buffer_rsrc_t rsD = make_rsrc(p.ws[0] + OFF_STAGE + (size_t)par * STAGE_PAR, STAGE_PAR);
    f32x4 v[8];
    auto job = [&](unsigned t) { Job j; if (t < (unsigned)J3) j = decode_job((int)t, -1, 0); else { j.u = (int)(t - J3) & 7; j.panel = NPA + NSH + ((int)(t - J3) >> 3); j.dq = -1; } return j; };
    if (tid == 0) red[8] = drawn;
    WAIT_L(0); BAR; FENCE;
    unsigned t = __builtin_amdgcn_readfirstlane(red[8]) - tbase;
    if (t < (unsigned)NCONV) {
        Job cur = job(t);
        if (tid == 0) draw_async(drawn, tick);
        block_loads(p, cur, wid, lane, v);
        for (;;) {
            unsigned mx = piece_max(v);
            asm volatile("" : "+v"(drawn), "+s"(mx));
            if (lane == 0) red[wid] = mx;
            if (tid == 0) red[9] = drawn;
            WAIT_L(0); BAR; FENCE;
            mx = block_max(red, lane);
            t = __builtin_amdgcn_readfirstlane(red[9]) - tbase;
            const int s = scale_exp(mx);
            const unsigned off = (unsigned)cur.panel * PANEL + (unsigned)cur.u * BLK;
            block_pack(cur, wid, v, s, [&](u32x4 o, int d) { __builtin_amdgcn_raw_buffer_store_b128(o, rsD, lane * 16, (int)(off + d), ST_AUX); });
            FENCE;
            const int fidx = cur.panel * 64 + cur.u;
            const bool more = t < (unsigned)NCONV;
            WAIT_V(0);
            if (more) { cur = job(t); if (tid == 0) draw_async(drawn, tick); block_loads(p, cur, wid, lane, v); }
            WAIT_L(0); BAR; FENCE;
            if (tid == 0) __hip_atomic_store(flags + fidx, ((u64)call << 32) | (unsigned)(mx ? s : ZSENT), RLX_SYS);
            if (!more) break;
        }
    }
    WAIT_L(0); BAR; FENCE;
}

__device__ __forceinline__ void comm_role(const Params& p, LAS unsigned char* lds, int cw, int own, unsigned call) {
    const int tid = threadIdx.x, wid = __builtin_amdgcn_readfirstlane(tid >> 6), lane = tid & 63;
    const unsigned par = call & 1u;
    gu32* tmo = (gu32*)(p.ws[0] + OFF_TMO);
    LAS unsigned* ctl = (LAS unsigned*)(lds + 4 * BLK);
    if (tid < 8) ctl[tid] = tid == 0 && own < 0 ? 1u : 0u;
    WAIT_L(0); BAR; FENCE;
    const int pos0 = wid == 0 ? 0 : 15 - wid;
    gu64* flagq = (gu64*)(p.ws[wid] + OFF_FLAG) + par * (NPAN * 64) + pos0;
    const __amdgpu_buffer_rsrc_t rsD = make_rsrc(p.ws[wid] + OFF_STAGE + (size_t)par * STAGE_PAR + pos0 * BLK, STAGE_PAR - pos0 * BLK);
    const __amdgpu_buffer_rsrc_t rsO = make_rsrc(p.out[wid], OUT_BYTES);
    int pidx = -1, psc = 0;
#define PUBLISH_PENDING do { if (pidx >= 0 && lane == 0) __hip_atomic_store(flagq + pidx, ((u64)call << 32) | (unsigned)psc, RLX_SYS); pidx = -1; } while (0)
    bool bad = false;
    if (wid > 0) {
        for (unsigned n = 0;; ++n) {
            if (lds_load(ctl) <= n) {
                WAIT_V(0); PUBLISH_PENDING;
                while (lds_load(ctl) <= n) __builtin_amdgcn_s_sleep(2);
            }
            FENCE;
            LAS unsigned* jw = ctl + 8 + 4 * (n & 3);
            const unsigned soff = lds_load(jw), fidx = lds_load(jw + 1), sc = lds_load(jw + 2), dq = lds_load(jw + 3);
            if (dq == 10u) break;
            if (dq >= 8u || dq == (unsigned)wid) {
                const LAS unsigned char* img = lds + (n & 3) * BLK + lane * 16;
#pragma unroll
                for (int g = 0; g < 4; ++g) {
                    u32x4 o[8];
#pragma unroll
                    for (int k = 0; k < 8; ++k) o[k] = *(const LAS u32x4*)(img + (g * 8 + k) * 1024);
                    if (g == 3) { WAIT_L(0); if (lane == 0) lds_store(ctl + wid, n + 1); }
#pragma unroll
                    for (int k = 0; k < 8; ++k) {
                        if (dq == 9u) __builtin_amdgcn_raw_buffer_store_b128(o[k], rsO, lane * 16, (int)(soff + (unsigned)(g * 8 + k) * (N * 4)), SLAB_AUX);
                        else __builtin_amdgcn_raw_buffer_store_b128(o[k], rsD, lane * 16, (int)(soff + (unsigned)(g * 8 + k) * 1024u), ST_AUX);
                        WAIT_WIN;
                        if (g == 0 && k == WIN - 1) PUBLISH_PENDING; }
                }
                if (dq != 9u) { pidx = (int)fidx; psc = (int)sc; }
            } else if (lane == 0) lds_store(ctl + wid, n + 1);
        }
        WAIT_V(0); PUBLISH_PENDING;
        if (own < 0 && lane == 0) __hip_atomic_store((gu32*)(p.ws[wid] + OFF_DONE + cw * 128) + ((8 - wid) & 7), call, RLX_SYS);
        return;
    }
    const unsigned tbase = (call - 1u) * DRAWS - (unsigned)(NSTAT * NCW);
    gu32* tick = (gu32*)(p.ws[0] + OFF_TICKET);
    unsigned n = own < 0 ? 1u : 0u;
    auto slot_free = [&]() {
        while (n >= 4u) { const unsigned c = __hip_atomic_load(ctl + 1 + (lane % 7), __ATOMIC_RELAXED, __HIP_MEMORY_SCOPE_WORKGROUP);
            if (__all(c + 3u >= n)) break; __builtin_amdgcn_s_sleep(2); }
        FENCE;
    };
    auto post = [&](unsigned soff, unsigned fidx, unsigned sc, unsigned dq) {
        if (lane == 0) { LAS unsigned* jw = ctl + 8 + 4 * (n & 3); jw[0] = soff; jw[1] = fidx; jw[2] = sc; jw[3] = dq; }
        WAIT_L(0);
        ++n; if (lane == 0) lds_store(ctl, n);
    };
    unsigned t1 = own < 0 ? NCW + cw : 0, t2 = own < 0 ? 2 * NCW + cw : 1, drawn = 0;
    const unsigned jend = own < 0 ? J3 : 8 * (2 + DX);
    if (own >= 0) {
        const u64 t0 = __builtin_amdgcn_s_memrealtime();
        while ((unsigned)__builtin_amdgcn_readfirstlane(__hip_atomic_load(tick, RLX_AGENT)) - tbase < (unsigned)OWN_GO && !give_up(tmo, call, t0)) __builtin_amdgcn_s_sleep(32);
    }
#pragma unroll 1
    while (t1 < jend) {
        const bool dyn = own < 0 && n + 2u >= (unsigned)NSTAT;
        if (dyn && lane == 0) draw_async(drawn, tick);
        const Job cur = decode_job((int)t1, own, p.rank);
        if (cur.dq != 0) {
            u64 fl; const u64 t0 = __builtin_amdgcn_s_memrealtime();
            for (;;) {
                fl = __hip_atomic_load(flagq + cur.panel * 64 + cur.u, RLX_SYS);
                if ((unsigned)__builtin_amdgcn_readfirstlane((int)(fl >> 32)) == call) break;
                if (bad || give_up(tmo, call, t0)) { bad = true; break; }
                __builtin_amdgcn_s_sleep(2);
            }
            slot_free();
            const unsigned off = (unsigned)cur.panel * PANEL + (unsigned)cur.u * BLK;
#pragma unroll
            for (int k = 0; k < 32; ++k) __builtin_amdgcn_raw_ptr_buffer_load_lds(rsD, (LAS void*)(lds + (n & 3) * BLK + k * 1024), 16, lane * 16, (int)(off + k * 1024), 0, 17);
            WAIT_V(0);
            if (!bad) post((unsigned)cur.panel * PANEL + (unsigned)cur.u * 7u * BLK, (unsigned)(cur.panel * 64 + cur.u * 7), (unsigned)fl, cur.dq < 0 ? 8u : (unsigned)cur.dq);
        }
        WAIT_V(0); asm volatile("" : "+v"(drawn));
        t1 = t2; t2 = dyn ? __builtin_amdgcn_readfirstlane(drawn) - tbase : t2 + (own < 0 ? NCW : 1);
    }
    if (own >= 0) { slot_free(); post(0u, 0u, 0u, 10u); return; }
    u64 ready = 0;
    while (t1 < (unsigned)JTOT) {
        const bool pending = t2 < (unsigned)JTOT;
        if (pending && lane == 0) draw_async(drawn, tick);
        const int tile = (int)(t1 - J3) >> 3;
        if (!((ready >> tile) & 1u)) {
            gu32* cnt = (gu32*)(p.ws[0] + OFF_OWN + lane * 128);
            const u64 t0 = __builtin_amdgcn_s_memrealtime();
            for (;;) {
                ready = __ballot(__hip_atomic_load(cnt, RLX_AGENT) >= 8u * call);
                if ((ready >> tile) & 1u) break;
                if (give_up(tmo, call, t0)) { bad = true; ready = ~0ull; break; }
                __builtin_amdgcn_s_sleep(8);
            }
        }
        int pm, pn; tile_of(2, tile & 7, 0, tile >> 3, p.rank, pm, pn);
        const unsigned off = (unsigned)((pm * 256 + (int)((t1 - J3) & 7) * 32) * (N * 4) + pn * 1024);
        slot_free();
        if (!bad) {
#pragma unroll
            for (int k = 0; k < 32; ++k) __builtin_amdgcn_raw_ptr_buffer_load_lds(rsO, (LAS void*)(lds + (n & 3) * BLK + k * 1024), 16, lane * 16, (int)(off + (unsigned)k * (N * 4)), 0, 18);
        } else {
            for (int k = 0; k < 32; ++k) *(LAS u32x4*)(lds + (n & 3) * BLK + k * 1024 + lane * 16) = (u32x4){0x7fc00000u, 0x7fc00000u, 0x7fc00000u, 0x7fc00000u};
        }
        WAIT_V(0); asm volatile("" : "+v"(drawn));
        post(off, 0u, 0u, 9u);
        t1 = t2; t2 = pending ? __builtin_amdgcn_readfirstlane(drawn) - tbase : ~0u;
    }
    slot_free(); post(0u, 0u, 0u, 10u);
    {
        const u64 t0 = __builtin_amdgcn_s_memrealtime();
        gu32* dn = (gu32*)(p.ws[0] + OFF_DONE + cw * 128) + (lane & 7);
        for (;;) {
            const unsigned val = __hip_atomic_load(dn, RLX_SYS);
            if (__all((lane & 7) == 0 || val >= call)) break;
            if (give_up(tmo, call, t0)) { bad = true; break; }
            __builtin_amdgcn_s_sleep(4);
        }
        __builtin_amdgcn_fence(__ATOMIC_ACQUIRE, "");
        if (bad && lane == 0) ((float*)p.out[0])[(size_t)(NSH + 2 * p.rank) * 256 + cw] = __builtin_nanf("");
    }
}

__device__ __forceinline__ void gemm_role(const Params& p, LAS unsigned char* lds, int kind, int id, int row, unsigned call) {
    const int own_id = kind == 2 ? row * NOWN + id : -1, ntiles = kind < 2 ? ROUNDS : row ? 1 : 4;
    int pm, pn; tile_of(kind, id, row, kind == 2 ? row : 0, p.rank, pm, pn);
    const int tid = threadIdx.x, wid = __builtin_amdgcn_readfirstlane(tid >> 6), lane = tid & 63, wr = wid >> 2, wc = wid & 3;
    const unsigned par = call & 1u;
    const unsigned voff = (unsigned)wid * 2048u + (unsigned)lane * 16u;
    const unsigned ldsw = (unsigned)wid * 2048u;
    const int aoff = wr * 8192 + lane * 16, boff = wc * 4096 + lane * 16;
    gu32* tmo = (gu32*)(p.ws[0] + OFF_TMO);
    const __amdgpu_buffer_rsrc_t rsS = make_rsrc(p.ws[0] + OFF_STAGE + (size_t)par * STAGE_PAR, STAGE_PAR);
    const u32x4 dsC = {(unsigned)__builtin_amdgcn_readfirstlane((int)(u64)p.out[0]), (unsigned)__builtin_amdgcn_readfirstlane((int)((u64)p.out[0] >> 32)) & 0xffffu, OUT_BYTES, 0x00020000u};
    gu64* flags = (gu64*)(p.ws[0] + OFF_FLAG) + par * (NPAN * 64);
    bool bad = false;
#define SA_(b, h) (((b) * 2 + (h)) * HTB)
#define SB_(b, h) ((4 + (b) * 2 + (h)) * HTB)
#define STAGE(bufoff, soff) do { \
    __builtin_amdgcn_raw_ptr_buffer_load_lds(rsS, (LAS void*)(lds + (bufoff) + ldsw), 16, (int)voff, (int)(soff), 0, GLDS_AUX); \
    __builtin_amdgcn_raw_ptr_buffer_load_lds(rsS, (LAS void*)(lds + (bufoff) + ldsw), 16, (int)voff, (int)(soff), 1024, GLDS_AUX); } while (0)
#define LDA(dst, b, h) do { _Pragma("unroll") for (int m = 0; m < 4; ++m) _Pragma("unroll") for (int k = 0; k < 2; ++k) dst[m][k] = *(const LAS f16x8*)(lds + SA_(b, h) + aoff + m * 2048 + k * 1024); } while (0)
#define LDB(dst, b, h) do { _Pragma("unroll") for (int n = 0; n < 2; ++n) _Pragma("unroll") for (int k = 0; k < 2; ++k) dst[n][k] = *(const LAS f16x8*)(lds + SB_(b, h) + boff + n * 2048 + k * 1024); } while (0)
    if (wr == 1) __builtin_amdgcn_s_setprio(1);
#define ZERO4 ((f32x4){0.f, 0.f, 0.f, 0.f})
#define MF2(Z, i, R, m, n, Bt) do { if (Z) asm volatile("v_mfma_f32_16x16x32_f16 %0, %1, %2, 0\n\tv_mfma_f32_16x16x32_f16 %0, %3, %4, %0" : "+{" R "}"(acc[i]) : "v"(At[m][0]), "v"(Bt[n][0]), "v"(At[m][1]), "v"(Bt[n][1])); \
      else asm volatile("v_mfma_f32_16x16x32_f16 %0, %1, %2, %0\n\tv_mfma_f32_16x16x32_f16 %0, %3, %4, %0" : "+{" R "}"(acc[i]) : "v"(At[m][0]), "v"(Bt[n][0]), "v"(At[m][1]), "v"(Bt[n][1])); } while (0)
#define MMA2_0(Z) do { \
      MF2(Z, 0, "v[128:131]", 0, 0, B0); MF2(Z, 1, "v[132:135]", 0, 1, B0); MF2(Z, 8, "v[160:163]", 0, 0, B1); MF2(Z, 9, "v[164:167]", 0, 1, B1); \
      MF2(Z, 2, "v[136:139]", 1, 0, B0); MF2(Z, 3, "v[140:143]", 1, 1, B0); MF2(Z, 10, "v[168:171]", 1, 0, B1); MF2(Z, 11, "v[172:175]", 1, 1, B1); \
      MF2(Z, 4, "v[144:147]", 2, 0, B0); MF2(Z, 5, "v[148:151]", 2, 1, B0); MF2(Z, 12, "v[176:179]", 2, 0, B1); MF2(Z, 13, "v[180:183]", 2, 1, B1); \
      MF2(Z, 6, "v[152:155]", 3, 0, B0); MF2(Z, 7, "v[156:159]", 3, 1, B0); MF2(Z, 14, "v[184:187]", 3, 0, B1); MF2(Z, 15, "v[188:191]", 3, 1, B1); \
      } while (0)
#define MMA2_1(Z) do { \
      MF2(Z, 16, "v[192:195]", 0, 0, B0); MF2(Z, 17, "v[196:199]", 0, 1, B0); MF2(Z, 24, "v[224:227]", 0, 0, B1); MF2(Z, 25, "v[228:231]", 0, 1, B1); \
      MF2(Z, 18, "v[200:203]", 1, 0, B0); MF2(Z, 19, "v[204:207]", 1, 1, B0); MF2(Z, 26, "v[232:235]", 1, 0, B1); MF2(Z, 27, "v[236:239]", 1, 1, B1); \
      MF2(Z, 20, "v[208:211]", 2, 0, B0); MF2(Z, 21, "v[212:215]", 2, 1, B0); MF2(Z, 28, "v[240:243]", 2, 0, B1); MF2(Z, 29, "v[244:247]", 2, 1, B1); \
      MF2(Z, 22, "v[216:219]", 3, 0, B0); MF2(Z, 23, "v[220:223]", 3, 1, B0); MF2(Z, 30, "v[248:251]", 3, 0, B1); MF2(Z, 31, "v[252:255]", 3, 1, B1); \
      } while (0)
#define MMA2(Z, ai) MMA2_##ai(Z)
#define ST4(POL) "buffer_store_dwordx4 v[112:115], %[vo], %[rs], %[so] offen " POL "\n\ts_add_i32 %[t], %[so], 0x20000\n\tbuffer_store_dwordx4 v[116:119], %[vo], %[rs], %[t] offen " POL "\n\t" \
      "s_add_i32 %[t], %[so], 0x40000\n\tbuffer_store_dwordx4 v[120:123], %[vo], %[rs], %[t] offen " POL "\n\ts_add_i32 %[t], %[so], 0x60000\n\tbuffer_store_dwordx4 v[124:127], %[vo], %[rs], %[t] offen " POL "\n\t"
#define STORE_TEXT(r) ".irp j, 0, 1, 2, 3\n\tv_ldexp_f32 v[112 + 4 * \\j], v[" #r " + \\j], %[ex]\n\tv_ldexp_f32 v[113 + 4 * \\j], v[" #r " + 4 + \\j], %[ex]\n\t" \
      "v_ldexp_f32 v[114 + 4 * \\j], v[" #r " + 32 + \\j], %[ex]\n\tv_ldexp_f32 v[115 + 4 * \\j], v[" #r " + 36 + \\j], %[ex]\n\t.endr\n\t" \
      "s_cmp_lt_i32 %[ow], 0\n\ts_cbranch_scc1 1f\n\t" ST4("nt sc1") "s_branch 2f\n\t1:\n\t" ST4("nt") "2:\n\ts_nop 0"
#define STORE_GROUP(r, R0, R1, R2, R3, i, cbase, ex_) do { unsigned _t; asm volatile(STORE_TEXT(r) : [t] "=&s"(_t) \
      : "{" R0 "}"(acc[i]), "{" R1 "}"(acc[(i) + 1]), "{" R2 "}"(acc[(i) + 8]), "{" R3 "}"(acc[(i) + 9]), [ex] "v"(ex_), [vo] "v"(coff), [rs] "s"(dsC), [so] "s"(cbase), [ow] "s"(own_id) \
      : "scc", "memory", "v112", "v113", "v114", "v115", "v116", "v117", "v118", "v119", "v120", "v121", "v122", "v123", "v124", "v125", "v126", "v127"); } while (0)
#define STORE_00(cbase, ex) STORE_GROUP(128, "v[128:131]", "v[132:135]", "v[160:163]", "v[164:167]", 0, (cbase) + 0u * (N * 4), ex)
#define STORE_01(cbase, ex) STORE_GROUP(136, "v[136:139]", "v[140:143]", "v[168:171]", "v[172:175]", 2, (cbase) + 16u * (N * 4), ex)
#define STORE_02(cbase, ex) STORE_GROUP(144, "v[144:147]", "v[148:151]", "v[176:179]", "v[180:183]", 4, (cbase) + 32u * (N * 4), ex)
#define STORE_03(cbase, ex) STORE_GROUP(152, "v[152:155]", "v[156:159]", "v[184:187]", "v[188:191]", 6, (cbase) + 48u * (N * 4), ex)
#define STORE_10(cbase, ex) STORE_GROUP(192, "v[192:195]", "v[196:199]", "v[224:227]", "v[228:231]", 16, (cbase) + 128u * (N * 4), ex)
#define STORE_11(cbase, ex) STORE_GROUP(200, "v[200:203]", "v[204:207]", "v[232:235]", "v[236:239]", 18, (cbase) + 144u * (N * 4), ex)
#define STORE_12(cbase, ex) STORE_GROUP(208, "v[208:211]", "v[212:215]", "v[240:243]", "v[244:247]", 20, (cbase) + 160u * (N * 4), ex)
#define STORE_13(cbase, ex) STORE_GROUP(216, "v[216:219]", "v[220:223]", "v[248:251]", "v[252:255]", 22, (cbase) + 176u * (N * 4), ex)
#define ST_L do { const int _e = -esc; STORE_00(cC, _e); STORE_01(cC, _e); STORE_02(cC, _e); } while (0)
#define ST_F1 do { STORE_03(cPrev, eprev); STORE_10(cPrev, eprev); STORE_11(cPrev, eprev); } while (0)
#define ST_F2 do { STORE_12(cPrev, eprev); STORE_13(cPrev, eprev); } while (0)
#define POLL(a, b, pos2, maskv, expv) do { const u64 _t0 = __builtin_amdgcn_s_memrealtime(); bool _ok = false; \
      for (int _n = 0;; ++_n) { \
        if (_n == 0 || (lane >> 4) == ((pos2) >> 3)) {     \
          const u64 _fa = __hip_atomic_load(flags + (a) * 64 + lane, RLX_SYS), _fb = __hip_atomic_load(flags + (NPA + (b)) * 64 + lane, RLX_SYS); \
          _ok = (unsigned)(_fa >> 32) == call && (unsigned)(_fb >> 32) == call; \
          expv = ((int)_fa == ZSENT || (int)_fb == ZSENT) ? ZSENT : (int)_fa + (int)_fb; } \
        maskv = __ballot(_ok); \
        if (((maskv >> (2 * (pos2))) & 3ull) == 3ull) break; \
        if (bad || ((_n & 63) == 63 && give_up(tmo, call, _t0))) { bad = true; maskv = ~0ull; expv = ZSENT; break; } \
        __builtin_amdgcn_s_sleep(16); } WAIT_L(0); } while (0)
#define SET_CMASK cmask = __ballot(ecur != esc && ecur != ZSENT)
#define RESCALE(pos_) do { int _d; asm volatile( \
        "s_bitcmp1_b64 %[cm], %[pos]\n\t" \
        "s_cbranch_scc0 1f\n\t" \
        "s_nop 15\n\t" \
        "v_readlane_b32 %[d], %[ec], %[pos]\n\t" \
        "s_nop 3\n\t" \
        "v_cmp_ne_u32 %[cm], %[d], %[ec]\n\t" \
        "v_cmp_ne_u32 vcc, 2.0, %[ec]\n\t" \
        "s_xor_b32 %[es], %[es], %[d]\n\t" \
        "s_xor_b32 %[d], %[es], %[d]\n\t" \
        "s_xor_b32 %[es], %[es], %[d]\n\t" \
        "s_sub_i32 %[d], %[es], %[d]\n\t" \
        "s_and_b64 %[cm], %[cm], vcc\n\t" \
        ".set gr_, 128\n\t" \
        ".rept 128\n\t" \
        "v_ldexp_f32 v[gr_], v[gr_], %[d]\n\t" \
        ".set gr_, gr_ + 1\n\t" \
        ".endr\n\t" \
        "s_nop 3\n\t" \
        "1:\n\t" \
        : [cm] "+s"(cmask), [es] "+s"(esc), [d] "=&s"(_d), \
          "+{v[128:131]}"(acc[0]), "+{v[132:135]}"(acc[1]), "+{v[136:139]}"(acc[2]), "+{v[140:143]}"(acc[3]), "+{v[144:147]}"(acc[4]), "+{v[148:151]}"(acc[5]), "+{v[152:155]}"(acc[6]), "+{v[156:159]}"(acc[7]), \
          "+{v[160:163]}"(acc[8]), "+{v[164:167]}"(acc[9]), "+{v[168:171]}"(acc[10]), "+{v[172:175]}"(acc[11]), "+{v[176:179]}"(acc[12]), "+{v[180:183]}"(acc[13]), "+{v[184:187]}"(acc[14]), "+{v[188:191]}"(acc[15]), \
          "+{v[192:195]}"(acc[16]), "+{v[196:199]}"(acc[17]), "+{v[200:203]}"(acc[18]), "+{v[204:207]}"(acc[19]), "+{v[208:211]}"(acc[20]), "+{v[212:215]}"(acc[21]), "+{v[216:219]}"(acc[22]), "+{v[220:223]}"(acc[23]), \
          "+{v[224:227]}"(acc[24]), "+{v[228:231]}"(acc[25]), "+{v[232:235]}"(acc[26]), "+{v[236:239]}"(acc[27]), "+{v[240:243]}"(acc[28]), "+{v[244:247]}"(acc[29]), "+{v[248:251]}"(acc[30]), "+{v[252:255]}"(acc[31]) \
        : [pos] "s"(pos_), [ec] "v"(ecur) : "vcc", "scc"); } while (0)
#define SET_NRDY do { nrdy = ~rmask ? __builtin_ctzll(~rmask) >> 1 : 32; evt = min(min(30, nrdy - 2), pubit); } while (0)
#define FETCH_FLAGS(a, b) do { const gu64* _pa = flags + (a) * 64 + lane; const gu64* _pb = flags + (NPA + (b)) * 64 + lane; \
      asm volatile("global_load_dwordx2 %0, %2, off sc0 sc1\n\tglobal_load_dwordx2 %1, %3, off sc0 sc1" : "=&v"(nfa), "=&v"(nfb) : "v"(_pa), "v"(_pb) : "memory"); } while (0)
#define BODY(FIRST, LAST, EXTRA) do { \
      const int pp = it, ppn = (pp + 1) & 31; \
      const unsigned a1 = cA + (unsigned)(pp * 2 + 1) * BLK; \
      const unsigned a2 = ((LAST) ? nA : cA) + (unsigned)(ppn * 2) * BLK, b2 = ((LAST) ? nB : cB) + (unsigned)(ppn * 2) * BLK; \
      const unsigned a3 = a2 + BLK, b3 = b2 + BLK; \
      LDB(B0, 0, 0); LDB(B1, 0, 1); SCHED; LDA(At, 0, 0); STAGE(SA_(1, 1), a1 + HTB); FENCE; \
      if (FIRST) { const int e = __builtin_amdgcn_readlane(ecur, 2 * pp); esc = e == ZSENT ? 0 : e; SET_CMASK; } else RESCALE(2 * pp); \
      if ((FIRST) && ui > 0) { ST_F1; WAIT_V(32); } else WAIT_V(8); \
      WAIT_L(0); BAR; MMA2(FIRST, 0); BAR; SCHED; \
      LDA(At, 0, 1); FENCE; \
      if (LAST) { if (has_next) {     \
          asm volatile("" : "+v"(nfa), "+v"(nfb)); \
          rmask = __ballot((unsigned)(nfa >> 32) == call && (unsigned)(nfb >> 32) == call); \
          enext = ((int)nfa == ZSENT || (int)nfb == ZSENT) ? ZSENT : (int)nfa + (int)nfb; \
          if (((rmask >> (2 * ppn)) & 3ull) != 3ull) POLL(npm, npn, ppn, rmask, enext); SET_NRDY; } } \
      else if (__builtin_expect(ppn >= nrdy, 0)) { \
          if (!(FIRST)) {             \
            asm volatile("" : "+v"(nfa), "+v"(nfb)); \
            rmask = __ballot((unsigned)(nfa >> 32) == call && (unsigned)(nfb >> 32) == call); \
            ecur = ((int)nfa == ZSENT || (int)nfb == ZSENT) ? ZSENT : (int)nfa + (int)nfb; } \
          if (((rmask >> (2 * ppn)) & 3ull) != 3ull) POLL(pm, pn, ppn, rmask, ecur); \
          SET_CMASK; SET_NRDY; } \
      STAGE(SB_(0, 0), b2); STAGE(SB_(0, 1), b2 + HTB); STAGE(SA_(0, 0), a2); \
      if ((FIRST) && ui > 0) { FENCE; ST_F2; WAIT_V(40); } else WAIT_V(8); \
      WAIT_L(0); BAR; MMA2(FIRST, 1); BAR; SCHED; \
      LDB(B0, 1, 0); LDB(B1, 1, 1); SCHED; LDA(At, 1, 0); STAGE(SA_(0, 1), a2 + HTB); FENCE; \
      RESCALE(2 * pp + 1); \
      if ((FIRST) && ui > 0) WAIT_V(28); else WAIT_V(8); \
      WAIT_L(0); BAR; MMA2(0, 0); BAR; SCHED; \
      LDA(At, 1, 1); FENCE; EXTRA; STAGE(SB_(1, 0), b3); STAGE(SB_(1, 1), b3 + HTB); STAGE(SA_(1, 0), a3); \
      if (LAST) { FENCE; ST_L; WAIT_V(20); } else if ((FIRST) && ui > 0) WAIT_V(16); else WAIT_V(8); \
      WAIT_L(0); BAR; MMA2(0, 1); BAR; SCHED; } while (0)
    f32x4 acc[32];
#pragma unroll
    for (int i = 0; i < 32; ++i) acc[i] = ZERO4;
    f16x8 At[4][2], B0[2][2], B1[2][2];
    unsigned cA = (unsigned)pm * PANEL, cB = (unsigned)(NPA + pn) * PANEL;
    const unsigned coff = (unsigned)((wr * 64 + (lane >> 4) * 4) * (N * 4) + (wc * 64 + (lane & 15) * 4) * 4);
    unsigned cC = (unsigned)pm * 256u * (N * 4) + (unsigned)pn * 1024u, cPrev = cC;
    u64 rmask, cmask = 0, nfa = 0, nfb = 0; int ecur, enext = 0, esc = 0, eprev = 0;
    int nrdy, evt, pubit = 64;
    POLL(pm, pn, 0, rmask, ecur); SET_NRDY;
    STAGE(SB_(0, 0), cB); STAGE(SB_(0, 1), cB + HTB); STAGE(SA_(0, 0), cA); STAGE(SA_(0, 1), cA + HTB);
    if (wr == 1) BAR;
    WAIT_V(2); BAR;
    STAGE(SB_(1, 0), cB + BLK); STAGE(SA_(1, 0), cA + BLK); STAGE(SB_(1, 1), cB + BLK + HTB);
    WAIT_V(6); BAR;
#pragma unroll 1
    for (int ui = 0; ui < ntiles; ++ui) {
        const bool has_next = ui + 1 < ntiles;
        int npm = pm, npn = pn; unsigned nA = 0, nB = 0;
        int it = 0;
        BODY(1, 0, { if (nrdy <= 2) FETCH_FLAGS(pm, pn); });
#pragma unroll 1
        for (it = 1; it < 31; ++it) {
            BODY(0, 0, {
                if (__builtin_expect(it >= evt, 0)) {
                    if (it == 30) { if (has_next) tile_of(kind, id, row, ui + 1, p.rank, npm, npn); FETCH_FLAGS(npm, npn); pubit = own_id >= 0 ? 1 : 64; }
                    else if (it + 2 >= nrdy) FETCH_FLAGS(pm, pn);
                    if (it == pubit) {
                        WAIT_V(0);
                        if (lane == 0) __hip_atomic_fetch_add((gu32*)(p.ws[0] + OFF_OWN + ((ui - 1) * NOWN + own_id) * 128), 1u, RLX_AGENT);
                        pubit = 64; evt = min(30, nrdy - 2);
                    }
                } });
        }
        nA = (unsigned)npm * PANEL; nB = (unsigned)(NPA + npn) * PANEL;
        BODY(0, 1, {});
        cPrev = cC; eprev = -esc; ecur = enext;
        cA = nA; cB = nB; pm = npm; pn = npn;
        cC = (unsigned)pm * 256u * (N * 4) + (unsigned)pn * 1024u;
    }
    asm volatile("s_nop 15" ::: "memory"); SCHED;
    ST_F1; ST_F2;
    WAIT_V(0);
    if (bad) { *(float*)(p.out[0] + cPrev + coff) = __builtin_nanf(""); WAIT_V(0); }
    if (own_id >= 0) {
        if (lane == 0) __hip_atomic_fetch_add((gu32*)(p.ws[0] + OFF_OWN + ((ntiles - 1) * NOWN + own_id) * 128), 1u, RLX_AGENT);
    }
    if (wr == 0) BAR;
    BAR;
}

__global__ __launch_bounds__(512, 2) void fused_kernel(Params p) {
    extern __shared__ __attribute__((aligned(16))) unsigned char shm[];
    LAS unsigned char* lds = (LAS unsigned char*)shm;
    gu32* cc = (gu32*)(p.ws[0] + OFF_CALL + blockIdx.x * 64);
    const int x = blockIdx.x & 7, o = blockIdx.x >> 3;
    unsigned call;
    if ((int)blockIdx.x < NCW) {
        const int wid = __builtin_amdgcn_readfirstlane(threadIdx.x >> 6), lane = threadIdx.x & 63;
        LAS unsigned* ctl = (LAS unsigned*)(lds + 4 * BLK);
        const Job j = decode_job(blockIdx.x, -1, 0);
        f32x4 v[8];
        block_loads(p, j, wid, lane, v);
        call = (unsigned)__builtin_amdgcn_readfirstlane(__hip_atomic_load(cc, RLX_AGENT)) + 1u;
        const unsigned pmx = piece_max(v);
        if (lane == 0) ctl[32 + wid] = pmx;
        WAIT_V(0); WAIT_L(0); BAR; FENCE;
        const unsigned mx = block_max(ctl + 32, lane);
        const int s = scale_exp(mx);
        block_pack(j, wid, v, s, [&](u32x4 o, int d) { *(LAS u32x4*)(lds + d + lane * 16) = o; });
        if (threadIdx.x == 0) { ctl[8] = (unsigned)j.panel * PANEL + (unsigned)j.u * 7u * BLK; ctl[9] = (unsigned)(j.panel * 64 + j.u * 7); ctl[10] = (unsigned)(mx ? s : ZSENT); ctl[11] = 8u; }
        if (threadIdx.x == 0) __hip_atomic_store(cc, call, RLX_AGENT);
    } else {
        unsigned drawn = 0;
        if (threadIdx.x == 0) draw_async(drawn, (gu32*)(p.ws[0] + OFF_TICKET + 128));
        call = (unsigned)__builtin_amdgcn_readfirstlane(__hip_atomic_load(cc, RLX_AGENT)) + 1u;
        WAIT_V(0); asm volatile("" : "+v"(drawn)); BAR;
        if (threadIdx.x == 0) __hip_atomic_store(cc, call, RLX_AGENT);
        convert_phase(p, lds, drawn, call);
    }
    call = (unsigned)__builtin_amdgcn_readfirstlane((int)call);
    int kind = 0, id = (x & 1) * 7 + ((o - 4) >> 2), row = (x >> 1) * 4 + (o & 3), snd = -1;
    if (o < 3) kind = -1;
    else if (o == 3) { kind = 2; id = x; row = 0; snd = DX ? -1 : x; }
    else if (id >= NSTR) { if (o & 1) { kind = 2; id = (x & 6) + ((o >> 1) & 1); row = 4; snd = id; } else kind = 1; }
    if ((int)blockIdx.x < NCW || snd >= 0) {
        comm_role(p, lds, blockIdx.x, snd, call);
        WAIT_L(0); BAR; FENCE;
    }
    if (kind < 0) return;
    gemm_role(p, lds, kind, id, row, call);
}

}

extern "C" void kernel_launch(int rank, int nranks, ncclComm_t comm, hipStream_t stream,
                              void* const* d_in, const int* in_sizes, int n_in,
                              void* d_out, int out_size, const int* in_dtypes, int out_dtype,
                              void* const* d_out_all, void* const* d_ws_all, size_t ws_bytes) {
    if (nranks != NR || (unsigned)rank >= (unsigned)NR || ws_bytes < WS_NEED || n_in < 2 ||
        in_sizes[0] != M * KS || in_sizes[1] != KS * N || out_size != M * N) {
        fprintf(stderr, "kernel_launch: unexpected problem shape\n");
        return;
    }
    Params p;
    p.x = (const float*)d_in[0]; p.w = (const float*)d_in[1]; p.rank = rank;
    for (int q = 0; q < NR; ++q) { p.ws[q] = (char*)d_ws_all[(rank + q) % NR]; p.out[q] = (char*)d_out_all[(rank + q) % NR]; }
    p.out[0] = (char*)d_out;
    HIP_CHECK(hipFuncSetAttribute((const void*)fused_kernel, hipFuncAttributeMaxDynamicSharedMemorySize, LDS_BYTES));
    hipLaunchKernelGGL(fused_kernel, dim3(256), dim3(512), LDS_BYTES, stream, p);
    HIP_CHECK(hipGetLastError());
}
```

```cpp
#include <hip/hip_runtime.h>
#include <rccl/rccl.h>
#include <cstdio>

#define HIP_CHECK(expr) do { hipError_t _e = (expr); if (_e != hipSuccess) { \
    fprintf(stderr, "HIP error %s at %s:%d\n", hipGetErrorString(_e), __FILE__, __LINE__); \
    return; } } while (0)

namespace {

#define LAS __attribute__((address_space(3)))
#define GAS __attribute__((address_space(1)))
typedef _Float16 f16x8 __attribute__((ext_vector_type(8)));
typedef _Float16 f16x2 __attribute__((ext_vector_type(2)));
typedef float f32x4 __attribute__((ext_vector_type(4)));
typedef unsigned u32x4 __attribute__((ext_vector_type(4)));
typedef unsigned long long u64;
typedef GAS unsigned gu32;
typedef GAS u64 gu64;

constexpr int M = 4096, N = 32768, KS = 512, NR = 8;
constexpr int NPA = 16, NPB = 128, NPAN = NPA + NPB;
constexpr unsigned BLK = 32768, PANEL = 64 * BLK, HTB = 16384;
constexpr int DX = 1;
constexpr int ROUNDS = 8, NSTR = 14 - DX, NSH = NSTR * ROUNDS + 4 * DX;
constexpr int NOWN = 8, OWN_TILES = 4 + DX;
constexpr int NCW = 24;
#define WIN 2
constexpr int LDS_BYTES = 8 * HTB + 256;
constexpr unsigned OFF_CALL = 0;
constexpr unsigned OFF_TMO = 16384;
constexpr unsigned OFF_OWN = 20480;
constexpr unsigned OFF_TICKET = 28672;
constexpr unsigned OFF_DONE = 32768;
constexpr unsigned OFF_FLAG = 65536;
constexpr unsigned OFF_STAGE = 1u << 20;
constexpr unsigned STAGE_PAR = NPAN * PANEL;
constexpr size_t WS_NEED = (size_t)OFF_STAGE + 2ull * STAGE_PAR;
constexpr unsigned OUT_BYTES = (unsigned)((size_t)M * N * 4);
constexpr u64 DEADLINE = 100000000ull;
constexpr int ZSENT = 0x40000000;
constexpr int J0 = 8 * (NPA + 14);
constexpr int J3 = 8 * (NPA + NSH);
constexpr int JTOT = J3 + NOWN * OWN_TILES * 8;
constexpr int OWN_GO = J0 + 8 * NCW;
constexpr int GLDS_AUX = 17, ST_AUX = 17, SRC_AUX = 2, SLAB_AUX = 19;

struct Params {
    const float* x; const float* w;
    char* ws[NR];
    char* out[NR];
    int rank;
};

#define RLX_SYS __ATOMIC_RELAXED, __HIP_MEMORY_SCOPE_SYSTEM
#define RLX_AGENT __ATOMIC_RELAXED, __HIP_MEMORY_SCOPE_AGENT
#define WAIT_V(n) asm volatile("s_waitcnt vmcnt(" #n ")" ::: "memory")
#define WAIT_L(n) do { static_assert((n) == 0, ""); __builtin_amdgcn_s_waitcnt(0xC07F); asm volatile("" ::: "memory"); } while (0)
#define BAR __builtin_amdgcn_s_barrier()
#define SCHED __builtin_amdgcn_sched_barrier(0)
#define FENCE do { asm volatile("" ::: "memory"); SCHED; } while (0)

__device__ __forceinline__ unsigned pk16(float a, float b, int s) {
    f16x2 v; v.x = (_Float16)__builtin_ldexpf(a, s); v.y = (_Float16)__builtin_ldexpf(b, s);
    return __builtin_bit_cast(unsigned, v);
}
__device__ __forceinline__ __amdgpu_buffer_rsrc_t make_rsrc(const char* base, unsigned bytes) {
    const u64 a = (u64)base;
    const u64 u = ((u64)(unsigned)__builtin_amdgcn_readfirstlane((int)(a >> 32)) << 32) | (unsigned)__builtin_amdgcn_readfirstlane((int)a);
    return __builtin_amdgcn_make_buffer_rsrc((void*)u, 0, __builtin_amdgcn_readfirstlane((int)bytes), 0x00020000);
}
__device__ __forceinline__ bool give_up(gu32* tmo, unsigned call, u64 t0) {
    const bool bad = (unsigned)__builtin_amdgcn_readfirstlane(__hip_atomic_load(tmo, RLX_AGENT)) == call ||
                     __builtin_amdgcn_s_memrealtime() - t0 > DEADLINE;
    if (bad) __hip_atomic_store(tmo, call, RLX_AGENT);
    return bad;
}

__device__ __forceinline__ void draw_async(unsigned& t, gu32* ctr) {
    asm volatile("global_atomic_add %0, %1, %2, off sc0" : "=v"(t) : "v"(ctr), "v"(1u) : "memory");
}

__device__ __forceinline__ int own_panel(int h, int rank) { return h < 2 ? NSH + 2 * rank + h : NSH + 16 + (rank >> 1); }
struct Job { int panel, u, dq; };
__device__ __forceinline__ Job decode_job(int n, int own, int rank) {
    Job j; j.dq = -1;
    if (own >= 0) { j.u = n / (2 + DX); j.dq = (own + n + 1) & 7; j.panel = NPA + own_panel(n % (2 + DX), (rank + j.dq) & 7); return j; }
    int c, k;
    if (n < J0) { c = 0; j.u = n / (NPA + 14); k = n % (NPA + 14) - NPA; if (k < 0) { j.panel = k + NPA; return j; } }
    else { const int m = n - J0, pr = m / (8 * (NSTR + 14)), r = m % (8 * (NSTR + 14));
        if (r < 8 * NSTR) { c = 2 * pr + 1; j.u = r / NSTR; k = r % NSTR; } else { c = 2 * pr + 2; j.u = (r - 8 * NSTR) / 14; k = (r - 8 * NSTR) % 14; } }
    j.panel = NPA + (k < NSTR ? c * NSTR + k : NSTR * ROUNDS + (c >> 1));
    return j;
}
__device__ __forceinline__ void tile_of(int kind, int id, int row, int c, int rank, int& pm, int& pn) {
    if (kind == 0) { pm = row; pn = c * NSTR + id; }
    else if (kind == 1) { pm = row + (c & 1); pn = NSTR * ROUNDS + (c >> 1); }
    else if (c < 4) { pm = (id >> 1) * 4 + c; pn = own_panel(id & 1, rank); }
    else { pm = (rank & 1) * 8 + id; pn = own_panel(2, rank); }
}

constexpr int NSTAT = J0 / NCW;
constexpr unsigned DRAWS = JTOT - NSTAT * NCW + NCW;
#define STR2(x) #x
#define STR(x) STR2(x)
#define WAIT_WIN asm volatile("s_waitcnt vmcnt(" STR(WIN) ")" ::: "memory")

__device__ __forceinline__ unsigned wave_max_u32(unsigned mx) {
#pragma unroll
    for (int k = 32; k >= 1; k >>= 1) mx = max(mx, (unsigned)__shfl_xor((int)mx, k));
    return __builtin_amdgcn_readfirstlane(mx);
}
__device__ __forceinline__ int scale_exp(unsigned mx) { return ((14 - ((int)(mx >> 23) - 127)) >> 4) * 16; }
__device__ __forceinline__ unsigned lds_load(LAS unsigned* w) { return __builtin_amdgcn_readfirstlane(__hip_atomic_load(w, __ATOMIC_RELAXED, __HIP_MEMORY_SCOPE_WORKGROUP)); }
__device__ __forceinline__ void lds_store(LAS unsigned* w, unsigned v) { __hip_atomic_store(w, v, __ATOMIC_RELAXED, __HIP_MEMORY_SCOPE_WORKGROUP); }

constexpr int NCONV = NPAN * 8;
constexpr unsigned CDRAWS = NCONV + (256 - NCW);

__device__ __forceinline__ void block_loads(const Params& p, const Job& j, int wid, int lane, f32x4 (&v)[8]) {
    const bool a = j.panel < NPA;
    const __amdgpu_buffer_rsrc_t rs = make_rsrc(a ? (const char*)p.x : (const char*)p.w, a ? M * KS * 4 : KS * N * 4);
    const unsigned l = a ? (unsigned)((lane & 15) * KS + (lane >> 4) * 8) * 4u : (unsigned)((lane >> 4) * 8 * N + (lane & 15) * 4) * 4u;
    const unsigned b = a ? (unsigned)(j.panel * 256 * KS + j.u * 64) * 4u + (wid & 3) * (32 * KS * 4) + (wid >> 2) * (128 * KS * 4)
                         : (unsigned)(j.u * 64 * N + (j.panel - NPA) * 256) * 4u + (wid & 3) * 256 + (wid >> 2) * (32 * N * 4);
    const unsigned s0 = a ? 16 : N * 4, s1 = a ? 128 : 2 * N * 4, s2 = a ? 16 * KS * 4 : 4 * N * 4;
#pragma unroll
    for (int r = 0; r < 8; ++r) v[r] = __builtin_bit_cast(f32x4, __builtin_amdgcn_raw_buffer_load_b128(rs, (int)l, (int)(b + (r & 1) * s0 + ((r >> 1) & 1) * s1 + (r >> 2) * s2), SRC_AUX));
}
__device__ __forceinline__ unsigned piece_max(const f32x4 (&v)[8]) {
    unsigned mx = 0;
#pragma unroll
    for (int r = 0; r < 8; ++r) { const u32x4 u = __builtin_bit_cast(u32x4, v[r]) & 0x7fffffffu;
#pragma unroll
        for (int c = 0; c < 4; ++c) { const unsigned b = u[c]; mx = max(mx, b < 0x7f800000u ? b : 0u); } }
    return wave_max_u32(mx);
}
__device__ __forceinline__ unsigned block_max(const LAS unsigned* red, int lane) {
    unsigned mx = red[lane & 7];
#pragma unroll
    for (int k = 4; k >= 1; k >>= 1) mx = max(mx, (unsigned)__shfl_xor((int)mx, k));
    return __builtin_amdgcn_readfirstlane(mx);
}
template <class Put> __device__ __forceinline__ void block_pack(const Job& j, int wid, const f32x4 (&v)[8], int s, Put put) {
    if (j.panel < NPA) {
#pragma unroll
        for (int i = 0; i < 4; ++i) { const f32x4 a = v[2 * i], c = v[2 * i + 1]; u32x4 o;
            o.x = pk16(a.x, a.y, s); o.y = pk16(a.z, a.w, s); o.z = pk16(c.x, c.y, s); o.w = pk16(c.z, c.w, s); put(o, wid * 4096 + i * 1024); }
    } else {
#pragma unroll
        for (int i = 0; i < 4; ++i) { u32x4 o;
            o.x = pk16(v[0][i], v[1][i], s); o.y = pk16(v[2][i], v[3][i], s); o.z = pk16(v[4][i], v[5][i], s); o.w = pk16(v[6][i], v[7][i], s);
            put(o, (wid & 3) * 4096 + (wid >> 2) * 1024 + (i >> 1) * HTB + (i & 1) * 2048); }
    }
}

__device__ __forceinline__ void convert_phase(const Params& p, LAS unsigned char* lds, unsigned drawn, unsigned call) {
    const int tid = threadIdx.x, wid = __builtin_amdgcn_readfirstlane(tid >> 6), lane = tid & 63;
    const unsigned par = call & 1u, tbase = (call - 1u) * CDRAWS;
    gu32* tick = (gu32*)(p.ws[0] + OFF_TICKET + 128);
    gu64* flags = (gu64*)(p.ws[0] + OFF_FLAG) + par * (NPAN * 64);
    LAS unsigned* red = (LAS unsigned*)lds;
    const __amdgpu_buffer_rsrc_t rsD = make_rsrc(p.ws[0] + OFF_STAGE + (size_t)par * STAGE_PAR, STAGE_PAR);
    f32x4 v[8];
    auto job = [&](unsigned t) { Job j; if (t < (unsigned)J3) j = decode_job((int)t, -1, 0); else { j.u = (int)(t - J3) & 7; j.panel = NPA + NSH + ((int)(t - J3) >> 3); j.dq = -1; } return j; };
    if (tid == 0) red[8] = drawn;
    WAIT_L(0); BAR; FENCE;
    unsigned t = __builtin_amdgcn_readfirstlane(red[8]) - tbase;
    if (t < (unsigned)NCONV) {
        Job cur = job(t);
        if (tid == 0) draw_async(drawn, tick);
        block_loads(p, cur, wid, lane, v);
        for (;;) {
            unsigned mx = piece_max(v);
            asm volatile("" : "+v"(drawn), "+s"(mx));
            if (lane == 0) red[wid] = mx;
            if (tid == 0) red[9] = drawn;
            WAIT_L(0); BAR; FENCE;
            mx = block_max(red, lane);
            t = __builtin_amdgcn_readfirstlane(red[9]) - tbase;
            const int s = scale_exp(mx);
            const unsigned off = (unsigned)cur.panel * PANEL + (unsigned)cur.u * BLK;
            block_pack(cur, wid, v, s, [&](u32x4 o, int d) { __builtin_amdgcn_raw_buffer_store_b128(o, rsD, lane * 16, (int)(off + d), ST_AUX); });
            FENCE;
            const int fidx = cur.panel * 64 + cur.u;
            const bool more = t < (unsigned)NCONV;
            WAIT_V(0);
            if (more) { cur = job(t); if (tid == 0) draw_async(drawn, tick); block_loads(p, cur, wid, lane, v); }
            WAIT_L(0); BAR; FENCE;
            if (tid == 0) __hip_atomic_store(flags + fidx, ((u64)call << 32) | (unsigned)(mx ? s : ZSENT), RLX_SYS);
            if (!more) break;
        }
    }
    WAIT_L(0); BAR; FENCE;
}

__device__ __forceinline__ void comm_role(const Params& p, LAS unsigned char* lds, int cw, int own, unsigned call) {
    const int tid = threadIdx.x, wid = __builtin_amdgcn_readfirstlane(tid >> 6), lane = tid & 63;
    const unsigned par = call & 1u;
    gu32* tmo = (gu32*)(p.ws[0] + OFF_TMO);
    LAS unsigned* ctl = (LAS unsigned*)(lds + 4 * BLK);
    if (tid < 8) ctl[tid] = tid == 0 && own < 0 ? 1u : 0u;
    WAIT_L(0); BAR; FENCE;
    const int pos0 = wid == 0 ? 0 : 15 - wid;
    gu64* flagq = (gu64*)(p.ws[wid] + OFF_FLAG) + par * (NPAN * 64) + pos0;
    const __amdgpu_buffer_rsrc_t rsD = make_rsrc(p.ws[wid] + OFF_STAGE + (size_t)par * STAGE_PAR + pos0 * BLK, STAGE_PAR - pos0 * BLK);
    const __amdgpu_buffer_rsrc_t rsO = make_rsrc(p.out[wid], OUT_BYTES);
    int pidx = -1, psc = 0;
#define PUBLISH_PENDING do { if (pidx >= 0 && lane == 0) __hip_atomic_store(flagq + pidx, ((u64)call << 32) | (unsigned)psc, RLX_SYS); pidx = -1; } while (0)
    bool bad = false;
    if (wid > 0) {
        for (unsigned n = 0;; ++n) {
            if (lds_load(ctl) <= n) {
                WAIT_V(0); PUBLISH_PENDING;
                while (lds_load(ctl) <= n) __builtin_amdgcn_s_sleep(2);
            }
            FENCE;
            LAS unsigned* jw = ctl + 8 + 4 * (n & 3);
            const unsigned soff = lds_load(jw), fidx = lds_load(jw + 1), sc = lds_load(jw + 2), dq = lds_load(jw + 3);
            if (dq == 10u) break;
            if (dq >= 8u || dq == (unsigned)wid) {
                const LAS unsigned char* img = lds + (n & 3) * BLK + lane * 16;
#pragma unroll
                for (int g = 0; g < 4; ++g) {
                    u32x4 o[8];
#pragma unroll
                    for (int k = 0; k < 8; ++k) o[k] = *(const LAS u32x4*)(img + (g * 8 + k) * 1024);
                    if (g == 3) { WAIT_L(0); if (lane == 0) lds_store(ctl + wid, n + 1); }
#pragma unroll
                    for (int k = 0; k < 8; ++k) {
                        if (dq == 9u) __builtin_amdgcn_raw_buffer_store_b128(o[k], rsO, lane * 16, (int)(soff + (unsigned)(g * 8 + k) * (N * 4)), SLAB_AUX);
                        else __builtin_amdgcn_raw_buffer_store_b128(o[k], rsD, lane * 16, (int)(soff + (unsigned)(g * 8 + k) * 1024u), ST_AUX);
                        WAIT_WIN;
                        if (g == 0 && k == WIN - 1) PUBLISH_PENDING; }
                }
                if (dq != 9u) { pidx = (int)fidx; psc = (int)sc; }
            } else if (lane == 0) lds_store(ctl + wid, n + 1);
        }
        WAIT_V(0); PUBLISH_PENDING;
        if (own < 0 && lane == 0) __hip_atomic_store((gu32*)(p.ws[wid] + OFF_DONE + cw * 128) + ((8 - wid) & 7), call, RLX_SYS);
        return;
    }
    const unsigned tbase = (call - 1u) * DRAWS - (unsigned)(NSTAT * NCW);
    gu32* tick = (gu32*)(p.ws[0] + OFF_TICKET);
    unsigned n = own < 0 ? 1u : 0u;
    auto slot_free = [&]() {
        while (n >= 4u) { const unsigned c = __hip_atomic_load(ctl + 1 + (lane % 7), __ATOMIC_RELAXED, __HIP_MEMORY_SCOPE_WORKGROUP);
            if (__all(c + 3u >= n)) break; __builtin_amdgcn_s_sleep(2); }
        FENCE;
    };
    auto post = [&](unsigned soff, unsigned fidx, unsigned sc, unsigned dq) {
        if (lane == 0) { LAS unsigned* jw = ctl + 8 + 4 * (n & 3); jw[0] = soff; jw[1] = fidx; jw[2] = sc; jw[3] = dq; }
        WAIT_L(0);
        ++n; if (lane == 0) lds_store(ctl, n);
    };
    unsigned t1 = own < 0 ? NCW + cw : 0, t2 = own < 0 ? 2 * NCW + cw : 1, drawn = 0;
    const unsigned jend = own < 0 ? J3 : 8 * (2 + DX);
    if (own >= 0) {
        const u64 t0 = __builtin_amdgcn_s_memrealtime();
        while ((unsigned)__builtin_amdgcn_readfirstlane(__hip_atomic_load(tick, RLX_AGENT)) - tbase < (unsigned)OWN_GO && !give_up(tmo, call, t0)) __builtin_amdgcn_s_sleep(32);
    }
#pragma unroll 1
    while (t1 < jend) {
        const bool dyn = own < 0 && n + 2u >= (unsigned)NSTAT;
        if (dyn && lane == 0) draw_async(drawn, tick);
        const Job cur = decode_job((int)t1, own, p.rank);
        if (cur.dq != 0) {
            u64 fl; const u64 t0 = __builtin_amdgcn_s_memrealtime();
            for (;;) {
                fl = __hip_atomic_load(flagq + cur.panel * 64 + cur.u, RLX_SYS);
                if ((unsigned)__builtin_amdgcn_readfirstlane((int)(fl >> 32)) == call) break;
                if (bad || give_up(tmo, call, t0)) { bad = true; break; }
                __builtin_amdgcn_s_sleep(2);
            }
            slot_free();
            const unsigned off = (unsigned)cur.panel * PANEL + (unsigned)cur.u * BLK;
#pragma unroll
            for (int k = 0; k < 32; ++k) __builtin_amdgcn_raw_ptr_buffer_load_lds(rsD, (LAS void*)(lds + (n & 3) * BLK + k * 1024), 16, lane * 16, (int)(off + k * 1024), 0, 17);
            WAIT_V(0);
            if (!bad) post((unsigned)cur.panel * PANEL + (unsigned)cur.u * 7u * BLK, (unsigned)(cur.panel * 64 + cur.u * 7), (unsigned)fl, cur.dq < 0 ? 8u : (unsigned)cur.dq);
        }
        WAIT_V(0); asm volatile("" : "+v"(drawn));
        t1 = t2; t2 = dyn ? __builtin_amdgcn_readfirstlane(drawn) - tbase : t2 + (own < 0 ? NCW : 1);
    }
    if (own >= 0) { slot_free(); post(0u, 0u, 0u, 10u); return; }
    u64 ready = 0;
    while (t1 < (unsigned)JTOT) {
        const bool pending = t2 < (unsigned)JTOT;
        if (pending && lane == 0) draw_async(drawn, tick);
        const int tile = (int)(t1 - J3) >> 3;
        if (!((ready >> tile) & 1u)) {
            gu32* cnt = (gu32*)(p.ws[0] + OFF_OWN + lane * 128);
            const u64 t0 = __builtin_amdgcn_s_memrealtime();
            for (;;) {
                ready = __ballot(__hip_atomic_load(cnt, RLX_AGENT) >= 8u * call);
                if ((ready >> tile) & 1u) break;
                if (give_up(tmo, call, t0)) { bad = true; ready = ~0ull; break; }
                __builtin_amdgcn_s_sleep(8);
            }
        }
        int pm, pn; tile_of(2, tile & 7, 0, tile >> 3, p.rank, pm, pn);
        const unsigned off = (unsigned)((pm * 256 + (int)((t1 - J3) & 7) * 32) * (N * 4) + pn * 1024);
        slot_free();
        if (!bad) {
#pragma unroll
            for (int k = 0; k < 32; ++k) __builtin_amdgcn_raw_ptr_buffer_load_lds(rsO, (LAS void*)(lds + (n & 3) * BLK + k * 1024), 16, lane * 16, (int)(off + (unsigned)k * (N * 4)), 0, 18);
        } else {
            for (int k = 0; k < 32; ++k) *(LAS u32x4*)(lds + (n & 3) * BLK + k * 1024 + lane * 16) = (u32x4){0x7fc00000u, 0x7fc00000u, 0x7fc00000u, 0x7fc00000u};
        }
        WAIT_V(0); asm volatile("" : "+v"(drawn));
        post(off, 0u, 0u, 9u);
        t1 = t2; t2 = pending ? __builtin_amdgcn_readfirstlane(drawn) - tbase : ~0u;
    }
    slot_free(); post(0u, 0u, 0u, 10u);
    {
        const u64 t0 = __builtin_amdgcn_s_memrealtime();
        gu32* dn = (gu32*)(p.ws[0] + OFF_DONE + cw * 128) + (lane & 7);
        for (;;) {
            const unsigned val = __hip_atomic_load(dn, RLX_SYS);
            if (__all((lane & 7) == 0 || val >= call)) break;
            if (give_up(tmo, call, t0)) { bad = true; break; }
            __builtin_amdgcn_s_sleep(4);
        }
        __builtin_amdgcn_fence(__ATOMIC_ACQUIRE, "");
        if (bad && lane == 0) ((float*)p.out[0])[(size_t)(NSH + 2 * p.rank) * 256 + cw] = __builtin_nanf("");
    }
}

__device__ __forceinline__ void gemm_role(const Params& p, LAS unsigned char* lds, int kind, int id, int row, unsigned call) {
    const int own_id = kind == 2 ? row * NOWN + id : -1, ntiles = kind < 2 ? ROUNDS : row ? 1 : 4;
    int pm, pn; tile_of(kind, id, row, kind == 2 ? row : 0, p.rank, pm, pn);
    const int tid = threadIdx.x, wid = __builtin_amdgcn_readfirstlane(tid >> 6), lane = tid & 63, wr = wid >> 2, wc = wid & 3;
    const unsigned par = call & 1u;
    const unsigned voff = (unsigned)wid * 2048u + (unsigned)lane * 16u;
    const unsigned ldsw = (unsigned)wid * 2048u;
    const int aoff = wr * 8192 + lane * 16, boff = wc * 4096 + lane * 16;
    gu32* tmo = (gu32*)(p.ws[0] + OFF_TMO);
    const __amdgpu_buffer_rsrc_t rsS = make_rsrc(p.ws[0] + OFF_STAGE + (size_t)par * STAGE_PAR, STAGE_PAR);
    const u32x4 dsC = {(unsigned)__builtin_amdgcn_readfirstlane((int)(u64)p.out[0]), (unsigned)__builtin_amdgcn_readfirstlane((int)((u64)p.out[0] >> 32)) & 0xffffu, OUT_BYTES, 0x00020000u};
    gu64* flags = (gu64*)(p.ws[0] + OFF_FLAG) + par * (NPAN * 64);
    bool bad = false;
#define SA_(b, h) (((b) * 2 + (h)) * HTB)
#define SB_(b, h) ((4 + (b) * 2 + (h)) * HTB)
#define STAGE(bufoff, soff) do { \
    __builtin_amdgcn_raw_ptr_buffer_load_lds(rsS, (LAS void*)(lds + (bufoff) + ldsw), 16, (int)voff, (int)(soff), 0, GLDS_AUX); \
    __builtin_amdgcn_raw_ptr_buffer_load_lds(rsS, (LAS void*)(lds + (bufoff) + ldsw), 16, (int)voff, (int)(soff), 1024, GLDS_AUX); } while (0)
#define LDA(dst, b, h) do { _Pragma("unroll") for (int m = 0; m < 4; ++m) _Pragma("unroll") for (int k = 0; k < 2; ++k) dst[m][k] = *(const LAS f16x8*)(lds + SA_(b, h) + aoff + m * 2048 + k * 1024); } while (0)
#define LDB(dst, b, h) do { _Pragma("unroll") for (int n = 0; n < 2; ++n) _Pragma("unroll") for (int k = 0; k < 2; ++k) dst[n][k] = *(const LAS f16x8*)(lds + SB_(b, h) + boff + n * 2048 + k * 1024); } while (0)
    if (wr == 1) __builtin_amdgcn_s_setprio(1);
#define ZERO4 ((f32x4){0.f, 0.f, 0.f, 0.f})
#define MF2(Z, i, R, m, n, Bt) do { if (Z) asm volatile("v_mfma_f32_16x16x32_f16 %0, %1, %2, 0\n\tv_mfma_f32_16x16x32_f16 %0, %3, %4, %0" : "+{" R "}"(acc[i]) : "v"(At[m][0]), "v"(Bt[n][0]), "v"(At[m][1]), "v"(Bt[n][1])); \
      else asm volatile("v_mfma_f32_16x16x32_f16 %0, %1, %2, %0\n\tv_mfma_f32_16x16x32_f16 %0, %3, %4, %0" : "+{" R "}"(acc[i]) : "v"(At[m][0]), "v"(Bt[n][0]), "v"(At[m][1]), "v"(Bt[n][1])); } while (0)
#define MMA2_0(Z) do { \
      MF2(Z, 0, "v[128:131]", 0, 0, B0); MF2(Z, 1, "v[132:135]", 0, 1, B0); MF2(Z, 8, "v[160:163]", 0, 0, B1); MF2(Z, 9, "v[164:167]", 0, 1, B1); \
      MF2(Z, 2, "v[136:139]", 1, 0, B0); MF2(Z, 3, "v[140:143]", 1, 1, B0); MF2(Z, 10, "v[168:171]", 1, 0, B1); MF2(Z, 11, "v[172:175]", 1, 1, B1); \
      MF2(Z, 4, "v[144:147]", 2, 0, B0); MF2(Z, 5, "v[148:151]", 2, 1, B0); MF2(Z, 12, "v[176:179]", 2, 0, B1); MF2(Z, 13, "v[180:183]", 2, 1, B1); \
      MF2(Z, 6, "v[152:155]", 3, 0, B0); MF2(Z, 7, "v[156:159]", 3, 1, B0); MF2(Z, 14, "v[184:187]", 3, 0, B1); MF2(Z, 15, "v[188:191]", 3, 1, B1); \
      } while (0)
#define MMA2_1(Z) do { \
      MF2(Z, 16, "v[192:195]", 0, 0, B0); MF2(Z, 17, "v[196:199]", 0, 1, B0); MF2(Z, 24, "v[224:227]", 0, 0, B1); MF2(Z, 25, "v[228:231]", 0, 1, B1); \
      MF2(Z, 18, "v[200:203]", 1, 0, B0); MF2(Z, 19, "v[204:207]", 1, 1, B0); MF2(Z, 26, "v[232:235]", 1, 0, B1); MF2(Z, 27, "v[236:239]", 1, 1, B1); \
      MF2(Z, 20, "v[208:211]", 2, 0, B0); MF2(Z, 21, "v[212:215]", 2, 1, B0); MF2(Z, 28, "v[240:243]", 2, 0, B1); MF2(Z, 29, "v[244:247]", 2, 1, B1); \
      MF2(Z, 22, "v[216:219]", 3, 0, B0); MF2(Z, 23, "v[220:223]", 3, 1, B0); MF2(Z, 30, "v[248:251]", 3, 0, B1); MF2(Z, 31, "v[252:255]", 3, 1, B1); \
      } while (0)
#define MMA2(Z, ai) MMA2_##ai(Z)
#define ST4(POL) "buffer_store_dwordx4 v[112:115], %[vo], %[rs], %[so] offen " POL "\n\ts_add_i32 %[t], %[so], 0x20000\n\tbuffer_store_dwordx4 v[116:119], %[vo], %[rs], %[t] offen " POL "\n\t" \
      "s_add_i32 %[t], %[so], 0x40000\n\tbuffer_store_dwordx4 v[120:123], %[vo], %[rs], %[t] offen " POL "\n\ts_add_i32 %[t], %[so], 0x60000\n\tbuffer_store_dwordx4 v[124:127], %[vo], %[rs], %[t] offen " POL "\n\t"
#define STORE_TEXT(r) ".irp j, 0, 1, 2, 3\n\tv_ldexp_f32 v[112 + 4 * \\j], v[" #r " + \\j], %[ex]\n\tv_ldexp_f32 v[113 + 4 * \\j], v[" #r " + 4 + \\j], %[ex]\n\t" \
      "v_ldexp_f32 v[114 + 4 * \\j], v[" #r " + 32 + \\j], %[ex]\n\tv_ldexp_f32 v[115 + 4 * \\j], v[" #r " + 36 + \\j], %[ex]\n\t.endr\n\t" \
      "s_cmp_lt_i32 %[ow], 0\n\ts_cbranch_scc1 1f\n\t" ST4("nt sc1") "s_branch 2f\n\t1:\n\t" ST4("nt") "2:\n\ts_nop 0"
#define STORE_GROUP(r, R0, R1, R2, R3, i, cbase, ex_) do { unsigned _t; asm volatile(STORE_TEXT(r) : [t] "=&s"(_t) \
      : "{" R0 "}"(acc[i]), "{" R1 "}"(acc[(i) + 1]), "{" R2 "}"(acc[(i) + 8]), "{" R3 "}"(acc[(i) + 9]), [ex] "v"(ex_), [vo] "v"(coff), [rs] "s"(dsC), [so] "s"(cbase), [ow] "s"(own_id) \
      : "scc", "memory", "v112", "v113", "v114", "v115", "v116", "v117", "v118", "v119", "v120", "v121", "v122", "v123", "v124", "v125", "v126", "v127"); } while (0)
#define STORE_00(cbase, ex) STORE_GROUP(128, "v[128:131]", "v[132:135]", "v[160:163]", "v[164:167]", 0, (cbase) + 0u * (N * 4), ex)
#define STORE_01(cbase, ex) STORE_GROUP(136, "v[136:139]", "v[140:143]", "v[168:171]", "v[172:175]", 2, (cbase) + 16u * (N * 4), ex)
#define STORE_02(cbase, ex) STORE_GROUP(144, "v[144:147]", "v[148:151]", "v[176:179]", "v[180:183]", 4, (cbase) + 32u * (N * 4), ex)
#define STORE_03(cbase, ex) STORE_GROUP(152, "v[152:155]", "v[156:159]", "v[184:187]", "v[188:191]", 6, (cbase) + 48u * (N * 4), ex)
#define STORE_10(cbase, ex) STORE_GROUP(192, "v[192:195]", "v[196:199]", "v[224:227]", "v[228:231]", 16, (cbase) + 128u * (N * 4), ex)
#define STORE_11(cbase, ex) STORE_GROUP(200, "v[200:203]", "v[204:207]", "v[232:235]", "v[236:239]", 18, (cbase) + 144u * (N * 4), ex)
#define STORE_12(cbase, ex) STORE_GROUP(208, "v[208:211]", "v[212:215]", "v[240:243]", "v[244:247]", 20, (cbase) + 160u * (N * 4), ex)
#define STORE_13(cbase, ex) STORE_GROUP(216, "v[216:219]", "v[220:223]", "v[248:251]", "v[252:255]", 22, (cbase) + 176u * (N * 4), ex)
#define ST_L do { const int _e = -esc; STORE_00(cC, _e); STORE_01(cC, _e); STORE_02(cC, _e); } while (0)
#define ST_F1 do { STORE_03(cPrev, eprev); STORE_10(cPrev, eprev); STORE_11(cPrev, eprev); } while (0)
#define ST_F2 do { STORE_12(cPrev, eprev); STORE_13(cPrev, eprev); } while (0)
#define POLL(a, b, pos2, maskv, expv) do { const u64 _t0 = __builtin_amdgcn_s_memrealtime(); bool _ok = false; \
      for (int _n = 0;; ++_n) { \
        if (_n == 0 || (lane >> 4) == ((pos2) >> 3)) {     \
          const u64 _fa = __hip_atomic_load(flags + (a) * 64 + lane, RLX_SYS), _fb = __hip_atomic_load(flags + (NPA + (b)) * 64 + lane, RLX_SYS); \
          _ok = (unsigned)(_fa >> 32) == call && (unsigned)(_fb >> 32) == call; \
          expv = ((int)_fa == ZSENT || (int)_fb == ZSENT) ? ZSENT : (int)_fa + (int)_fb; } \
        maskv = __ballot(_ok); \
        if (((maskv >> (2 * (pos2))) & 3ull) == 3ull) break; \
        if (bad || ((_n & 63) == 63 && give_up(tmo, call, _t0))) { bad = true; maskv = ~0ull; expv = ZSENT; break; } \
        __builtin_amdgcn_s_sleep(16); } WAIT_L(0); } while (0)
#define SET_CMASK cmask = __ballot(ecur != esc && ecur != ZSENT)
#define RESCALE(pos_) do { int _d; asm volatile( \
        "s_bitcmp1_b64 %[cm], %[pos]\n\t" \
        "s_cbranch_scc0 1f\n\t" \
        "s_nop 15\n\t" \
        "v_readlane_b32 %[d], %[ec], %[pos]\n\t" \
        "s_nop 3\n\t" \
        "v_cmp_ne_u32 %[cm], %[d], %[ec]\n\t" \
        "v_cmp_ne_u32 vcc, 2.0, %[ec]\n\t" \
        "s_xor_b32 %[es], %[es], %[d]\n\t" \
        "s_xor_b32 %[d], %[es], %[d]\n\t" \
        "s_xor_b32 %[es], %[es], %[d]\n\t" \
        "s_sub_i32 %[d], %[es], %[d]\n\t" \
        "s_and_b64 %[cm], %[cm], vcc\n\t" \
        ".set gr_, 128\n\t" \
        ".rept 128\n\t" \
        "v_ldexp_f32 v[gr_], v[gr_], %[d]\n\t" \
        ".set gr_, gr_ + 1\n\t" \
        ".endr\n\t" \
        "s_nop 3\n\t" \
        "1:\n\t" \
        : [cm] "+s"(cmask), [es] "+s"(esc), [d] "=&s"(_d), \
          "+{v[128:131]}"(acc[0]), "+{v[132:135]}"(acc[1]), "+{v[136:139]}"(acc[2]), "+{v[140:143]}"(acc[3]), "+{v[144:147]}"(acc[4]), "+{v[148:151]}"(acc[5]), "+{v[152:155]}"(acc[6]), "+{v[156:159]}"(acc[7]), \
          "+{v[160:163]}"(acc[8]), "+{v[164:167]}"(acc[9]), "+{v[168:171]}"(acc[10]), "+{v[172:175]}"(acc[11]), "+{v[176:179]}"(acc[12]), "+{v[180:183]}"(acc[13]), "+{v[184:187]}"(acc[14]), "+{v[188:191]}"(acc[15]), \
          "+{v[192:195]}"(acc[16]), "+{v[196:199]}"(acc[17]), "+{v[200:203]}"(acc[18]), "+{v[204:207]}"(acc[19]), "+{v[208:211]}"(acc[20]), "+{v[212:215]}"(acc[21]), "+{v[216:219]}"(acc[22]), "+{v[220:223]}"(acc[23]), \
          "+{v[224:227]}"(acc[24]), "+{v[228:231]}"(acc[25]), "+{v[232:235]}"(acc[26]), "+{v[236:239]}"(acc[27]), "+{v[240:243]}"(acc[28]), "+{v[244:247]}"(acc[29]), "+{v[248:251]}"(acc[30]), "+{v[252:255]}"(acc[31]) \
        : [pos] "s"(pos_), [ec] "v"(ecur) : "vcc", "scc"); } while (0)
#define SET_NRDY do { nrdy = ~rmask ? __builtin_ctzll(~rmask) >> 1 : 32; evt = min(min(30, nrdy - 2), pubit); } while (0)
#define FETCH_FLAGS(a, b) do { const gu64* _pa = flags + (a) * 64 + lane; const gu64* _pb = flags + (NPA + (b)) * 64 + lane; \
      asm volatile("global_load_dwordx2 %0, %2, off sc0 sc1\n\tglobal_load_dwordx2 %1, %3, off sc0 sc1" : "=&v"(nfa), "=&v"(nfb) : "v"(_pa), "v"(_pb) : "memory"); } while (0)
#define BODY(FIRST, LAST, EXTRA) do { \
      const int pp = it, ppn = (pp + 1) & 31; \
      const unsigned a1 = cA + (unsigned)(pp * 2 + 1) * BLK; \
      const unsigned a2 = ((LAST) ? nA : cA) + (unsigned)(ppn * 2) * BLK, b2 = ((LAST) ? nB : cB) + (unsigned)(ppn * 2) * BLK; \
      const unsigned a3 = a2 + BLK, b3 = b2 + BLK; \
      LDB(B0, 0, 0); LDB(B1, 0, 1); SCHED; LDA(At, 0, 0); STAGE(SA_(1, 1), a1 + HTB); FENCE; \
      if (FIRST) { const int e = __builtin_amdgcn_readlane(ecur, 2 * pp); esc = e == ZSENT ? 0 : e; SET_CMASK; } else RESCALE(2 * pp); \
      if ((FIRST) && ui > 0) { ST_F1; WAIT_V(32); } else WAIT_V(8); \
      WAIT_L(0); BAR; MMA2(FIRST, 0); BAR; FENCE; \
      LDA(At, 0, 1); FENCE; \
      if (LAST) { if (has_next) {     \
          asm volatile("" : "+v"(nfa), "+v"(nfb)); \
          rmask = __ballot((unsigned)(nfa >> 32) == call && (unsigned)(nfb >> 32) == call); \
          enext = ((int)nfa == ZSENT || (int)nfb == ZSENT) ? ZSENT : (int)nfa + (int)nfb; \
          if (((rmask >> (2 * ppn)) & 3ull) != 3ull) POLL(npm, npn, ppn, rmask, enext); SET_NRDY; } } \
      else if (__builtin_expect(ppn >= nrdy, 0)) { \
          if (!(FIRST)) {             \
            asm volatile("" : "+v"(nfa), "+v"(nfb)); \
            rmask = __ballot((unsigned)(nfa >> 32) == call && (unsigned)(nfb >> 32) == call); \
            ecur = ((int)nfa == ZSENT || (int)nfb == ZSENT) ? ZSENT : (int)nfa + (int)nfb; } \
          if (((rmask >> (2 * ppn)) & 3ull) != 3ull) POLL(pm, pn, ppn, rmask, ecur); \
          SET_CMASK; SET_NRDY; } \
      STAGE(SB_(0, 0), b2); STAGE(SB_(0, 1), b2 + HTB); STAGE(SA_(0, 0), a2); \
      if ((FIRST) && ui > 0) { FENCE; ST_F2; WAIT_V(40); } else WAIT_V(8); \
      WAIT_L(0); BAR; MMA2(FIRST, 1); BAR; FENCE; \
      LDB(B0, 1, 0); LDB(B1, 1, 1); SCHED; LDA(At, 1, 0); STAGE(SA_(0, 1), a2 + HTB); FENCE; \
      RESCALE(2 * pp + 1); \
      if ((FIRST) && ui > 0) WAIT_V(28); else WAIT_V(8); \
      WAIT_L(0); BAR; MMA2(0, 0); BAR; FENCE; \
      LDA(At, 1, 1); FENCE; EXTRA; STAGE(SB_(1, 0), b3); STAGE(SB_(1, 1), b3 + HTB); STAGE(SA_(1, 0), a3); \
      if (LAST) { FENCE; ST_L; WAIT_V(20); } else if ((FIRST) && ui > 0) WAIT_V(16); else WAIT_V(8); \
      WAIT_L(0); BAR; MMA2(0, 1); BAR; FENCE; } while (0)
    f32x4 acc[32];
#pragma unroll
    for (int i = 0; i < 32; ++i) acc[i] = ZERO4;
    f16x8 At[4][2], B0[2][2], B1[2][2];
    unsigned cA = (unsigned)pm * PANEL, cB = (unsigned)(NPA + pn) * PANEL;
    const unsigned coff = (unsigned)((wr * 64 + (lane >> 4) * 4) * (N * 4) + (wc * 64 + (lane & 15) * 4) * 4);
    unsigned cC = (unsigned)pm * 256u * (N * 4) + (unsigned)pn * 1024u, cPrev = cC;
    u64 rmask, cmask = 0, nfa = 0, nfb = 0; int ecur, enext = 0, esc = 0, eprev = 0;
    int nrdy, evt, pubit = 64;
    POLL(pm, pn, 0, rmask, ecur); SET_NRDY;
    STAGE(SB_(0, 0), cB); STAGE(SB_(0, 1), cB + HTB); STAGE(SA_(0, 0), cA); STAGE(SA_(0, 1), cA + HTB);
    if (wr == 1) BAR;
    WAIT_V(2); BAR;
    STAGE(SB_(1, 0), cB + BLK); STAGE(SA_(1, 0), cA + BLK); STAGE(SB_(1, 1), cB + BLK + HTB);
    WAIT_V(6); BAR; FENCE;
#pragma unroll 1
    for (int ui = 0; ui < ntiles; ++ui) {
        const bool has_next = ui + 1 < ntiles;
        int npm = pm, npn = pn; unsigned nA = 0, nB = 0;
        int it = 0;
        BODY(1, 0, { if (nrdy <= 2) FETCH_FLAGS(pm, pn); });
#pragma unroll 1
        for (it = 1; it < 31; ++it) {
            BODY(0, 0, {
                if (__builtin_expect(it >= evt, 0)) {
                    if (it == 30) { if (has_next) tile_of(kind, id, row, ui + 1, p.rank, npm, npn); FETCH_FLAGS(npm, npn); pubit = own_id >= 0 ? 1 : 64; }
                    else if (it + 2 >= nrdy) FETCH_FLAGS(pm, pn);
                    if (it == pubit) {
                        WAIT_V(0);
                        if (lane == 0) __hip_atomic_fetch_add((gu32*)(p.ws[0] + OFF_OWN + ((ui - 1) * NOWN + own_id) * 128), 1u, RLX_AGENT);
                        pubit = 64; evt = min(30, nrdy - 2);
                    }
                } });
        }
        nA = (unsigned)npm * PANEL; nB = (unsigned)(NPA + npn) * PANEL;
        BODY(0, 1, {});
        cPrev = cC; eprev = -esc; ecur = enext;
        cA = nA; cB = nB; pm = npm; pn = npn;
        cC = (unsigned)pm * 256u * (N * 4) + (unsigned)pn * 1024u;
    }
    asm volatile("s_nop 15" ::: "memory"); SCHED;
    ST_F1; ST_F2;
    WAIT_V(0);
    if (bad) { *(float*)(p.out[0] + cPrev + coff) = __builtin_nanf(""); WAIT_V(0); }
    if (own_id >= 0) {
        if (lane == 0) __hip_atomic_fetch_add((gu32*)(p.ws[0] + OFF_OWN + ((ntiles - 1) * NOWN + own_id) * 128), 1u, RLX_AGENT);
    }
    if (wr == 0) BAR;
    BAR;
}

__global__ __launch_bounds__(512, 2) void fused_kernel(Params p) {
    extern __shared__ __attribute__((aligned(16))) unsigned char shm[];
    LAS unsigned char* lds = (LAS unsigned char*)shm;
    gu32* cc = (gu32*)(p.ws[0] + OFF_CALL + blockIdx.x * 64);
    const int x = blockIdx.x & 7, o = blockIdx.x >> 3;
    unsigned call;
    if ((int)blockIdx.x < NCW) {
        const int wid = __builtin_amdgcn_readfirstlane(threadIdx.x >> 6), lane = threadIdx.x & 63;
        LAS unsigned* ctl = (LAS unsigned*)(lds + 4 * BLK);
        const Job j = decode_job(blockIdx.x, -1, 0);
        f32x4 v[8];
        block_loads(p, j, wid, lane, v);
        call = (unsigned)__builtin_amdgcn_readfirstlane(__hip_atomic_load(cc, RLX_AGENT)) + 1u;
        const unsigned pmx = piece_max(v);
        if (lane == 0) ctl[32 + wid] = pmx;
        WAIT_V(0); WAIT_L(0); BAR; FENCE;
        const unsigned mx = block_max(ctl + 32, lane);
        const int s = scale_exp(mx);
        block_pack(j, wid, v, s, [&](u32x4 o, int d) { *(LAS u32x4*)(lds + d + lane * 16) = o; });
        if (threadIdx.x == 0) { ctl[8] = (unsigned)j.panel * PANEL + (unsigned)j.u * 7u * BLK; ctl[9] = (unsigned)(j.panel * 64 + j.u * 7); ctl[10] = (unsigned)(mx ? s : ZSENT); ctl[11] = 8u; }
        if (threadIdx.x == 0) __hip_atomic_store(cc, call, RLX_AGENT);
    } else {
        unsigned drawn = 0;
        if (threadIdx.x == 0) draw_async(drawn, (gu32*)(p.ws[0] + OFF_TICKET + 128));
        call = (unsigned)__builtin_amdgcn_readfirstlane(__hip_atomic_load(cc, RLX_AGENT)) + 1u;
        WAIT_V(0); asm volatile("" : "+v"(drawn)); BAR;
        if (threadIdx.x == 0) __hip_atomic_store(cc, call, RLX_AGENT);
        convert_phase(p, lds, drawn, call);
    }
    call = (unsigned)__builtin_amdgcn_readfirstlane((int)call);
    int kind = 0, id = (x & 1) * 7 + ((o - 4) >> 2), row = (x >> 1) * 4 + (o & 3), snd = -1;
    if (o < 3) kind = -1;
    else if (o == 3) { kind = 2; id = x; row = 0; snd = DX ? -1 : x; }
    else if (id >= NSTR) { if (o & 1) { kind = 2; id = (x & 6) + ((o >> 1) & 1); row = 4; snd = id; } else kind = 1; }
    if ((int)blockIdx.x < NCW || snd >= 0) {
        comm_role(p, lds, blockIdx.x, snd, call);
        WAIT_L(0); BAR; FENCE;
    }
    if (kind < 0) return;
    gemm_role(p, lds, kind, id, row, call);
}

}

extern "C" void kernel_launch(int rank, int nranks, ncclComm_t comm, hipStream_t stream,
                              void* const* d_in, const int* in_sizes, int n_in,
                              void* d_out, int out_size, const int* in_dtypes, int out_dtype,
                              void* const* d_out_all, void* const* d_ws_all, size_t ws_bytes) {
    if (nranks != NR || (unsigned)rank >= (unsigned)NR || ws_bytes < WS_NEED || n_in < 2 ||
        in_sizes[0] != M * KS || in_sizes[1] != KS * N || out_size != M * N) {
        fprintf(stderr, "kernel_launch: unexpected problem shape\n");
        return;
    }
    Params p;
    p.x = (const float*)d_in[0]; p.w = (const float*)d_in[1]; p.rank = rank;
    for (int q = 0; q < NR; ++q) { p.ws[q] = (char*)d_ws_all[(rank + q) % NR]; p.out[q] = (char*)d_out_all[(rank + q) % NR]; }
    p.out[0] = (char*)d_out;
    HIP_CHECK(hipFuncSetAttribute((const void*)fused_kernel, hipFuncAttributeMaxDynamicSharedMemorySize, LDS_BYTES));
    hipLaunchKernelGGL(fused_kernel, dim3(256), dim3(512), LDS_BYTES, stream, p);
    HIP_CHECK(hipGetLastError());
}
```
